```python
import numpy as np

kernel_name = "gpu_ag_gemm_m4096_k4096_n4096_w8a8_dqrelu"


ABSMAX_RTOL = 0.001

COMPUTE_DTYPE = "int8"
EPILOGUE = "dequant_relu"
IN_QUANT = "int8"
OUT_QUANT = None
IN_DTYPES = {"x": "int8", "w_mat": "int8",
             "scale_x": "float32", "scale_w": "float32"}
WORLD_SIZE = 8

def setup_inputs():

    x = np.random.randint(-127, 128, size=(4096, 4096)).astype(np.int8)
    w_mat = np.random.randint(-127, 128, size=(4096, 4096)).astype(np.int8)
    scale_x = (0.02 * (1.0 + np.random.rand(1))).astype(np.float32)
    scale_w = (0.02 * (1.0 + np.random.rand(1))).astype(np.float32)
    return {"x": x, "w_mat": w_mat, "scale_x": scale_x, "scale_w": scale_w}

def reference(x, w_mat, scale_x, scale_w):
    acc = x.astype(np.float64) @ w_mat.astype(np.float64)
    y = (acc.astype(np.float32) * (scale_x * scale_w)).astype(np.float32)
    return np.maximum(y, 0.0).astype(np.float32)
```

```cpp
#include <hip/hip_runtime.h>
#include <rccl/rccl.h>
#include <cstdio>
#include <utility>

#define HIP_CHECK(expr) do { hipError_t _e = (expr); if (_e != hipSuccess) { \
    fprintf(stderr, "HIP error %s at %s:%d\n", hipGetErrorString(_e), __FILE__, __LINE__); \
    return; } } while (0)

typedef unsigned u32x4 __attribute__((ext_vector_type(4)));
typedef int i32x4 __attribute__((ext_vector_type(4)));
typedef int i32x2 __attribute__((ext_vector_type(2)));
typedef float f32x4 __attribute__((ext_vector_type(4)));
#define LAS __attribute__((address_space(3)))

constexpr int NR = 8;
constexpr int MS = 512;
constexpr int KD = 4096;
constexpr int NL = 512;
constexpr size_t SHARD = (size_t)MS * KD;
constexpr size_t CNT_OFF = 0;
constexpr size_t TMO_OFF = 16 << 10;
constexpr size_t TS_OFF = 32 << 10;
constexpr size_t FLAG_OFF = 128 << 10;
constexpr size_t DATA_OFF = 1u << 20;
constexpr size_t WS_NEED = DATA_OFF + 2 * NR * SHARD;
constexpr unsigned long long DEADLINE = 20000000ull;
constexpr int W_LDS = KD * 32;
constexpr int LDS_BYTES = W_LDS + 64;

struct Args {
  const char* x; const char* w; const float* sx; const float* sw; float* out;
  char* ws[NR]; int rank; int pad;
};

template <int N> __device__ __forceinline__ void waitvm() { asm volatile("s_waitcnt vmcnt(%0)" ::"n"(N) : "memory"); }
template <typename F, int... I> __device__ __forceinline__ void sfor_impl(F&& f, std::integer_sequence<int, I...>) { (f(std::integral_constant<int, I>{}), ...); }
template <int N, typename F> __device__ __forceinline__ void sfor(F&& f) { sfor_impl(f, std::make_integer_sequence<int, N>{}); }
template <int OFF> __device__ __forceinline__ void gload(u32x4& d, const char* p) {
  asm volatile("global_load_dwordx4 %0, %1, off offset:%2" : "=v"(d) : "v"(p), "n"(OFF) : "memory"); }
__device__ __forceinline__ void gstore_wt(char* q, const u32x4& v) {
  asm volatile("global_store_dwordx4 %0, %1, off sc0 sc1\n\ts_nop 1" ::"v"(q), "v"(v) : "memory"); }
__device__ __forceinline__ void fstore(unsigned* fp, unsigned v) {
  asm volatile("global_store_dword %0, %1, off sc0 sc1" ::"v"(fp), "v"(v) : "memory"); }

template <int L, int NU, int D, int F> struct Sched {
  static constexpr int before_iter(int c) {
    int n = (D < NU ? D : NU) * L;
    for (int i = 0; i < c; ++i) { if (i + D < NU) n += L; n += L; if (i - F >= 0) n += 1; }
    return n;
  }
  static constexpr int load_end(int c) { return c < D ? (c + 1) * L : before_iter(c - D) + L; }
  static constexpr int store_end(int c) { return before_iter(c) + (c + D < NU ? L : 0) + L; }
  static constexpr int wait_load(int c) { return before_iter(c) + (c + D < NU ? L : 0) - load_end(c); }
  static constexpr int wait_store_in(int c) { return store_end(c) - store_end(c - F); }
  static constexpr int wait_store_tail(int f) { return before_iter(NU) + (f - (NU - F > 0 ? NU - F : 0)) - store_end(f); }
};

template <int L, int NU, int D, int F>
__device__ __forceinline__ void pusher(const char* x, char* pws, int rank, unsigned* cnt, int w, int lane) {
  using S = Sched<L, NU, D, F>;
  constexpr int NW = 2048 / (L * NU), CPG = 64 / L, UPX = 4 * CPG;
  u32x4 buf[D + 1][L];
  auto ld = [&](auto ic) { constexpr int c = ic;
    const int u = w + c * NW, xx = u & 7, pc = u >> 3, cc = pc % CPG, g = pc / CPG;
    const char* p = x + (size_t)(g * 128 + xx * 16 + (lane & 15)) * KD + cc * (L * 64) + (lane >> 4) * 16;
    sfor<L>([&](auto ii) { constexpr int i = ii; gload<i * 64>(buf[c % (D + 1)][i], p); }); };
  unsigned cv;
  asm volatile("global_load_dword %0, %1, off sc1" : "=v"(cv) : "v"(cnt) : "memory");
  sfor<(D < NU ? D : NU)>([&](auto ic) { ld(ic); });
  asm volatile("s_waitcnt vmcnt(%1)" : "+v"(cv) : "n"((D < NU ? D : NU) * L) : "memory");
  const unsigned call = __builtin_amdgcn_readfirstlane(cv) + 1;
  char* dst = pws + DATA_OFF + (size_t)(call & 1) * (NR * SHARD) + (size_t)rank * SHARD + lane * 16;
  unsigned* flags = (unsigned*)(pws + FLAG_OFF) + rank * (8 * UPX);
  auto fl = [&](int c) { const int u = w + c * NW;
    if (lane == 0) fstore(flags + (u & 7) * UPX + (u >> 3), call); };
  sfor<NU>([&](auto ic) { constexpr int c = ic;
    if constexpr (c + D < NU) ld(std::integral_constant<int, c + D>{});
    waitvm<S::wait_load(c)>();
    char* q = dst + (size_t)(w + c * NW) * (L * 1024);
    sfor<L>([&](auto ii) { constexpr int i = ii; gstore_wt(q + i * 1024, buf[c % (D + 1)][i]); });
    if constexpr (c - F >= 0) { waitvm<S::wait_store_in(c)>(); fl(c - F); }
  });
  sfor<(F < NU ? F : NU)>([&](auto ic) { constexpr int f = (NU - F > 0 ? NU - F : 0) + ic;
    waitvm<S::wait_store_tail(f)>(); fl(f); });
  if (lane == 0) __hip_atomic_store(cnt, call, __ATOMIC_RELAXED, __HIP_MEMORY_SCOPE_AGENT);
}

template <int L, int NU, int D, int F, int OST>
__global__ void __launch_bounds__(320, 2) ag_gemm(Args a) {
  extern __shared__ __attribute__((aligned(16))) char lds[];
  constexpr int NW = 2048 / (L * NU), CPG = 64 / L, UPX = 4 * CPG;
  static_assert(UPX <= 64, "one flag per lane");
  const int lane = threadIdx.x & 63, wid = __builtin_amdgcn_readfirstlane(threadIdx.x >> 6);
  const int b = blockIdx.x;
  const unsigned rank = (unsigned)a.rank;
  if (rank >= NR) return;
  char* myws = a.ws[rank];
  unsigned* cnt = (unsigned*)(myws + CNT_OFF) + b * 8 + wid;
  volatile LAS unsigned* wdone = (volatile LAS unsigned*)(LAS char*)(lds + W_LDS);
  if (threadIdx.x == 0) *wdone = 0;
  asm volatile("s_waitcnt lgkmcnt(0)" ::: "memory");
  __builtin_amdgcn_s_barrier();

  if (wid == 4) {
    if (b < 7 * NW) {
      const unsigned peer = (rank + 1 + b % 7) % NR;
      pusher<L, NU, D, F>(a.x, a.ws[peer], rank, cnt, b / 7, lane);
    }
    return;
  }

  const int xx = b & 7, idx = b >> 3, ns = idx & 15, h = idx >> 4;
  const unsigned src = (rank + 4 * h + wid) % NR;
  const bool own = (src == rank);
  unsigned call = __hip_atomic_load(cnt, __ATOMIC_RELAXED, __HIP_MEMORY_SCOPE_AGENT) + 1;
  {
    const char* wsrc = a.w + (size_t)(wid * 1024 + (lane >> 1)) * NL + ns * 32 + (((lane & 1) ^ ((lane >> 5) & 1)) * 16);
#pragma unroll
    for (int i = 0; i < 32; ++i)
      __builtin_amdgcn_global_load_lds((const void*)(wsrc + (size_t)i * 32 * NL), (LAS void*)(LAS char*)(lds + wid * 32768 + i * 1024), 16, 0, 0);
  }
  call = __builtin_amdgcn_readfirstlane(call);
  const float scale = a.sx[0] * a.sw[0];
  asm volatile("s_waitcnt vmcnt(0)" ::: "memory");
  if (lane == 0) __hip_atomic_fetch_add((LAS unsigned*)wdone, 1u, __ATOMIC_RELEASE, __HIP_MEMORY_SCOPE_WORKGROUP);
  if (lane == 0) __hip_atomic_store(cnt, call, __ATOMIC_RELAXED, __HIP_MEMORY_SCOPE_AGENT);

  const char* abase = own ? a.x : myws + DATA_OFF + (size_t)(call & 1) * (NR * SHARD) + (size_t)src * SHARD;
  const __amdgpu_buffer_rsrc_t rs = __builtin_amdgcn_make_buffer_rsrc((void*)abase, 0, (int)SHARD, 0x00020000);
  const int voff = own ? (xx * 16 + (lane & 15)) * KD + (lane >> 4) * 16 : xx * (L * 1024) + lane * 16;
  const int sc = own ? 64 * L : 8 * L * 1024, si = own ? 64 : 1024;
  unsigned* flags = (unsigned*)(myws + FLAG_OFF) + src * (8 * UPX) + xx * UPX + (lane < UPX ? lane : 0);
  unsigned* tmo = (unsigned*)(myws + TMO_OFF);
  const unsigned wb0 = (unsigned)(size_t)(LAS char*)lds + (16 * (lane >> 4) + ((lane & 15) >> 1)) * 32 + 8 * (lane & 1) + 16 * ((lane >> 4) & 1);
  const unsigned wb1 = wb0 ^ 16;
  int ready = own ? UPX : 0;
  bool bad = false;

  while (__hip_atomic_load((LAS unsigned*)wdone, __ATOMIC_ACQUIRE, __HIP_MEMORY_SCOPE_WORKGROUP) < 4) __builtin_amdgcn_s_sleep(1);

  auto count_ready = [&](unsigned v) {
    unsigned long long m = __ballot(v >= call);
    if (UPX < 64) m &= (1ull << (UPX & 63)) - 1;
    return m == ~0ull ? 64 : (int)__builtin_ctzll(~m); };
  auto loadu = [&](i32x4* dstv, int u) {
    const int so = (u / CPG) * (128 * KD) + (u % CPG) * sc;
#pragma unroll
    for (int i = 0; i < L; ++i) dstv[i] = __builtin_bit_cast(i32x4, __builtin_amdgcn_raw_buffer_load_b128(rs, voff, so + i * si, 17)); };
  i32x4 acc0 = {0, 0, 0, 0}, acc1 = {0, 0, 0, 0};
  auto comp = [&](const i32x4* xv, int u) {
    const unsigned kb = (u % CPG) * (L * 2048);
#pragma unroll
    for (int i = 0; i < L; ++i) {
      i32x2 w00 = __builtin_amdgcn_ds_read_tr8_b64_v2i32((LAS i32x2*)(wb0 + kb + i * 2048));
      i32x2 w01 = __builtin_amdgcn_ds_read_tr8_b64_v2i32((LAS i32x2*)(wb0 + kb + i * 2048 + 256));
      i32x2 w10 = __builtin_amdgcn_ds_read_tr8_b64_v2i32((LAS i32x2*)(wb1 + kb + i * 2048));
      i32x2 w11 = __builtin_amdgcn_ds_read_tr8_b64_v2i32((LAS i32x2*)(wb1 + kb + i * 2048 + 256));
      i32x4 wa = {w00[0], w00[1], w01[0], w01[1]}, wbv = {w10[0], w10[1], w11[0], w11[1]};
      acc0 = __builtin_amdgcn_mfma_i32_16x16x64_i8(wa, xv[i], acc0, 0, 0, 0);
      acc1 = __builtin_amdgcn_mfma_i32_16x16x64_i8(wbv, xv[i], acc1, 0, 0, 0);
    }
    if (u % CPG == CPG - 1) {
      const int row = src * MS + (u / CPG) * 128 + xx * 16 + (lane & 15);
      float* o = a.out + (size_t)row * NL + ns * 32 + 4 * (lane >> 4);
      f32x4 y0, y1;
#pragma unroll
      for (int r = 0; r < 4; ++r) { y0[r] = fmaxf((float)acc0[r] * scale, 0.f); y1[r] = fmaxf((float)acc1[r] * scale, 0.f); }
      if (bad) { const float qn = __builtin_nanf(""); y0 = f32x4{qn, qn, qn, qn}; y1 = y0; }
      if (OST == 0) { *(f32x4*)o = y0; *(f32x4*)(o + 16) = y1; }
      else if (OST == 1) { __builtin_nontemporal_store(y0, (f32x4*)o); __builtin_nontemporal_store(y1, (f32x4*)(o + 16)); }
      else { gstore_wt((char*)o, __builtin_bit_cast(u32x4, y0)); gstore_wt((char*)(o + 16), __builtin_bit_cast(u32x4, y1)); }
      acc0 = i32x4{0, 0, 0, 0}; acc1 = i32x4{0, 0, 0, 0};
    } };

  i32x4 xa[L], xb[L];
  int u = 0;
  while (u < UPX) {
    if (ready <= u) {
      const unsigned long long t0 = __builtin_amdgcn_s_memrealtime();
      while (true) {
        ready = count_ready(__hip_atomic_load(flags, __ATOMIC_RELAXED, __HIP_MEMORY_SCOPE_SYSTEM));
        if (ready > u) break;
        if (__hip_atomic_load(tmo, __ATOMIC_RELAXED, __HIP_MEMORY_SCOPE_AGENT) == call ||
            __builtin_amdgcn_s_memrealtime() - t0 > DEADLINE) {
          __hip_atomic_store(tmo, call, __ATOMIC_RELAXED, __HIP_MEMORY_SCOPE_AGENT);
          bad = true; ready = UPX; break;
        }
        __builtin_amdgcn_s_sleep(1);
      }
      asm volatile("" ::: "memory");
    }
    if (ready > u + 1) {
      loadu(xa, u); loadu(xb, u + 1);
      unsigned fv = 0;
      if (ready < UPX) fv = __hip_atomic_load(flags, __ATOMIC_RELAXED, __HIP_MEMORY_SCOPE_SYSTEM);
      comp(xa, u); comp(xb, u + 1);
      if (ready < UPX) ready = count_ready(fv);
      u += 2;
    } else {
      loadu(xa, u);
      unsigned fv = 0;
      if (ready < UPX) fv = __hip_atomic_load(flags, __ATOMIC_RELAXED, __HIP_MEMORY_SCOPE_SYSTEM);
      comp(xa, u);
      if (ready < UPX) ready = count_ready(fv);
      u += 1;
    }
    asm volatile("" ::: "memory");
  }
}

extern "C" void kernel_launch(int rank, int nranks, ncclComm_t comm, hipStream_t stream,
                              void* const* d_in, const int* in_sizes, int n_in,
                              void* d_out, int out_size, const int* in_dtypes, int out_dtype,
                              void* const* d_out_all, void* const* d_ws_all, size_t ws_bytes) {
  if (nranks != NR || ws_bytes < WS_NEED) { fprintf(stderr, "unexpected configuration\n"); return; }
  auto kfn = ag_gemm<8, 32, 3, 2, 0>;
  static bool init = false;
  if (!init) {
    HIP_CHECK(hipFuncSetAttribute((const void*)kfn, hipFuncAttributeMaxDynamicSharedMemorySize, LDS_BYTES));
    init = true;
  }
  Args a;
  a.x = (const char*)d_in[0]; a.w = (const char*)d_in[1];
  a.sx = (const float*)d_in[2]; a.sw = (const float*)d_in[3];
  a.out = (float*)d_out;
  for (int p = 0; p < NR; ++p) a.ws[p] = (char*)d_ws_all[p];
  a.rank = rank; a.pad = 0;
  hipLaunchKernelGGL(kfn, dim3(256), dim3(320), LDS_BYTES, stream, a);
  HIP_CHECK(hipGetLastError());
}
```

```cpp
#include <hip/hip_runtime.h>
#include <rccl/rccl.h>
#include <cstdio>
#include <utility>

#define HIP_CHECK(expr) do { hipError_t _e = (expr); if (_e != hipSuccess) { \
    fprintf(stderr, "HIP error %s at %s:%d\n", hipGetErrorString(_e), __FILE__, __LINE__); \
    return; } } while (0)

typedef unsigned u32x4 __attribute__((ext_vector_type(4)));
typedef int i32x4 __attribute__((ext_vector_type(4)));
typedef int i32x2 __attribute__((ext_vector_type(2)));
typedef float f32x4 __attribute__((ext_vector_type(4)));
#define LAS __attribute__((address_space(3)))

constexpr int NR = 8;
constexpr int MS = 512;
constexpr int KD = 4096;
constexpr int NL = 512;
constexpr size_t SHARD = (size_t)MS * KD;
constexpr size_t CNT_OFF = 0;
constexpr size_t TMO_OFF = 16 << 10;
constexpr size_t DUMMY_OFF = 20 << 10;
constexpr size_t TS_OFF = 32 << 10;
constexpr size_t FLAG_OFF = 128 << 10;
constexpr size_t DATA_OFF = 1u << 20;
constexpr size_t WS_NEED = DATA_OFF + 2 * NR * SHARD;
constexpr unsigned long long DEADLINE = 20000000ull;
constexpr int W_LDS = KD * 32;
constexpr int LDS_BYTES = W_LDS;
constexpr int L = 4;
constexpr int CPG = 64 / L, UPX = 4 * CPG, NUNITS = 2048 / L;

struct Args {
  const char* x; const char* w; const float* sx; const float* sw; float* out;
  char* ws[NR]; int rank; int pad;
};

template <int N> __device__ __forceinline__ void waitvm() { asm volatile("s_waitcnt vmcnt(%0)" ::"n"(N) : "memory"); }
template <typename F, int... I> __device__ __forceinline__ void sfor_impl(F&& f, std::integer_sequence<int, I...>) { (f(std::integral_constant<int, I>{}), ...); }
template <int N, typename F> __device__ __forceinline__ void sfor(F&& f) { sfor_impl(f, std::make_integer_sequence<int, N>{}); }
template <int OFF> __device__ __forceinline__ void gload(u32x4& d, const char* p) {
  asm volatile("global_load_dwordx4 %0, %1, off offset:%2" : "=v"(d) : "v"(p), "n"(OFF) : "memory"); }
template <int OFF> __device__ __forceinline__ void gstore_wt(char* q, const u32x4& v) {
  asm volatile("global_store_dwordx4 %0, %1, off offset:%2 sc0 sc1\n\ts_nop 1" ::"v"(q), "v"(v), "n"(OFF) : "memory"); }
__device__ __forceinline__ void fstore(unsigned* fp, unsigned v) {
  asm volatile("global_store_dword %0, %1, off sc0 sc1" ::"v"(fp), "v"(v) : "memory"); }

template <int D, int F, int NW>
__device__ __forceinline__ void pusher(const char* x, char* pws, int rank, unsigned* cnt, unsigned* dummy, int w, int lane) {
  u32x4 buf[D + 1][L];
  auto src_of = [&](int u) { u = u < NUNITS ? u : w; const int xx = u & 7, pc = u >> 3, cc = pc % CPG, g = pc / CPG;
    return x + (size_t)(g * 128 + xx * 16 + (lane & 15)) * KD + cc * (L * 64) + (lane >> 4) * 16; };
  unsigned cv;
  asm volatile("global_load_dword %0, %1, off sc1" : "=v"(cv) : "v"(cnt) : "memory");
  sfor<D>([&](auto ic) { constexpr int c = ic; const char* p = src_of(w + c * NW);
    sfor<L>([&](auto ii) { constexpr int i = ii; gload<i * 64>(buf[c][i], p); }); });
  asm volatile("s_waitcnt vmcnt(0)" : "+v"(cv) :: "memory");
  const unsigned call = __builtin_amdgcn_readfirstlane(cv) + 1;
  char* dst = pws + DATA_OFF + (size_t)(call & 1) * (NR * SHARD) + (size_t)rank * SHARD + lane * 16;
  unsigned* flags = (unsigned*)(pws + FLAG_OFF) + rank * NUNITS;
  int u = w;
  while (true) {
    bool done = false;
    sfor<D + 1>([&](auto ic) { constexpr int c = ic;
      if (!done) {
        const char* p = src_of(u + D * NW);
        char* q = dst + (size_t)u * (L * 1024);
        if constexpr (F == 0 && D == 1) {
          sfor<L>([&](auto ii) { constexpr int i = ii; gstore_wt<i * 1024>(q, buf[c][i]); });
          sfor<L>([&](auto ii) { constexpr int i = ii; gload<i * 64>(buf[(c + D) % (D + 1)][i], p); });
          waitvm<0>();
        } else {
          sfor<L>([&](auto ii) { constexpr int i = ii; gload<i * 64>(buf[(c + D) % (D + 1)][i], p); });
          waitvm<D * (2 * L + 1)>();
          sfor<L>([&](auto ii) { constexpr int i = ii; gstore_wt<i * 1024>(q, buf[c][i]); });
          waitvm<F * (2 * L + 1)>();
        }
        const int fu = u - F * NW;
        if (lane == 0) fstore(fu >= 0 ? flags + (fu & 7) * UPX + (fu >> 3) : dummy, call);
        u += NW;
        done = u >= NUNITS;
      } });
    if (done) break;
  }
  if constexpr (F > 0) {
    waitvm<0>();
    for (int f = F; f >= 1; --f) { const int fu = u - f * NW; if (lane == 0 && fu >= 0) fstore(flags + (fu & 7) * UPX + (fu >> 3), call); }
  }
  if (lane == 0) __hip_atomic_store(cnt, call, __ATOMIC_RELAXED, __HIP_MEMORY_SCOPE_AGENT);
}

template <int D, int F, int NP, int PW, int NG, int CW, int OST, int MODE>
__global__ void __launch_bounds__(64 * CW, 2) ag_gemm(Args a) {
  extern __shared__ __attribute__((aligned(16))) char lds[];
  const int lane = threadIdx.x & 63, wid = __builtin_amdgcn_readfirstlane(threadIdx.x >> 6);
  const int b = blockIdx.x;
  const unsigned rank = (unsigned)a.rank;
  if (rank >= NR) return;
  char* myws = a.ws[rank];
  unsigned* cnt = (unsigned*)(myws + CNT_OFF) + b * 8 + wid;

  if (b < NP) {
    constexpr int NW = NP * PW / 7;
    const int gw = b * PW + wid;
    if (wid < PW && gw < 7 * NW && !(MODE & 2)) {
      const unsigned peer = (rank + 1 + gw % 7) % NR;
      pusher<D, F, NW>(a.x, a.ws[peer], rank, cnt, (unsigned*)(myws + DUMMY_OFF) + gw, gw / 7, lane);
    }
    return;
  }

  const int cb = b - NP;
  const int q = (cb & 7) * (NG * 2) + (cb >> 3), grp = q >> 4, ns = q & 15;
  const int task = wid * NG + grp;
  const bool own = task >= 56;
  const unsigned src = own ? rank : (rank + 1 + task % 7) % NR;
  const int xx = own ? (task - 56) & 7 : task / 7;
  unsigned call = __hip_atomic_load(cnt, __ATOMIC_RELAXED, __HIP_MEMORY_SCOPE_AGENT) + 1;
  const float scale = a.sx[0] * a.sw[0];
  if (wid < 4) {
    const char* wsrc = a.w + (size_t)(wid * 1024 + (lane >> 1)) * NL + ns * 32 + (((lane & 1) ^ ((lane >> 5) & 1)) * 16);
#pragma unroll 8
    for (int i = 0; i < 32; ++i)
      __builtin_amdgcn_global_load_lds((const void*)(wsrc + (size_t)i * 32 * NL), (LAS void*)(LAS char*)(lds + wid * 32768 + i * 1024), 16, 0, 0);
  }
  asm volatile("s_waitcnt vmcnt(0)" ::: "memory");
  call = __builtin_amdgcn_readfirstlane(call);
  if (lane == 0) __hip_atomic_store(cnt, call, __ATOMIC_RELAXED, __HIP_MEMORY_SCOPE_AGENT);
  __builtin_amdgcn_s_barrier();
  asm volatile("" ::: "memory");
  if (task >= 64) return;

  const char* abase = own ? a.x : myws + DATA_OFF + (size_t)(call & 1) * (NR * SHARD) + (size_t)src * SHARD;
  const __amdgpu_buffer_rsrc_t rs = __builtin_amdgcn_make_buffer_rsrc((void*)abase, 0, (int)SHARD, 0x00020000);
  const int voff = own ? (xx * 16 + (lane & 15)) * KD + (lane >> 4) * 16 : xx * (L * 1024) + lane * 16;
  const int sc = own ? 64 * L : 8 * L * 1024, si = own ? 64 : 1024;
  unsigned* flags = (unsigned*)(myws + FLAG_OFF) + src * NUNITS + xx * UPX + lane;
  unsigned* tmo = (unsigned*)(myws + TMO_OFF);
  LAS char* wb0 = (LAS char*)lds + (16 * (lane >> 4) + ((lane & 15) >> 1)) * 32 + 8 * (lane & 1) + 16 * ((lane >> 4) & 1);
  LAS char* wb1 = (LAS char*)lds + (16 * (lane >> 4) + ((lane & 15) >> 1)) * 32 + 8 * (lane & 1) + 16 * (((lane >> 4) & 1) ^ 1);
  int ready = (own || (MODE & 1)) ? UPX : 0;
  bool bad = false;

  auto count_ready = [&](unsigned v) {
    const unsigned long long m = __ballot(v >= call);
    return m == ~0ull ? 64 : (int)__builtin_ctzll(~m); };
  auto loadu = [&](i32x4* dstv, int u) {
    const int so = (u / CPG) * (128 * KD) + (u % CPG) * sc;
#pragma unroll
    for (int i = 0; i < L; ++i) dstv[i] = __builtin_bit_cast(i32x4, __builtin_amdgcn_raw_buffer_load_b128(rs, voff, so + i * si, 17)); };
  i32x4 acc0 = {0, 0, 0, 0}, acc1 = {0, 0, 0, 0};
  auto comp = [&](const i32x4* xv, int u) {
    const unsigned kb = (u % CPG) * (L * 2048);
#pragma unroll
    for (int i = 0; i < L; ++i) {
      i32x2 w00 = __builtin_amdgcn_ds_read_tr8_b64_v2i32((LAS i32x2*)(wb0 + kb + i * 2048));
      i32x2 w01 = __builtin_amdgcn_ds_read_tr8_b64_v2i32((LAS i32x2*)(wb0 + kb + i * 2048 + 256));
      i32x2 w10 = __builtin_amdgcn_ds_read_tr8_b64_v2i32((LAS i32x2*)(wb1 + kb + i * 2048));
      i32x2 w11 = __builtin_amdgcn_ds_read_tr8_b64_v2i32((LAS i32x2*)(wb1 + kb + i * 2048 + 256));
      i32x4 wa = {w00[0], w00[1], w01[0], w01[1]}, wbv = {w10[0], w10[1], w11[0], w11[1]};
      acc0 = __builtin_amdgcn_mfma_i32_16x16x64_i8(wa, xv[i], acc0, 0, 0, 0);
      acc1 = __builtin_amdgcn_mfma_i32_16x16x64_i8(wbv, xv[i], acc1, 0, 0, 0);
    } };
  auto flush = [&](int g) {
      const int row = src * MS + g * 128 + xx * 16 + (lane & 15);
      float* o = a.out + (size_t)row * NL + ns * 32 + 4 * (lane >> 4);
      f32x4 y0, y1;
#pragma unroll
      for (int r = 0; r < 4; ++r) { y0[r] = fmaxf((float)acc0[r] * scale, 0.f); y1[r] = fmaxf((float)acc1[r] * scale, 0.f); }
      if (bad) { const float qn = __builtin_nanf(""); y0 = f32x4{qn, qn, qn, qn}; y1 = y0; }
      if (OST == 0) { *(f32x4*)o = y0; *(f32x4*)(o + 16) = y1; }
      else if (OST == 1) { __builtin_nontemporal_store(y0, (f32x4*)o); __builtin_nontemporal_store(y1, (f32x4*)(o + 16)); }
      else { gstore_wt<0>((char*)o, __builtin_bit_cast(u32x4, y0)); gstore_wt<64>((char*)o, __builtin_bit_cast(u32x4, y1)); }
      acc0 = i32x4{0, 0, 0, 0}; acc1 = i32x4{0, 0, 0, 0}; };

  i32x4 xv[4][L];
  int u = 0;
  while (u < UPX) {
    if (ready <= u) {
      const unsigned long long t0 = __builtin_amdgcn_s_memrealtime();
      for (unsigned it = 1;; ++it) {
        ready = count_ready(__hip_atomic_load(flags, __ATOMIC_RELAXED, __HIP_MEMORY_SCOPE_SYSTEM));
        if (ready > u) break;
        if ((it & 63) == 0 && (__hip_atomic_load(tmo, __ATOMIC_RELAXED, __HIP_MEMORY_SCOPE_AGENT) == call ||
            __builtin_amdgcn_s_memrealtime() - t0 > DEADLINE)) {
          __hip_atomic_store(tmo, call, __ATOMIC_RELAXED, __HIP_MEMORY_SCOPE_AGENT);
          bad = true; ready = UPX; break;
        }
        if (!(MODE & 4)) __builtin_amdgcn_s_sleep(1);
      }
      asm volatile("" ::: "memory");
    }
    int lim = (u | (CPG - 1)) + 1;
    lim = (ready < lim ? ready : lim) - u;
    unsigned fv = 0;
    if (lim >= 4) {
      loadu(xv[0], u); loadu(xv[1], u + 1); loadu(xv[2], u + 2); loadu(xv[3], u + 3);
      if (ready < UPX) fv = __hip_atomic_load(flags, __ATOMIC_RELAXED, __HIP_MEMORY_SCOPE_SYSTEM);
      comp(xv[0], u); comp(xv[1], u + 1); comp(xv[2], u + 2); comp(xv[3], u + 3);
      u += 4;
    } else if (lim >= 2) {
      loadu(xv[0], u); loadu(xv[1], u + 1);
      if (ready < UPX) fv = __hip_atomic_load(flags, __ATOMIC_RELAXED, __HIP_MEMORY_SCOPE_SYSTEM);
      comp(xv[0], u); comp(xv[1], u + 1);
      u += 2;
    } else {
      loadu(xv[0], u);
      if (ready < UPX) fv = __hip_atomic_load(flags, __ATOMIC_RELAXED, __HIP_MEMORY_SCOPE_SYSTEM);
      comp(xv[0], u);
      u += 1;
    }
    if (ready < UPX) ready = count_ready(fv);
    if (u % CPG == 0) flush(u / CPG - 1);
    asm volatile("" ::: "memory");
  }
}

extern "C" void kernel_launch(int rank, int nranks, ncclComm_t comm, hipStream_t stream,
                              void* const* d_in, const int* in_sizes, int n_in,
                              void* d_out, int out_size, const int* in_dtypes, int out_dtype,
                              void* const* d_out_all, void* const* d_ws_all, size_t ws_bytes) {
  if (nranks != NR || ws_bytes < WS_NEED) { fprintf(stderr, "unexpected configuration\n"); return; }
  constexpr int NP = 56, NG = 12, CW = 6;
  auto kfn = ag_gemm<1, 0, NP, 2, NG, CW, 2, 0>;
  static bool init = false;
  if (!init) {
    HIP_CHECK(hipFuncSetAttribute((const void*)kfn, hipFuncAttributeMaxDynamicSharedMemorySize, LDS_BYTES));
    init = true;
  }
  Args a;
  a.x = (const char*)d_in[0]; a.w = (const char*)d_in[1];
  a.sx = (const float*)d_in[2]; a.sw = (const float*)d_in[3];
  a.out = (float*)d_out;
  for (int p = 0; p < NR; ++p) a.ws[p] = (char*)d_ws_all[p];
  a.rank = rank; a.pad = 0;
  hipLaunchKernelGGL(kfn, dim3(NP + NG * 16), dim3(64 * CW), LDS_BYTES, stream, a);
  HIP_CHECK(hipGetLastError());
}
```

```cpp
#include <hip/hip_runtime.h>
#include <rccl/rccl.h>
#include <cstdio>
#include <utility>

#define HIP_CHECK(expr) do { hipError_t _e = (expr); if (_e != hipSuccess) { \
    fprintf(stderr, "HIP error %s at %s:%d\n", hipGetErrorString(_e), __FILE__, __LINE__); \
    return; } } while (0)

typedef unsigned u32x4 __attribute__((ext_vector_type(4)));
typedef int i32x4 __attribute__((ext_vector_type(4)));
typedef int i32x2 __attribute__((ext_vector_type(2)));
typedef float f32x4 __attribute__((ext_vector_type(4)));
typedef unsigned long long u64;
#define LAS __attribute__((address_space(3)))

constexpr int NR = 8;
constexpr int MS = 512;
constexpr int KD = 4096;
constexpr int NL = 512;
constexpr size_t SHARD = (size_t)MS * KD;
constexpr size_t CNT_OFF = 0;
constexpr size_t TMO_OFF = 16 << 10;
constexpr size_t TK_OFF = 20 << 10;
constexpr size_t TS_OFF = 32 << 10;
constexpr size_t XF_OFF = 128 << 10;
constexpr size_t WF_OFF = 144 << 10;
constexpr size_t PF_OFF = 160 << 10;
constexpr size_t LF_OFF = 192 << 10;
constexpr size_t PK_OFF = 2u << 20;
constexpr size_t PL_OFF = 4u << 20;
constexpr size_t WQ_OFF = 6u << 20;
constexpr size_t XG_OFF = 10u << 20;
constexpr size_t WS_NEED = XG_OFF + 2 * NR * SHARD;
constexpr u64 DEADLINE = 20000000ull;
constexpr int W_LDS = KD * 32;
constexpr int CW = 16;
constexpr int SCR = 1088;
constexpr int LDS_BYTES = W_LDS + CW * SCR + 16;
constexpr int CAP = 320;

struct Args {
  const char* x; const char* w; const float* sx; const float* sw;
  char* ws[NR]; float* out[NR]; int rank; int pad;
  unsigned tblO[48], tblR[32];
};

template <typename F, int... I> __device__ __forceinline__ void sfor_impl(F&& f, std::integer_sequence<int, I...>) { (f(std::integral_constant<int, I>{}), ...); }
template <int N, typename F> __device__ __forceinline__ void sfor(F&& f) { sfor_impl(f, std::make_integer_sequence<int, N>{}); }
__device__ __forceinline__ void waitvm0() { asm volatile("s_waitcnt vmcnt(0)" ::: "memory"); }
__device__ __forceinline__ void gload_s(u32x4& d, unsigned voff, const char* sb) {
  asm volatile("global_load_dwordx4 %0, %1, %2" : "=v"(d) : "v"(voff), "s"(sb) : "memory"); }
template <int OFF> __device__ __forceinline__ void gstore_s(unsigned voff, char* sb, const u32x4& v) {
  asm volatile("global_store_dwordx4 %0, %1, %2 offset:%3 sc0 sc1\n\ts_nop 1" ::"v"(voff), "v"(v), "s"(sb), "n"(OFF) : "memory"); }
template <int OFF> __device__ __forceinline__ void gstore_wt(char* q, const u32x4& v) {
  asm volatile("global_store_dwordx4 %0, %1, off offset:%2 sc0 sc1\n\ts_nop 1" ::"v"(q), "v"(v), "n"(OFF) : "memory"); }
__device__ __forceinline__ void fstore(unsigned* fp, unsigned v) {
  asm volatile("global_store_dword %0, %1, off sc0 sc1" ::"v"(fp), "v"(v) : "memory"); }
__device__ __forceinline__ unsigned fload(const unsigned* fp) { return __hip_atomic_load(fp, __ATOMIC_RELAXED, __HIP_MEMORY_SCOPE_SYSTEM); }
__device__ __forceinline__ unsigned mbcnt(u64 m, unsigned base) { return __builtin_amdgcn_mbcnt_hi((unsigned)(m >> 32), __builtin_amdgcn_mbcnt_lo((unsigned)m, base)); }
__device__ __forceinline__ u64 lomask(int n) { return n >= 64 ? ~0ull : (1ull << n) - 1; }

template <int NB, int G> constexpr int first_group() { return NB <= G ? NB : NB % G == 0 ? G : NB % G * 2 > G ? NB % G : G + NB % G; }
template <int NB, int G> __device__ __forceinline__ int xorder(unsigned u) {
  constexpr unsigned F = first_group<NB, G>();
  if (u < F * 16) return u % F * 16 + u / F;
  u -= F * 16;
  const unsigned gi = u / (16 * G), r = u % (16 * G);
  return (F + gi * G + r % G) * 16 + r / G;
}

struct Unit { char* q; unsigned* fp; u64 m[4]; u64 fm; };

template <int HB, int BB, int G, int U, int DIRB, int MODE>
__device__ __forceinline__ void pusher(const Args& a, unsigned rank, unsigned peer, unsigned* cnt, int w0, int nw, int lane, u64* tsp) {
  constexpr int YB = 32 - HB - BB, NTR = 4 * (HB + YB);
  char* myws = a.ws[rank]; char* pws = a.ws[peer];
  const bool cross = ((rank ^ peer) & 4) != 0;
  const int total = cross ? 128 + (YB + BB) * 16 : (HB + BB) * 16;
  const unsigned vx = (lane & 15) * KD + (lane >> 4) * 16;
  const unsigned vw = (lane >> 1) * NL + (((lane & 1) ^ (lane >> 5)) * 16);
  const unsigned vd = lane * 16;
  u32x4 buf[2][4];
  auto loads = [&](u32x4* d, const char* sb, unsigned vo, int st, u64 em) {
    if constexpr (MODE & 256)
    asm volatile("s_mov_b64 exec, %9\n\tglobal_load_dwordx4 %0, %4, %8\n\tglobal_load_dwordx4 %1, %5, %8\n\t"
                 "global_load_dwordx4 %2, %6, %8\n\tglobal_load_dwordx4 %3, %7, %8\n\ts_mov_b64 exec, -1"
                 : "+v"(d[0]), "+v"(d[1]), "+v"(d[2]), "+v"(d[3]) : "v"(vo), "v"(vo + st), "v"(vo + 2 * st), "v"(vo + 3 * st), "s"(sb), "s"(em) : "memory");
    else
    asm volatile("s_mov_b64 exec, %9\n\tglobal_load_dwordx4 %0, %4, %8 sc0 sc1\n\tglobal_load_dwordx4 %1, %5, %8 sc0 sc1\n\t"
                 "global_load_dwordx4 %2, %6, %8 sc0 sc1\n\tglobal_load_dwordx4 %3, %7, %8 sc0 sc1\n\ts_mov_b64 exec, -1"
                 : "+v"(d[0]), "+v"(d[1]), "+v"(d[2]), "+v"(d[3]) : "v"(vo), "v"(vo + st), "v"(vo + 2 * st), "v"(vo + 3 * st), "s"(sb), "s"(em) : "memory"); };
  auto id_of = [&](int u) {
    if (BB > 0 && u >= total - BB * 16) return (HB + YB) * 16 + xorder<BB ? BB : 1, G>(u - (total - BB * 16));
    return !cross ? xorder<HB, G>(u) : u < 128 ? 512 + u : HB * 16 + xorder<YB, G>(u - 128); };
  auto src_of = [&](int id) { return id >= 512 ? a.w + (size_t)(id - 512) * (32 * NL) + (peer & 3) * 128 : a.x + (size_t)(id >> 4) * (16 * KD) + (id & 15) * 256; };
  sfor<8>([&](auto ii) { constexpr int i = ii; buf[i >> 2][i & 3] = u32x4{0, 0, 0, 0}; });
  unsigned cv;
  asm volatile("global_load_dword %0, %1, off sc1" : "=v"(cv) : "v"(cnt) : "memory");
  int id = id_of(w0);
  loads(buf[0], src_of(id), id >= 512 ? vw : vx, id >= 512 ? 32 : 64, ~0ull);
  const int o = w0 + (lane >> 2) * nw;
  int code = ((const unsigned char*)a.tblR)[o < NTR ? o : 0];
  asm volatile("s_waitcnt vmcnt(0)" : "+v"(cv), "+v"(code) :: "memory");
  const unsigned ti = (((code & 31) * 4 + ((code >> 5) & 3)) * 4 + (rank & 3)) * 4 + (lane & 3), vfr = ti % U * 64 + ti / U * 4;
  u64 todo = cross && !(MODE & 4) ? __ballot(o < NTR && (code >> 5) != peer && !((code & 31) >= DIRB && (code & 31) < HB && (code >> 5) != rank)) : 0ull;
  const unsigned call = __builtin_amdgcn_readfirstlane(cv) + 1;
  char* dx = pws + XG_OFF + (size_t)(call & 1) * (NR * SHARD) + (size_t)rank * SHARD;
  char* dw = pws + WQ_OFF + (size_t)(call & 1) * SHARD + (size_t)(rank & 3) * (SHARD / 4) - 512 * 4096;
  unsigned* fx = (unsigned*)(pws + XF_OFF) + rank * 512;
  unsigned* fw = (unsigned*)(pws + WF_OFF) + (rank & 3) * 128 - 512;
  unsigned* tkp = (unsigned*)(myws + TK_OFF) + peer * 32;
  const unsigned* lf = (const unsigned*)(myws + LF_OFF) + ((peer & 3) * 32 + w0) * 64 + lane;
  unsigned* tmo = (unsigned*)(myws + TMO_OFF);
  const unsigned tbase = (call - 1) * (unsigned)(total - nw) - 2 * nw;
  auto xw_unit = [&](int i) { Unit r; r.q = (i >= 512 ? dw : dx) + (size_t)i * 4096;
    r.fp = (i >= 512 ? fw : fx) + i; r.m[0] = r.m[1] = r.m[2] = r.m[3] = ~0ull; r.fm = 1; return r; };
  Unit cur = xw_unit(id);
  unsigned fval = call, vf = 0, one = 1, tv = 0, pv = 0;
  int tk = w0 + nw;
  u64 rdy = 0;
  const u64 t0 = __builtin_amdgcn_s_memrealtime();
  for (unsigned it = 1;; ++it) {
    bool done = false;
    sfor<2>([&](auto ic) { constexpr int c = ic;
      if (!done) {
        asm volatile("s_mov_b64 exec, %6\n\tglobal_store_dwordx4 %0, %1, %5 sc0 sc1\n\t"
                     "s_mov_b64 exec, %7\n\tglobal_store_dwordx4 %0, %2, %5 offset:1024 sc0 sc1\n\t"
                     "s_mov_b64 exec, %8\n\tglobal_store_dwordx4 %0, %3, %5 offset:2048 sc0 sc1\n\t"
                     "s_mov_b64 exec, %9\n\tglobal_store_dwordx4 %0, %4, %5 offset:3072 sc0 sc1\n\ts_mov_b64 exec, -1\n\ts_nop 1"
                     :: "v"(vd), "v"(buf[c][0]), "v"(buf[c][1]), "v"(buf[c][2]), "v"(buf[c][3]), "s"(cur.q),
                        "s"(cur.m[0]), "s"(cur.m[1]), "s"(cur.m[2]), "s"(cur.m[3]) : "memory");
        Unit nxt; const char* sb; unsigned vo, nval, nvf; int st; u64 lm = ~0ull;
        const int lim = rdy ? total - BB * 16 : total;
        const bool draw = tk < lim;
        if (draw) { nvf = 0; const int i = id_of(tk); nxt = xw_unit(i); sb = src_of(i); vo = i >= 512 ? vw : vx; st = i >= 512 ? 32 : 64; nval = call; }
        else {
          vo = vd; st = 1024; nvf = vfr; nval = 2 * call + ((pv & 0xff) == 0 ? 1 : 0);
          if (rdy) {
            const int l0 = __builtin_ctzll(rdy);
            const int cd = __builtin_amdgcn_readlane(code, l0), ui = ((cd & 31) * 4 + ((cd >> 5) & 3)) * 4;
            sb = myws + PL_OFF + (size_t)(ui + (peer & 3)) * 4096;
            const int uo = ui + (rank & 3);
            nxt.q = pws + PK_OFF + (size_t)uo * 4096;
            nxt.fp = (unsigned*)(pws + PF_OFF);
            sfor<4>([&](auto ii) { constexpr int i = ii; nxt.m[i] = lomask((__builtin_amdgcn_readlane(pv, l0 + i) & 0xff) * 4); });
            nxt.fm = 0xfull << l0;
            todo &= ~nxt.fm;
          } else { sb = myws; nxt.q = myws; nxt.fp = tmo; nxt.m[0] = nxt.m[1] = nxt.m[2] = nxt.m[3] = 0; nxt.fm = 0; lm = 0; }
        }
        loads(buf[c ^ 1], sb, vo, st, lm);
        const u64 pm = tk + 2 * nw >= total - BB * 16 ? todo : 0ull;
        asm volatile("s_cmp_lt_i32 %4, %11\n\ts_cselect_b64 exec, 1, 0\n\tglobal_atomic_add %0, %2, %3, off sc0\n\t"
                     "s_mov_b64 exec, %6\n\tglobal_load_dword %1, %5, off sc0 sc1\n\ts_mov_b64 exec, -1\n\t"
                     "s_waitcnt vmcnt(0)\n\t"
                     "s_mov_b64 exec, %10\n\tglobal_store_dword %7, %8, %9 sc0 sc1\n\ts_mov_b64 exec, -1\n\ts_nop 1"
                     : "+v"(tv), "+v"(pv) : "v"(tkp), "v"(one), "s"(tk), "v"(lf), "s"(pm), "v"(vf), "v"(fval), "s"(cur.fp), "s"(cur.fm), "s"(lim) : "memory", "scc");
        if (draw) tk = (int)(__builtin_amdgcn_readfirstlane(tv) - tbase);
        const u64 bm = __ballot((pv >> 8) == (call & 0xffffff)) & pm;
        rdy = bm & (bm >> 1) & (bm >> 2) & (bm >> 3) & 0x1111111111111111ull;
        const bool idle = nxt.fm == 0;
        done = idle && !todo;
        if (idle && !rdy && !done) {
          __builtin_amdgcn_s_sleep(2);
          if ((it & 63) == 0 && (__hip_atomic_load(tmo, __ATOMIC_RELAXED, __HIP_MEMORY_SCOPE_AGENT) == call || __builtin_amdgcn_s_memrealtime() - t0 > DEADLINE)) {
            __hip_atomic_store(tmo, call, __ATOMIC_RELAXED, __HIP_MEMORY_SCOPE_AGENT); done = true; }
        }
        cur = nxt; fval = nval; vf = nvf;
      } });
    if (done) break;
  }
  if (lane == 0) __hip_atomic_store(cnt, call, __ATOMIC_RELAXED, __HIP_MEMORY_SCOPE_AGENT);
}

template <int HB, int BB, int G, int WI, int WX, int PW, int CO, int CR, int UW, int DIRB, int RAWB, int MODE>
__global__ void __launch_bounds__(64 * CW, 4) ag_gemm(Args a) {
  extern __shared__ __attribute__((aligned(16))) char lds[];
  constexpr int YB = 32 - HB - BB, NTO = 32 + 3 * (HB + BB) + 4 * (YB + BB), NTR = 4 * (HB + YB), NPW = 3 * WI + 4 * WX, NP = (NPW + PW - 1) / PW;
  static_assert(NTO <= 192, "table");
  constexpr int U = CO * 16 * UW;
  static_assert(U >= 128 && U <= 512 && WX >= 8 && WX <= 32, "flag layout");
  const int lane = threadIdx.x & 63, wid = __builtin_amdgcn_readfirstlane(threadIdx.x >> 6);
  const int b = blockIdx.x;
  const unsigned rank = (unsigned)a.rank;
  if (rank >= NR) return;
  char* myws = a.ws[rank];
  unsigned* cnt = (unsigned*)(myws + CNT_OFF) + b * CW + wid;

  if (b < NP) {
    if (wid < PW) {
      constexpr int WM = WI < WX ? WI : WX;
      int d, w0;
      if constexpr (MODE & 512) {
        static_assert(!(MODE & 512) || (WI % PW == 0 && WX % PW == 0), "");
        if (b < 7 * WM / PW) { d = 1 + b % 7; w0 = b / 7 * PW + wid; }
        else if (WI > WX) { d = 1 + (b - 7 * WM / PW) % 3; w0 = WM + (b - 7 * WM / PW) / 3 * PW + wid; }
        else { d = 4 + (b - 7 * WM / PW) % 4; w0 = WM + (b - 7 * WM / PW) / 4 * PW + wid; }
      } else if constexpr (MODE & 1024) {
        static_assert(!(MODE & 1024) || (3 * WI % PW == 0 && 4 * WX % PW == 0), "");
        constexpr int NI = 3 * WI / PW, NX = 4 * WX / PW, NM = NI < NX ? NI : NX;
        const bool in = b < 2 * NM ? ((b ^ (b >> 3)) & 1) != 0 : NI > NX;
        const int gw = (b < 2 * NM ? b >> 1 : b - NM) * PW + wid;
        if (in) { d = 1 + gw % 3; w0 = gw / 3; } else { d = 4 + gw % 4; w0 = gw / 4; }
      } else {
        const int gw = b * PW + wid;
        if (gw >= NPW) return;
        if (gw < 7 * WM) { d = 1 + gw % 7; w0 = gw / 7; }
        else if (WI > WX) { d = 1 + (gw - 7 * WM) % 3; w0 = WM + (gw - 7 * WM) / 3; }
        else { d = 4 + (gw - 7 * WM) % 4; w0 = WM + (gw - 7 * WM) / 4; }
      }
      pusher<HB, BB, G, U, DIRB, MODE>(a, rank, (rank ^ d) & 7, cnt, w0, d >= 4 ? WX : WI, lane, (u64*)(myws + TS_OFF) + 10240 + d * 64);
    }
    return;
  }

  if (MODE & 8) return;
  const int cb = b - NP;
  const bool ownw = (cb >> 3) < 2 * CO;
  const int q = ownw ? (cb & 7) * (2 * CO) + (cb >> 3) : CO * 16 + (cb & 7) * (2 * CR) + (cb >> 3) - 2 * CO, grp = q >> 4, ns = q & 15;
  const unsigned dest = ownw ? rank : ((rank ^ 4) & 4) | (ns >> 2);
  const int col0 = ownw ? ns * 32 : (rank & 3) * 128 + (ns & 3) * 32;
  unsigned call = __hip_atomic_load(cnt, __ATOMIC_RELAXED, __HIP_MEMORY_SCOPE_AGENT);
  const float scale = a.sx[0] * a.sw[0];
  unsigned* tmo = (unsigned*)(myws + TMO_OFF);
  auto expired = [&](u64 t0) {
    if (__hip_atomic_load(tmo, __ATOMIC_RELAXED, __HIP_MEMORY_SCOPE_AGENT) == call || __builtin_amdgcn_s_memrealtime() - t0 > DEADLINE) {
      __hip_atomic_store(tmo, call, __ATOMIC_RELAXED, __HIP_MEMORY_SCOPE_AGENT); return true; }
    return false; };
  LAS int* tick = (LAS int*)((LAS char*)lds + W_LDS + CW * SCR);
  LAS unsigned* wmask = (LAS unsigned*)tick + 1;
  if (threadIdx.x == 0) { *tick = ownw ? CW - UW : CW - 1; *wmask = 0; }
  if (ownw) {
    const char* wsrc = a.w + (size_t)(wid * 256 + (lane >> 1)) * NL + ns * 32 + (((lane & 1) ^ ((lane >> 5) & 1)) * 16);
#pragma unroll
    for (int i = 0; i < 8; ++i)
      __builtin_amdgcn_global_load_lds((const void*)(wsrc + (size_t)i * 32 * NL), (LAS void*)(LAS char*)(lds + wid * 8192 + i * 1024), 16, 0, 0);
    call = __builtin_amdgcn_readfirstlane(call) + 1;
  } else {
    const int su = wid * (CR * 16) + (grp - CO) * 16 + ns;
    u32x4 t[4];
    if (su < 512) { const char* sb = a.x + (size_t)(su >> 4) * (16 * KD) + (su & 15) * 256;
      sfor<4>([&](auto ii) { constexpr int i = ii; gload_s(t[i], (lane & 15) * KD + (lane >> 4) * 16, sb + i * 64); }); }
    asm volatile("s_waitcnt lgkmcnt(0)" ::: "memory");
    __builtin_amdgcn_s_barrier();
    asm volatile("s_waitcnt vmcnt(0)" : "+v"(t[0]), "+v"(t[1]), "+v"(t[2]), "+v"(t[3]) :: "memory");
    call = __builtin_amdgcn_readfirstlane(call) + 1;
    if (su < 512) { char* sq = myws + XG_OFF + (size_t)(call & 1) * (NR * SHARD) + (size_t)rank * SHARD + (size_t)su * 4096;
      sfor<4>([&](auto ii) { constexpr int i = ii; gstore_s<i * 1024>(lane * 16, sq, t[i]); });
      waitvm0();
      if (lane == 0) fstore((unsigned*)(myws + XF_OFF) + rank * 512 + su, call); }
  }
  if (lane == 0) __hip_atomic_store(cnt, call, __ATOMIC_RELAXED, __HIP_MEMORY_SCOPE_AGENT);
  if (ownw) {
    asm volatile("s_waitcnt vmcnt(0) lgkmcnt(0)" ::: "memory");
    __builtin_amdgcn_s_barrier();
    asm volatile("" ::: "memory");
  } else if (wid == CW - 1) {
    const char* wsrc = myws + WQ_OFF + (size_t)(call & 1) * SHARD + (size_t)(dest & 3) * (SHARD / 4) + (ns & 3) * 1024 + lane * 16;
    const unsigned* fw = (const unsigned*)(myws + WF_OFF) + (dest & 3) * 128 + lane;
    const u64 t0 = __builtin_amdgcn_s_memrealtime();
    __builtin_amdgcn_s_setprio(2);
    unsigned pub = 0;
    for (int h = 0; h < 2; ++h) {
      u64 got = 0;
      for (unsigned it = 1;; ++it) {
        const unsigned fl = fload(fw + h * 64);
        asm volatile("s_waitcnt vmcnt(0)" ::: "memory");
        const u64 g8 = got & (got >> 4), g2 = g8 & (g8 >> 2), g1 = g2 & (g2 >> 1);
        const unsigned cm = (h ? 0xff : 0) | (unsigned)__ballot((g1 >> ((lane & 7) * 8)) & 1 && lane < 8) << (h * 8);
        if (cm != pub) { pub = cm; if (lane == 0) __hip_atomic_store(wmask, cm, __ATOMIC_RELEASE, __HIP_MEMORY_SCOPE_WORKGROUP); }
        u64 m = __ballot(fl >= call) & ~got;
        if (!~got) break;
        if (h == 0 && !~(got | m)) it = 0;
        got |= m;
        const bool none = !m;
        for (; m; m &= m - 1) { const int kb = h * 64 + __builtin_ctzll(m);
          __builtin_amdgcn_global_load_lds((const void*)(wsrc + (size_t)kb * 4096), (LAS void*)(LAS char*)(lds + kb * 1024), 16, 0, 17); }
        if (it == 0) break;
        if ((it & 63) == 0 && expired(t0)) { h = 2; if (lane == 0) __hip_atomic_store(wmask, 0xffffu, __ATOMIC_RELEASE, __HIP_MEMORY_SCOPE_WORKGROUP); break; }
        if (none) __builtin_amdgcn_s_sleep(4);
      }
    }
    __builtin_amdgcn_s_setprio(0);
  }
  LAS char* scr = (LAS char*)lds + W_LDS + wid * SCR;

  if (ownw && wid >= CW - UW) {
    if (MODE & (4 | 32)) return;
    __builtin_amdgcn_s_setprio(2);
    const int uw = (grp * 16 + ns) * UW + (wid - (CW - UW));
    const int TI = (lane & 15) * U + uw;
    const bool valid = TI < (HB + YB) * 64 && !((TI >> 6) >= HB && ((TI >> 4) & 3) == (rank & 3));
    unsigned todo = (unsigned)__ballot(valid) & 0xffff;
    const unsigned* pf = (const unsigned*)(myws + PF_OFF) + uw * 16 + (lane & 15);
    const __amdgpu_buffer_rsrc_t rp = __builtin_amdgcn_make_buffer_rsrc((void*)(myws + PK_OFF), 0, 2048 * 1024, 0x00020000);
    float* out = a.out[rank];
    const u64 t0 = __builtin_amdgcn_s_memrealtime();
    bool bad = false;
    for (unsigned it = 1; todo; ++it) {
      const unsigned v = fload(pf);
      unsigned m = (unsigned)__ballot(v >= 2 * call) & todo;
      asm volatile("" ::: "memory");
      if (!m) {
        if ((it & 63) == 0 && expired(t0)) { bad = true; m = todo; }
        else { __builtin_amdgcn_s_sleep(2); continue; }
      }
      todo &= ~m;
      if (!bad) m &= ~(unsigned)__ballot(v & 1);
      while (m) {
        int ti[2]; u32x4 d[2];
#pragma unroll
        for (int s = 0; s < 2; ++s) { ti[s] = m ? __builtin_ctz(m) * U + uw : -1; m &= m - 1;
          d[s] = __builtin_amdgcn_raw_buffer_load_b128(rp, ti[s] >= 0 ? lane * 16 : 0x7fffffff, (ti[s] & 2047) * 1024, 17); }
#pragma unroll
        for (int s = 0; s < 2; ++s) if (ti[s] >= 0) {
          const int blk = ti[s] >> 6, src = (blk < HB ? (rank ^ 4) & 4 : rank & 4) | ((ti[s] >> 4) & 3);
          float* o = out + (size_t)(src * MS + blk * 16 + (lane & 15)) * NL + (ti[s] & 15) * 32 + 4 * (lane >> 4);
          f32x4 y[2];
          *(LAS u32x4*)(scr + lane * 16) = d[s];
          asm volatile("" ::: "memory");
          const unsigned kb = *(LAS unsigned char*)(scr + lane);
          u64 mk[8]; int n = 0;
#pragma unroll
          for (int j = 0; j < 8; ++j) { mk[j] = __ballot((kb >> j) & 1); n += __builtin_popcountll(mk[j]); }
          n = n > CAP ? CAP : n;
          int base = 0;
#pragma unroll
          for (int j = 0; j < 8; ++j) {
            unsigned pos = mbcnt(mk[j], base); pos = pos < CAP ? pos : 0;
            const float val = (float)(((int)*(LAS signed char*)(scr + 64 + 2 * n + pos) << 16) | (int)*(LAS unsigned short*)(scr + 64 + 2 * pos)) * scale;
            y[j >> 2][j & 3] = ((kb >> j) & 1) ? val : 0.f;
            base += __builtin_popcountll(mk[j]); }
          asm volatile("" ::: "memory");
          if (bad) { const float qn = __builtin_nanf(""); y[0] = f32x4{qn, qn, qn, qn}; y[1] = y[0]; }
          gstore_wt<0>((char*)o, __builtin_bit_cast(u32x4, y[0])); gstore_wt<64>((char*)o, __builtin_bit_cast(u32x4, y[1]));
        }
      }
    }
    return;
  }

  bool bad = false, wok = false;
  unsigned wm = ownw ? 0xffff : 0;
  LAS char* wb0 = (LAS char*)lds + (16 * (lane >> 4) + ((lane & 15) >> 1)) * 32 + 8 * (lane & 1) + 16 * ((lane >> 4) & 1);
  LAS char* wb1 = (LAS char*)lds + (16 * (lane >> 4) + ((lane & 15) >> 1)) * 32 + 8 * (lane & 1) + 16 * (((lane >> 4) & 1) ^ 1);
  auto next = [&]() { int t = 0; if (lane == 0) t = __hip_atomic_fetch_add(tick, 1, __ATOMIC_RELAXED, __HIP_MEMORY_SCOPE_WORKGROUP);
    return __builtin_amdgcn_readfirstlane(t); };
  auto wpoll = [&]() { return (unsigned)__builtin_amdgcn_readfirstlane(__hip_atomic_load(wmask, __ATOMIC_ACQUIRE, __HIP_MEMORY_SCOPE_WORKGROUP)); };
  for (int tk = !ownw && wid == CW - 1 ? next() : wid;; tk = next()) {
    const int pos = ownw ? tk * CO + grp : tk * CR + (grp - CO);
    if (pos >= (ownw ? NTO : NTR)) break;
    const int code = a.tblO[(pos >> 2) + (ownw ? 0 : 48)] >> (pos & 3) * 8;
    const unsigned src = (code >> 5) & 7; const int blk = code & 31;
    if (!ownw && src == dest) continue;
    const char* abase = myws + XG_OFF + (size_t)(call & 1) * (NR * SHARD) + (size_t)src * SHARD + (size_t)blk * (16 * KD);
    const __amdgpu_buffer_rsrc_t rs = __builtin_amdgcn_make_buffer_rsrc((void*)abase, 0, 16 * KD, 0x00020000);
    const unsigned* flags = (const unsigned*)(myws + XF_OFF) + src * 512 + blk * 16 + (lane & 15);
    unsigned todo = 0xffff, ready = (bad || (MODE & 1)) ? 0xffff : 0;
    auto loadu = [&](i32x4* dstv, int c) {
#pragma unroll
      for (int i = 0; i < 4; ++i) dstv[i] = __builtin_bit_cast(i32x4, __builtin_amdgcn_raw_buffer_load_b128(rs, lane * 16 + i * 1024, c << 12, 17)); };
    i32x4 acc[2] = {{0, 0, 0, 0}, {0, 0, 0, 0}};
    auto comp = [&](const i32x4* xv, int c) {
      const unsigned kb = c * 8192;
#pragma unroll
      for (int i = 0; i < 4; ++i) {
        i32x2 w00 = __builtin_amdgcn_ds_read_tr8_b64_v2i32((LAS i32x2*)(wb0 + kb + i * 2048));
        i32x2 w01 = __builtin_amdgcn_ds_read_tr8_b64_v2i32((LAS i32x2*)(wb0 + kb + i * 2048 + 256));
        i32x2 w10 = __builtin_amdgcn_ds_read_tr8_b64_v2i32((LAS i32x2*)(wb1 + kb + i * 2048));
        i32x2 w11 = __builtin_amdgcn_ds_read_tr8_b64_v2i32((LAS i32x2*)(wb1 + kb + i * 2048 + 256));
        i32x4 wa = {w00[0], w00[1], w01[0], w01[1]}, wbv = {w10[0], w10[1], w11[0], w11[1]};
        acc[0] = __builtin_amdgcn_mfma_i32_16x16x64_i8(wa, xv[i], acc[0], 0, 0, 0);
        acc[1] = __builtin_amdgcn_mfma_i32_16x16x64_i8(wbv, xv[i], acc[1], 0, 0, 0);
      } };
    i32x4 xv[4][4];
    while (todo) {
      if (wm != 0xffff) wm = wpoll();
      unsigned m = ready & todo & wm;
      if (!m) {
        const u64 t0 = __builtin_amdgcn_s_memrealtime();
        for (unsigned it = 1;; ++it) {
          if (todo & wm & ~ready) ready = (unsigned)__ballot(fload(flags) >= call) & 0xffff;
          m = ready & todo & wm;
          if (m) break;
          if ((it & 63) == 0 && expired(t0)) { bad = true; ready = wm = 0xffff; m = todo; break; }
          if (todo == 0xffff) __builtin_amdgcn_s_sleep(8); else __builtin_amdgcn_s_sleep(1);
          if (wm != 0xffff) wm = wpoll();
        }
        asm volatile("" ::: "memory");
      }
      const int n = __builtin_popcount(m);
      const int c0 = __builtin_ctz(m); const unsigned m1 = m & (m - 1);
      unsigned fv = 0;
      if (n >= 4) {
        const int c1 = __builtin_ctz(m1); const unsigned m2 = m1 & (m1 - 1);
        const int c2 = __builtin_ctz(m2); const unsigned m3 = m2 & (m2 - 1);
        const int c3 = __builtin_ctz(m3);
        loadu(xv[0], c0); loadu(xv[1], c1); loadu(xv[2], c2); loadu(xv[3], c3);
        if (ready != 0xffff) fv = fload(flags);
        comp(xv[0], c0); comp(xv[1], c1); comp(xv[2], c2); comp(xv[3], c3);
        todo &= ~(m ^ (m3 & (m3 - 1)));
      } else if (n >= 2) {
        const int c1 = __builtin_ctz(m1);
        loadu(xv[0], c0); loadu(xv[1], c1);
        if (ready != 0xffff) fv = fload(flags);
        comp(xv[0], c0); comp(xv[1], c1);
        todo &= ~(m ^ (m1 & (m1 - 1)));
      } else {
        loadu(xv[0], c0);
        if (ready != 0xffff) fv = fload(flags);
        comp(xv[0], c0);
        todo &= ~m;
      }
      if (ready != 0xffff) ready = (unsigned)__ballot(fv >= call) & 0xffff;
      asm volatile("" ::: "memory");
    }
    if (!ownw && !wok) { wok = true; bad |= __hip_atomic_load(tmo, __ATOMIC_RELAXED, __HIP_MEMORY_SCOPE_AGENT) == call; }
    f32x4 y[2];
#pragma unroll
    for (int j = 0; j < 8; ++j) y[j >> 2][j & 3] = fmaxf((float)acc[j >> 2][j & 3] * scale, 0.f);
    const bool late = blk < HB && src != rank;
    const bool direct = late && blk >= DIRB;
    const int ui = (blk * 4 + (src & 3)) * 4;
    int n64 = 0;
    if (!ownw && !bad && !(late && blk >= RAWB)) {
      u64 mk[8]; int n = 0; unsigned kb = 0, big = 0;
#pragma unroll
      for (int j = 0; j < 8; ++j) { const bool k = y[j >> 2][j & 3] > 0.f; mk[j] = __ballot(k); n += __builtin_popcountll(mk[j]);
        kb |= k ? 1u << j : 0u; big |= k ? (unsigned)(acc[j >> 2][j & 3] + (1 << 23)) : 0u; }
      if (n <= CAP && !__ballot(big >= (1u << 24))) {
        int base = 0;
#pragma unroll
        for (int j = 0; j < 8; ++j) {
          const unsigned pos = mbcnt(mk[j], base);
          const bool k = (kb >> j) & 1;
          *(LAS short*)(scr + (k ? 64 + 2 * pos : 1024)) = (short)acc[j >> 2][j & 3];
          *(LAS char*)(scr + (k ? 64 + 2 * n + pos : 1024)) = (char)(acc[j >> 2][j & 3] >> 16);
          base += __builtin_popcountll(mk[j]); }
        *(LAS char*)(scr + lane) = (char)kb;
        asm volatile("" ::: "memory");
        n64 = (64 + 3 * n + 63) >> 6;
        char* pk = (direct ? a.ws[dest] + PK_OFF + (size_t)(ui + (rank & 3)) * 4096 : myws + PL_OFF + (size_t)(ui + (dest & 3)) * 4096) + (ns & 3) * 1024 + lane * 16;
        const u32x4 d0 = *(LAS u32x4*)(scr + lane * 16);
        if (lane < n64 * 4) gstore_wt<0>(pk, d0);
        asm volatile("" ::: "memory");
      }
    }
    if (n64 == 0) {
      if (bad) { const float qn = __builtin_nanf(""); y[0] = f32x4{qn, qn, qn, qn}; y[1] = y[0]; }
      float* o = a.out[dest] + (size_t)(src * MS + blk * 16 + (lane & 15)) * NL + col0 + 4 * (lane >> 4);
      gstore_wt<0>((char*)o, __builtin_bit_cast(u32x4, y[0])); gstore_wt<64>((char*)o, __builtin_bit_cast(u32x4, y[1]));
    }
    if (!ownw) {
      const int ti = (ui + (rank & 3)) * 4 + (ns & 3), o = pos;
      unsigned* fp = direct ? (unsigned*)(a.ws[dest] + PF_OFF) + (ti % U) * 16 + ti / U
                            : (unsigned*)(myws + LF_OFF) + ((dest & 3) * 32 + o % WX) * 64 + (o / WX) * 4 + (ns & 3);
      const unsigned fv = direct ? 2 * call + (n64 == 0 ? 1 : 0) : (call << 8) | n64;
      waitvm0();
      if (lane == 0) fstore(fp, fv);
    }
  }
}

template <int HB, int BB, int G> static void make_table(unsigned char* to, unsigned char* tr, int rank, float wdelay) {
  constexpr int YB = 32 - HB - BB;
  int n = 0, m = 0;
  for (int j = 0; j < 32; ++j) { to[n++] = rank << 5 | j; if (j < HB) tr[m++] = rank << 5 | j; }
  int hi = 0, yi = 0;
  while (hi < HB + BB || yi < YB + BB) {
    const float th = hi < HB + BB ? (float)hi : 1e9f, ty = yi < YB + BB ? wdelay + yi : 1e9f;
    if (th <= ty) { const int e = hi + (hi == 0 ? first_group<HB, G>() : hi == HB ? first_group<BB, G>() : G);
      for (; hi < e; ++hi) for (int d = 1; d < 4; ++d) { const int c = (rank ^ d) << 5 | (hi < HB ? hi : hi + YB); to[n++] = c; if (hi < HB) tr[m++] = c; } }
    else { const int e = yi + (yi == 0 ? first_group<YB, G>() : yi == YB ? first_group<BB, G>() : G);
      for (; yi < e; ++yi) for (int d = 4; d < 8; ++d) { const int c = (rank ^ d) << 5 | (HB + yi); to[n++] = c; if (yi < YB) tr[m++] = c; } }
  }
  while (n < 192) to[n++] = 0;
  while (m < 128) tr[m++] = 0;
}

extern "C" void kernel_launch(int rank, int nranks, ncclComm_t comm, hipStream_t stream,
                              void* const* d_in, const int* in_sizes, int n_in,
                              void* d_out, int out_size, const int* in_dtypes, int out_dtype,
                              void* const* d_out_all, void* const* d_ws_all, size_t ws_bytes) {
  if (nranks != NR || (unsigned)rank >= NR || ws_bytes < WS_NEED) { fprintf(stderr, "unexpected configuration\n"); return; }
  constexpr int HB = 23, BB = 0, G = 4, WI = 18, WX = 18, PW = 2, CO = 6, CR = 6;
  constexpr int GRID = (3 * WI + 4 * WX + PW - 1) / PW + (CO + CR) * 16;
  static_assert(GRID <= 256, "one workgroup per CU, all resident");
  auto kfn = ag_gemm<HB, BB, G, WI, WX, PW, CO, CR, 4, 15, 19, 0>;
  static bool init = false;
  if (!init) {
    HIP_CHECK(hipFuncSetAttribute((const void*)kfn, hipFuncAttributeMaxDynamicSharedMemorySize, LDS_BYTES));
    init = true;
  }
  Args a;
  a.x = (const char*)d_in[0]; a.w = (const char*)d_in[1];
  a.sx = (const float*)d_in[2]; a.sw = (const float*)d_in[3];
  for (int p = 0; p < NR; ++p) { a.ws[p] = (char*)d_ws_all[p]; a.out[p] = (float*)d_out_all[p]; }
  a.rank = rank; a.pad = 0;
  make_table<HB, BB, G>((unsigned char*)a.tblO, (unsigned char*)a.tblR, rank, 9.f);
  hipLaunchKernelGGL(kfn, dim3(GRID), dim3(64 * CW), LDS_BYTES, stream, a);
  HIP_CHECK(hipGetLastError());
}
```

```cpp
#include <hip/hip_runtime.h>
#include <rccl/rccl.h>
#include <cstdio>
#include <utility>

#define HIP_CHECK(expr) do { hipError_t _e = (expr); if (_e != hipSuccess) { \
    fprintf(stderr, "HIP error %s at %s:%d\n", hipGetErrorString(_e), __FILE__, __LINE__); \
    return; } } while (0)

typedef unsigned u32x4 __attribute__((ext_vector_type(4)));
typedef int i32x4 __attribute__((ext_vector_type(4)));
typedef int i32x2 __attribute__((ext_vector_type(2)));
typedef float f32x4 __attribute__((ext_vector_type(4)));
typedef unsigned long long u64;
#define LAS __attribute__((address_space(3)))

constexpr int NR = 8;
constexpr int MS = 512;
constexpr int KD = 4096;
constexpr int NL = 512;
constexpr size_t SHARD = (size_t)MS * KD;
constexpr size_t CNT_OFF = 0;
constexpr size_t TMO_OFF = 16 << 10;
constexpr size_t TK_OFF = 20 << 10;
constexpr size_t XF_OFF = 128 << 10;
constexpr size_t WF_OFF = 144 << 10;
constexpr size_t CF_OFF = 152 << 10;
constexpr size_t PF_OFF = 256 << 10;
constexpr size_t LF_OFF = 192 << 10;
constexpr size_t PK_OFF = 2u << 20;
constexpr size_t PL_OFF = 4u << 20;
constexpr size_t WQ_OFF = 6u << 20;
constexpr size_t XG_OFF = 10u << 20;
constexpr size_t WS_NEED = XG_OFF + 2 * NR * SHARD;
constexpr u64 DEADLINE = 20000000ull;
constexpr int CW = 16;
constexpr int SCR = 4096;
constexpr int ACC_OFF = CW * SCR + 256;
constexpr int LDS_BYTES = ACC_OFF + 16 * 4096;
constexpr unsigned MARK = 0xffffffffu;
constexpr int XMARK = (int)0x80000000;
constexpr int NCL = 21;

struct Args { const char* x; const char* w; char* ws[NR]; int rank, pad; const float* sx; const float* sw; float* out[NR]; };

template <typename F, int... I> __device__ __forceinline__ void sfor_impl(F&& f, std::integer_sequence<int, I...>) { (f(std::integral_constant<int, I>{}), ...); }
template <int N, typename F> __device__ __forceinline__ void sfor(F&& f) { sfor_impl(f, std::make_integer_sequence<int, N>{}); }
__device__ __forceinline__ void waitvm0() { asm volatile("s_waitcnt vmcnt(0)" ::: "memory"); }
template <int OFF> __device__ __forceinline__ void gstore_wt(char* q, const u32x4& v) {
  asm volatile("global_store_dwordx4 %0, %1, off offset:%2 sc0 sc1\n\ts_nop 1" ::"v"(q), "v"(v), "n"(OFF) : "memory"); }
__device__ __forceinline__ void fstore(unsigned* fp, unsigned v) {
  asm volatile("global_store_dword %0, %1, off sc0 sc1" ::"v"(fp), "v"(v) : "memory"); }
__device__ __forceinline__ unsigned fload(const unsigned* fp) { return __hip_atomic_load(fp, __ATOMIC_RELAXED, __HIP_MEMORY_SCOPE_SYSTEM); }
__device__ __forceinline__ unsigned mbcnt(u64 m, unsigned base) { return __builtin_amdgcn_mbcnt_hi((unsigned)(m >> 32), __builtin_amdgcn_mbcnt_lo((unsigned)m, base)); }
__device__ __forceinline__ u64 lomask(int n) { return n >= 64 ? ~0ull : (1ull << n) - 1; }

__device__ __forceinline__ bool expired(unsigned* tmo, unsigned call, u64 t0) {
  unsigned v;
  asm volatile("global_load_dword %0, %1, %2 sc1\n\ts_waitcnt vmcnt(0)" : "=&v"(v) : "v"(0), "s"(tmo) : "memory");
  if (__builtin_amdgcn_readfirstlane(v) != call && __builtin_amdgcn_s_memrealtime() - t0 <= DEADLINE) return false;
  asm volatile("global_store_dword %0, %1, %2 sc1\n\ts_nop 1" :: "v"(0), "v"(call), "s"(tmo) : "memory");
  return true;
}

template <int NB, int G> __device__ __forceinline__ int xorder(unsigned u) {
  constexpr unsigned F = NB % G ? NB % G : G;
  if (u < F * 16) return u % F * 16 + u / F;
  u -= F * 16;
  const unsigned gi = u / (16 * G), r = u % (16 * G);
  return (F + gi * G + r % G) * 16 + r / G;
}
constexpr int R0 = 8;
template <int HB> __host__ __device__ __forceinline__ unsigned pos_of(unsigned d, unsigned blk) {
  const int r = d < 4 ? blk : blk - HB + R0, c = r < R0 ? 0 : r - R0 < 32 - HB ? r - R0 : 32 - HB;
  return 3 * r + 4 * c + d - 1; }
template <int HB> __host__ __device__ __forceinline__ unsigned code_of(unsigned p) {
  static_assert(R0 + 32 - HB <= HB, "");
  if (p >= 128 - HB) return p - (128 - HB);
  const unsigned q = p - 3 * R0;
  if (q < 7 * (32 - HB)) return (q % 7 + 1) << 5 | (q / 7 + (q % 7 < 3 ? R0 : HB));
  if (p >= 3 * R0) p -= 4 * (32 - HB);
  return (p % 3 + 1) << 5 | p / 3; }

template <int OFF> __device__ __forceinline__ void mload(u32x4& d, unsigned vo, const char* sb, u64 em) {
  asm volatile("s_mov_b64 exec, %3\n\tglobal_load_dwordx4 %0, %1, %2 offset:%4 sc0 sc1\n\ts_mov_b64 exec, -1" : "+v"(d) : "v"(vo), "s"(sb), "s"(em), "n"(OFF) : "memory"); }
template <int OFF> __device__ __forceinline__ void mstore(const u32x4& d, unsigned vo, char* sb, u64 em) {
  asm volatile("s_mov_b64 exec, %3\n\tglobal_store_dwordx4 %0, %1, %2 offset:%4 sc0 sc1\n\ts_mov_b64 exec, -1\n\ts_nop 1" :: "v"(vo), "v"(d), "s"(sb), "s"(em), "n"(OFF) : "memory"); }
template <int SP> struct Unit { char* q; unsigned* fp; u64 m[SP]; u64 fm; };

template <int HB, int G, int NG, int U, int SP, int NSL, int RB, int TB, int MODE>
__device__ __forceinline__ void pusher(const char* ax, const char* aw, char* myws, char* pws, float* out, unsigned rank, unsigned dl, unsigned* cnt, int w0, int nw, int lane) {
  constexpr int YB = 32 - HB;
  __builtin_amdgcn_s_waitcnt(0x0f70);
  const unsigned peer = rank ^ dl;
  const bool cross = dl >= 4;
  const unsigned total = cross ? 128 + YB * 16 : HB * 16, totalr = total + (cross && !(MODE & 4) ? (NSL + SP - 1) / SP : 0);
  const unsigned vx = (lane & 15) * KD + (lane >> 4) * 16;
  const unsigned vw = (lane >> 2) * NL + (lane & 3) * 16;
  const unsigned vd = lane * 16, vd2 = vd + 4096;
  u32x4 buf[2][SP];
  auto loads = [&](u32x4* d, const char* sb, unsigned vo, int st, const u64* m) {
    asm volatile("s_mov_b64 exec, %9\n\tglobal_load_dwordx4 %0, %4, %8 sc0 sc1\n\ts_mov_b64 exec, %10\n\tglobal_load_dwordx4 %1, %5, %8 sc0 sc1\n\t"
                 "s_mov_b64 exec, %11\n\tglobal_load_dwordx4 %2, %6, %8 sc0 sc1\n\ts_mov_b64 exec, %12\n\tglobal_load_dwordx4 %3, %7, %8 sc0 sc1\n\ts_mov_b64 exec, -1"
                 : "+v"(d[0]), "+v"(d[1]), "+v"(d[2]), "+v"(d[3]) : "v"(vo), "v"(vo + st), "v"(vo + 2 * st), "v"(vo + 3 * st), "s"(sb), "s"(m[0]), "s"(m[1]), "s"(m[2]), "s"(m[3]) : "memory");
    sfor<SP - 4>([&](auto ii) { constexpr int i = ii; mload<i * 1024>(d[4 + i], vd2, sb, m[4 + i]); }); };
  auto id_of = [&](unsigned u) { return !cross ? xorder<HB, G>(u) : u < 128 ? 512 + u : HB * 16 + xorder<YB, G>(u - 128); };
  auto src_of = [&](int id) { return id >= 512 ? aw + (size_t)((id - 512) >> 1) * (64 * NL) + (peer & 3) * 128 + (id & 1) * 64 : ax + (size_t)(id >> 4) * (16 * KD) + (id & 15) * 256; };
  sfor<2 * SP>([&](auto ii) { constexpr int i = ii; buf[i / SP][i % SP] = u32x4{0, 0, 0, 0}; });
  unsigned cv;
  asm volatile("global_load_dword %0, %1, off sc1" : "=v"(cv) : "v"(cnt) : "memory");
  int id = id_of(w0);
  { const u64 m[8] = {~0ull, ~0ull, ~0ull, ~0ull, 0, 0, 0, 0}; loads(buf[0], src_of(id), id >= 512 ? vw : vx, id >= 512 ? 16 * NL : 64, m); }
  u64 todo = TB && !cross ? lomask(NCL) : 0ull;
  asm volatile("s_waitcnt vmcnt(0)" : "+v"(cv) :: "memory");
  const unsigned call = __builtin_amdgcn_readfirstlane(cv) + 1;
  char* dx = pws + XG_OFF + (size_t)(call & 1) * (NR * SHARD) + (size_t)rank * SHARD;
  char* dw = pws + WQ_OFF + (size_t)(call & 1) * SHARD + (size_t)(rank & 3) * (SHARD / 4) - 512 * 4096;
  unsigned* fx = (unsigned*)(pws + XF_OFF);
  unsigned* fw = (unsigned*)(pws + WF_OFF) + (rank & 3) * 128 - 512;
  unsigned* tkp = (unsigned*)(myws + TK_OFF) + dl * 32;
  const unsigned* lf = cross ? (const unsigned*)(myws + LF_OFF) + (dl & 3) * 512 : (const unsigned*)(myws + CF_OFF) + (peer & 3) * 32;
  const unsigned gate = total - (cross ? 0 : TB * 16);
  unsigned* tmo = (unsigned*)(myws + TMO_OFF);
  const unsigned tbase = (call - 1) * (totalr - nw) - (cross ? 3 : 2) * nw;
  auto xw_unit = [&](int i) { Unit<SP> r; r.q = (i >= 512 ? dw : dx) + (size_t)i * 4096;
    const unsigned p = pos_of<HB>(dl, (i >> 4) & 31);
    r.fp = i >= 512 ? fw + i : fx + (p % NG * 16 + (i & 15)) * 16 + p / NG;
    sfor<SP>([&](auto ii) { constexpr int j = ii; r.m[j] = j < 4 ? ~0ull : 0; }); r.fm = 1; return r; };
  auto stores = [&](const u32x4* d, const Unit<SP>& u) {
    asm volatile("s_mov_b64 exec, %6\n\tglobal_store_dwordx4 %0, %1, %5 sc0 sc1\n\t"
                 "s_mov_b64 exec, %7\n\tglobal_store_dwordx4 %0, %2, %5 offset:1024 sc0 sc1\n\t"
                 "s_mov_b64 exec, %8\n\tglobal_store_dwordx4 %0, %3, %5 offset:2048 sc0 sc1\n\t"
                 "s_mov_b64 exec, %9\n\tglobal_store_dwordx4 %0, %4, %5 offset:3072 sc0 sc1\n\ts_mov_b64 exec, -1\n\ts_nop 1"
                 :: "v"(vd), "v"(d[0]), "v"(d[1]), "v"(d[2]), "v"(d[3]), "s"(u.q), "s"(u.m[0]), "s"(u.m[1]), "s"(u.m[2]), "s"(u.m[3]) : "memory");
    sfor<SP - 4>([&](auto ii) { constexpr int i = ii; mstore<i * 1024>(d[4 + i], vd2, u.q, u.m[4 + i]); }); };
  Unit<SP> cur = xw_unit(id);
  stores(buf[0], cur);
  if (RB && cross) {
    for (int j = w0; j < 24 * RB; j += nw) {
      float* o = out + (size_t)((peer ^ (j / (RB * 8) + 1)) * MS + (HB - RB) * 16 + j % (RB * 8) / 2 * 4 + (lane >> 4)) * NL + (peer & 3) * 128 + (j & 1) * 64 + (lane & 15) * 4;
      gstore_wt<0>((char*)o, u32x4{MARK, MARK, MARK, MARK}); }
  }
  unsigned fval = call, vf = 0, one = 1, tv = 0, pv = 0;
  unsigned tk = w0 + nw, tk2 = w0 + 2 * nw;
  u64 rdy = 0;
  const u64 t0 = __builtin_amdgcn_s_memrealtime();
  for (unsigned it = 1;; ++it) {
    bool done = false;
    sfor<2>([&](auto ic) { constexpr int c = ic;
      if (!done) {
        Unit<SP> nxt; const char* sb; unsigned vo, nval, nvf; int st;
        if (tk < (todo ? gate : total)) { nvf = 0; const int i = id_of(tk); nxt = xw_unit(i); sb = src_of(i); vo = i >= 512 ? vw : vx; st = i >= 512 ? 16 * NL : 64; nval = call; }
        else {
          const unsigned s0 = (tk - total) * SP, g = (s0 + lane) * 4 + (rank & 3);
          vo = vd; st = 1024; nvf = (g % U * 16 + g / U) * 4; nval = pv;
          sb = myws + PL_OFF + ((dl & 3) * 512 + (s0 & 511)) * 1024;
          nxt.q = pws + PK_OFF + ((rank & 3) * 512 + (s0 & 511)) * 1024;
          nxt.fp = (unsigned*)(pws + PF_OFF);
          const unsigned n = (rdy >> lane) & 1 ? (pv & 31) * 4 : 0;
          sfor<SP>([&](auto ii) { constexpr int i = ii; nxt.m[i] = lomask(__builtin_amdgcn_readlane(n, i)); });
          nxt.fm = rdy;
        }
        loads(buf[c ^ 1], sb, vo, st, nxt.m);
        const bool take = nxt.fm != 0;
        if (cross && take) tk = tk2;
        const unsigned sn = (tk - total) * SP;
        const u64 pm = !cross ? (tk + 2 * nw >= gate ? todo : 0ull) : tk - total < totalr - total ? lomask(min(NSL - sn, (unsigned)SP)) : 0ull;
        asm volatile("s_cmp_lg_u64 %4, 0\n\ts_cselect_b64 exec, 1, 0\n\tglobal_atomic_add %0, %2, %3, off sc0\n\t"
                     "s_mov_b64 exec, %6\n\tglobal_load_dword %1, %11, %5 sc0 sc1\n\ts_mov_b64 exec, -1\n\t"
                     "s_waitcnt vmcnt(0)\n\t"
                     "s_mov_b64 exec, %10\n\tglobal_store_dword %7, %8, %9 sc0 sc1\n\ts_mov_b64 exec, -1\n\ts_nop 1"
                     : "+v"(tv), "+v"(pv) : "v"(tkp), "v"(one), "s"(nxt.fm), "s"(lf + (cross ? sn & 511 : 0)), "s"(pm), "v"(vf), "v"(fval), "s"(cur.fp), "s"(cur.fm), "v"(lane * 4) : "memory", "scc");
        stores(buf[c ^ 1], nxt);
        if (RB && c == 0 && it == 1 && cross && lane == 0) fstore((unsigned*)(myws + CF_OFF) + 256 + (dl & 3) * 32 + w0, call);
        if (take) (cross ? tk2 : tk) = __builtin_amdgcn_readfirstlane(tv) - tbase;
        const u64 bm = __ballot((pv >> 16) == (call & 0xffff)) & pm;
        rdy = cross && bm == pm ? pm : 0;
        if (TB && !cross && (bm == todo || tk >= total)) todo = 0;
        done = !take && tk >= totalr;
        if (!take && !rdy && !done) {
          __builtin_amdgcn_s_sleep(2);
          if ((it & 63) == 0 && expired(tmo, call, t0)) done = true;
        }
        cur = nxt; fval = nval; vf = nvf;
      } });
    if (done) break;
  }
  if (lane == 0) __hip_atomic_store(cnt, call, __ATOMIC_RELAXED, __HIP_MEMORY_SCOPE_AGENT);
}

template <int HB, int G, int WI, int WX, int PW, int NG, int NU, int SP, int DH, int DY, int RB, int TB, int GB, int MODE>
__global__ void __launch_bounds__(64 * CW, 4) ag_gemm(Args a) {
  extern __shared__ __attribute__((aligned(16))) char lds[];
  constexpr int NPW = 3 * WI + 4 * WX, NP = (NPW + PW - 1) / PW, NPOS = 160 - HB, U = NU > 0 ? NU * CW : -NU * NP / 16 * 16;
  constexpr int NSL = (3 * DH + HB + 3 * DY) * 4, NDH = HB - RB - DH, NFL = NSL + (NDH + 32 - HB - DY) * 12;
  static_assert(NFL + SP <= 512 && SP >= 4 && SP <= 8 && 4 * NFL + 48 * RB <= 16 * U, "");
  static_assert(DH <= HB - RB && NG * 16 >= NPOS && NG <= 16 && NG % 2 == 0 && 3 * TB <= NG && HB - TB - GB >= R0 + 32 - HB, "");
  static_assert(U >= 128 && U <= 1024 && RB * 64 <= U && WX >= 8 && WX <= 32 && WX % 2 == 0, "flag layout");
  const int lane = threadIdx.x & 63, wid = __builtin_amdgcn_readfirstlane(threadIdx.x >> 6);
  const int b = blockIdx.x;
  const int cb = b - NP, grp = (cb >> 5) * 2 + ((cb >> 2) & 1), ns = (cb & 3) * 4 + ((cb >> 3) & 3);
  const bool ownw = (ns & 1) == 0 || cb >= NG * 16;
  constexpr int WM = WI < WX ? WI : WX;
  const int gw = b * PW + wid;
  int d, w0 = 0;
  if (b >= NP) d = ownw ? 0 : 4 + (ns >> 2);
  else if (wid >= PW) d = 1 + b % 3;
  else if (WI == WX || gw < 7 * WM) { d = 1 + gw % 7; w0 = gw / 7; }
  else if (WI > WX) { d = 1 + (gw - 7 * WM) % 3; w0 = WM + (gw - 7 * WM) / 3; }
  else { d = 4 + (gw - 7 * WM) % 4; w0 = WM + (gw - 7 * WM) / 4; }
  const char* ax = a.x; const char* aw = a.w; char* myws = a.ws[0]; char* dws = a.ws[d & 7]; float* dout = a.out[b < NP ? 0 : d & 7];
  const unsigned rank = a.rank & 7;
  asm volatile("" :: "s"(ax), "s"(aw), "s"(myws), "s"(dws), "s"(dout), "s"(rank));
  unsigned* cnt = (unsigned*)(myws + CNT_OFF) + b * CW + wid;

  if (b < NP && (NU > 0 || wid < CW + NU)) {
    if (wid < PW) {
      if (gw >= NPW) return;
      pusher<HB, G, NG, U, SP, NSL, RB, TB, MODE>(ax, aw, myws, dws, dout, rank, d, cnt, w0, d >= 4 ? WX : WI, lane);
    } else if (TB && wid == PW && b < 3 * NCL) {
      static_assert(!TB || NP >= 3 * NCL, "");
      const unsigned peer = rank ^ d, j0 = b / 3;
      const unsigned call = __builtin_amdgcn_readfirstlane(__hip_atomic_load(cnt, __ATOMIC_RELAXED, __HIP_MEMORY_SCOPE_AGENT)) + 1;
      char* q = myws + XG_OFF + (size_t)(call & 1) * (NR * SHARD) + (size_t)peer * SHARD + (size_t)(HB - TB) * (16 * KD) + lane * 16;
      for (int j = j0; j < TB * 64; j += NCL) gstore_wt<0>(q + j * 1024, u32x4{(unsigned)XMARK, (unsigned)XMARK, (unsigned)XMARK, (unsigned)XMARK});
      waitvm0();
      if (lane < 2) fstore(lane ? (unsigned*)(myws + CF_OFF) + 128 + b % 3 * 32 + j0 : (unsigned*)(dws + CF_OFF) + (rank & 3) * 32 + j0, lane ? call : call << 16);
      if (lane == 0) __hip_atomic_store(cnt, call, __ATOMIC_RELAXED, __HIP_MEMORY_SCOPE_AGENT);
    }
    return;
  }

  if (MODE & 8) return;
  unsigned call = __hip_atomic_load(cnt, __ATOMIC_RELAXED, __HIP_MEMORY_SCOPE_AGENT);
  typedef const __attribute__((address_space(4))) float* cfp;
  const float sc = *(cfp)a.sx * *(cfp)a.sw, scale = fabsf(sc);
  const bool neg = sc < 0.f;
  unsigned* tmo = (unsigned*)(myws + TMO_OFF);
  LAS char* scr = (LAS char*)lds + wid * SCR;

  if (NU > 0 ? cb >= NG * 16 : cb < 0) {
    call = __builtin_amdgcn_readfirstlane(call) + 1;
    if (lane == 0) __hip_atomic_store(cnt, call, __ATOMIC_RELAXED, __HIP_MEMORY_SCOPE_AGENT);
    if (MODE & 4) return;
    const int uw = NU > 0 ? wid * NU + cb - NG * 16 : (wid - (CW + NU)) * NP + b;
    if (uw >= U) return;
    const int gi = (lane & 15) * U + uw;
    const unsigned tl = (unsigned)__ballot(gi >= 4 * NFL && gi < 4 * NFL + 48 * RB) & 0xffff;
    unsigned todo = (unsigned)__ballot(gi < 4 * NFL) & 0xffff, m = 0, v = 0;
    const unsigned* pf = (const unsigned*)(myws + PF_OFF) + uw * 16 + (lane & 15);
    float* out = dout + (lane & 15) * NL + 4 * (lane >> 4);
    const u64 t0 = __builtin_amdgcn_s_memrealtime();
    bool bad = false;
    for (unsigned it = 1; todo | m; ++it) {
      if (!m) {
        v = fload(pf);
        m = (unsigned)__ballot((v >> 16) == (call & 0xffff)) & todo;
        asm volatile("" ::: "memory");
        if (!m) {
          if ((it & 63) == 0 && expired(tmo, call, t0)) { bad = true; m = todo; }
          else { __builtin_amdgcn_s_sleep(MODE & 16 ? 8 : 2); continue; }
        }
        todo &= ~m;
        if (!bad) m &= (unsigned)__ballot(v & 31);
        if (!m) continue;
      }
      const int k = __builtin_ctz(m), g = __builtin_amdgcn_readlane(gi, k);
      const int ti = (__builtin_amdgcn_readlane(v, k) >> 5) & 2047, blk = ti >> 6, src = (blk < HB ? (rank ^ 4) & 4 : rank & 4) | ((ti >> 4) & 3);
      m &= m - 1;
      float* o = out + (size_t)(src * MS + blk * 16) * NL + (ti & 15) * 32;
      u32x4 d;
      asm volatile("global_load_dwordx4 %0, %1, %2 sc0 sc1\n\ts_waitcnt vmcnt(0)" : "=&v"(d) : "v"(lane * 16), "s"(myws + PK_OFF + ((g & 3) * 512 + (g >> 2)) * 1024) : "memory");
      *(LAS u32x4*)(scr + lane * 16) = d;
      asm volatile("" ::: "memory");
      const unsigned kb = *(LAS unsigned char*)(scr + lane);
      unsigned pos = 0, w[8]; int n = 0; u64 mk[8];
#pragma unroll
      for (int j = 0; j < 8; ++j) { mk[j] = __ballot((kb >> j) & 1); n += __builtin_popcountll(mk[j]); pos = mbcnt(mk[j], pos); w[j] = 0; }
      const int nb = 64 + ((n + 1) & ~1) * 2;
      const LAS unsigned* np = (const LAS unsigned*)(scr + nb) + (pos >> 3);
      const unsigned n0 = np[0], n1 = np[1];
      LAS char* lp = scr + 64 + 2 * pos;
#define RD(j, k) "s_mov_b64 exec, %" #k "\n\tds_read_u16 %" #j ", %8\n\tv_add_u32 %8, 2, %8\n\t"
      asm volatile(RD(0, 9) RD(1, 10) RD(2, 11) RD(3, 12) RD(4, 13) RD(5, 14) RD(6, 15) RD(7, 16) "s_mov_b64 exec, -1"
                   : "+v"(w[0]), "+v"(w[1]), "+v"(w[2]), "+v"(w[3]), "+v"(w[4]), "+v"(w[5]), "+v"(w[6]), "+v"(w[7]), "+v"(lp)
                   : "s"(mk[0]), "s"(mk[1]), "s"(mk[2]), "s"(mk[3]), "s"(mk[4]), "s"(mk[5]), "s"(mk[6]), "s"(mk[7]) : "memory");
      unsigned nib = __builtin_amdgcn_alignbit(n1, n0, (pos & 7) * 4);
#define HI(j, k) "s_mov_b64 exec, %" #k "\n\tv_and_b32_sdwa %" #j ", %8, %17 dst_sel:WORD_1 dst_unused:UNUSED_PRESERVE src0_sel:DWORD src1_sel:DWORD\n\tv_lshrrev_b32 %8, 4, %8\n\t"
      asm volatile("s_waitcnt lgkmcnt(0)\n\t" HI(7, 16) HI(6, 15) HI(5, 14) HI(4, 13) HI(3, 12) HI(2, 11) HI(1, 10) HI(0, 9) "s_mov_b64 exec, -1"
                   : "+v"(w[0]), "+v"(w[1]), "+v"(w[2]), "+v"(w[3]), "+v"(w[4]), "+v"(w[5]), "+v"(w[6]), "+v"(w[7]), "+v"(nib)
                   : "s"(mk[0]), "s"(mk[1]), "s"(mk[2]), "s"(mk[3]), "s"(mk[4]), "s"(mk[5]), "s"(mk[6]), "s"(mk[7]), "v"(15) : "memory");
      int eb = nb + ((n + 7) >> 3) * 4;
#pragma unroll
      for (int j = 0; j < 8; ++j) mk[j] = __ballot(w[j] >= 15u << 16);
#pragma unroll
      for (int j = 0; j < 8; ++j) if (__builtin_expect(mk[j] != 0, 0)) {
        const unsigned x = *(LAS unsigned short*)(scr + eb + 2 * mbcnt(mk[j], 0));
        if (w[j] >= 15u << 16) w[j] = x << 16 | (w[j] & 0xffff);
        eb += 2 * __builtin_popcountll(mk[j]); }
      f32x4 y[2];
#pragma unroll
      for (int j = 0; j < 8; ++j) y[j >> 2][j & 3] = fmaxf((float)w[j] * scale, 0.f);
      asm volatile("" ::: "memory");
      if (bad) { const float qn = __builtin_nanf(""); y[0] = f32x4{qn, qn, qn, qn}; y[1] = y[0]; }
      gstore_wt<0>((char*)o, __builtin_bit_cast(u32x4, y[0])); gstore_wt<64>((char*)o, __builtin_bit_cast(u32x4, y[1]));
    }
    if (tl) {
      const int t = __builtin_amdgcn_readlane(gi, __builtin_ctz(tl)) - 4 * NFL, sn = (t >> 2) & 3;
      const int ti = ((HB - RB + (t >> 4) / 3) * 4 + (sn ^ ((t >> 4) % 3 + 1))) * 16 + (t & 15);
      const float* o = out + (size_t)((((rank ^ 4) & 4) | ((ti >> 4) & 3)) * MS + (ti >> 6) * 16) * NL + (ti & 15) * 32;
      const unsigned* mf = (const unsigned*)(myws + CF_OFF) + 256 + ((sn ^ rank) & 3) * 32 + lane % WX;
      for (unsigned it = 1; __ballot(fload(mf) < call); ++it) {
        if ((it & 63) == 0 && expired(tmo, call, t0)) { gstore_wt<0>((char*)o, u32x4{MARK, MARK, MARK, MARK}); return; }
        __builtin_amdgcn_s_sleep(4); }
      for (unsigned it = 1;; ++it) {
        u32x4 p, q;
        asm volatile("global_load_dwordx4 %0, %2, off sc0 sc1\n\tglobal_load_dwordx4 %1, %2, off offset:64 sc0 sc1\n\ts_waitcnt vmcnt(0)"
                     : "=&v"(p), "=&v"(q) : "v"(o) : "memory");
        const unsigned mx = max(max(max(p[0], p[1]), max(p[2], p[3])), max(max(q[0], q[1]), max(q[2], q[3])));
        if (!__ballot(mx == MARK) || ((it & 63) == 0 && expired(tmo, call, t0))) break;
        __builtin_amdgcn_s_sleep(1);
      }
    }
    return;
  }

  const unsigned dest = rank ^ d;
  const int col0 = ownw ? (ns >> 1) * 64 : (rank & 3) * 128 + (ns & 2) * 32;
  LAS int* arr = (LAS int*)((LAS char*)lds + CW * SCR);
  if (threadIdx.x < 32) arr[threadIdx.x] = 0;
#pragma unroll
  for (int i = 0; i < 4; ++i) *(LAS u32x4*)((LAS char*)lds + ACC_OFF + i * 16384 + threadIdx.x * 16) = u32x4{0, 0, 0, 0};
  bool bad = false;
  i32x4 wf[4][4];
  if (ownw) {
    const __amdgpu_buffer_rsrc_t rw = __builtin_amdgcn_make_buffer_rsrc((void*)(aw + (size_t)wid * (256 * NL) + col0), 0, 256 * NL, 0x00020000);
#pragma unroll
    for (int i = 0; i < 16; ++i) wf[i >> 2][i & 3] = __builtin_bit_cast(i32x4, __builtin_amdgcn_raw_buffer_load_b128(rw, (lane >> 2) * NL + (lane & 3) * 16, i * 16 * NL, 0));
  }
  call = __builtin_amdgcn_readfirstlane(call) + 1;
  if (lane == 0) __hip_atomic_store(cnt, call, __ATOMIC_RELAXED, __HIP_MEMORY_SCOPE_AGENT);
  asm volatile("s_waitcnt lgkmcnt(0)" ::: "memory");
  __builtin_amdgcn_s_barrier();
  asm volatile("" ::: "memory");
  if (!ownw) {
    const unsigned* fw = (const unsigned*)(myws + WF_OFF) + (dest & 3) * 128 + wid * 8 + (lane & 3) * 2 + ((ns >> 1) & 1);
    const u64 t0 = __builtin_amdgcn_s_memrealtime();
    for (unsigned it = 1; __ballot(fload(fw) < call); ++it) {
      if ((it & 63) == 0 && expired(tmo, call, t0)) { bad = true; break; }
      __builtin_amdgcn_s_sleep(4); }
    asm volatile("" ::: "memory");
    const __amdgpu_buffer_rsrc_t rw = __builtin_amdgcn_make_buffer_rsrc((void*)(myws + WQ_OFF + (size_t)(call & 1) * SHARD + (size_t)(dest & 3) * (SHARD / 4) + wid * 32768 + (ns & 2) * 2048), 0, 32768, 0x00020000);
#pragma unroll
    for (int i = 0; i < 16; ++i) wf[i >> 2][i & 3] = __builtin_bit_cast(i32x4, __builtin_amdgcn_raw_buffer_load_b128(rw, lane * 16 + (i & 3) * 1024, (i >> 2) * 8192, 17));
  }
  {
    LAS char* wr = scr + ((lane >> 1) & 1) * 2048 + (lane >> 2) * 32;
    LAS char* rb = scr + (16 * (lane >> 4) + ((lane & 15) >> 1)) * 32 + 8 * (lane & 1);
#pragma unroll
    for (int i = 0; i < 4; ++i) {
#pragma unroll
      for (int j = 0; j < 4; ++j) *(LAS i32x4*)(wr + j * 512 + (((lane ^ j) & 1) * 16)) = wf[i][j];
      asm volatile("" ::: "memory");
#pragma unroll
      for (int t = 0; t < 4; ++t) {
        LAS char* p = rb + (t >> 1) * 2048 + 16 * (((lane >> 4) ^ t) & 1);
        const i32x2 u0 = __builtin_amdgcn_ds_read_tr8_b64_v2i32((LAS i32x2*)p), u1 = __builtin_amdgcn_ds_read_tr8_b64_v2i32((LAS i32x2*)(p + 256));
        wf[i][t] = i32x4{u0[0], u0[1], u1[0], u1[1]}; }
      asm volatile("" ::: "memory");
    }
  }
  const unsigned pz = (lane & 15) * NG + grp, code = code_of<HB>(pz) ^ rank << 5;
  unsigned todo = (unsigned)__ballot(pz < (ownw ? NPOS : 128) && (ownw || (code >> 5) != dest)) & 0xffff;
  const unsigned mine = (unsigned)__ballot(pz >= 128 - HB) & 0xffff;
  const unsigned spec = TB ? (unsigned)__ballot(pz < 128 - HB && pz >= 128 - HB - 3 * TB) & 0xffff : 0;
  unsigned pofs = XF_OFF + ((grp * 16 + wid) * 16 + (lane & 15)) * 4;
  if (TB && spec) { const unsigned pg = __builtin_ctz(spec) * NG + grp - 3 * GB;
    if (lane == 16) pofs = XF_OFF + ((pg % NG * 16 + wid) * 16 + pg / NG) * 4;
    if (lane >= 34 && lane < 34 + NCL) pofs = CF_OFF + (128 + (pg - 4 * (32 - HB)) % 3 * 32 + lane - 34) * 4; }
  unsigned ready = bad || (MODE & 1) ? 0xffff : mine, near = 0;
  auto poll = [&]() { return fload((const unsigned*)(myws + pofs)); };
  auto seen = [&](unsigned fv) {
    const u64 fb = __ballot(fv >= call);
    ready |= (unsigned)fb & 0xffff;
    if (TB) near = (~fb & (lomask(NCL) << 34 | 1 << 16)) == 0 ? spec : 0;
    asm volatile("" ::: "memory"); };
  const char* xg = myws + XG_OFF + (size_t)(call & 1) * (NR * SHARD) + wid * 4096;
  const unsigned vg = lane * 16, vx = (lane & 15) * KD + (lane >> 4) * 16;
  auto loadu = [&](i32x4* dv, int s) {
    const unsigned cd = __builtin_amdgcn_readlane(code, s); const bool me = (mine >> s) & 1;
    const __amdgpu_buffer_rsrc_t rs = __builtin_amdgcn_make_buffer_rsrc((void*)(me ? ax + (cd & 31) * (16 * KD) + wid * 256 : xg + (size_t)cd * (16 * KD)), 0, 16 * KD, 0x00020000);
    const unsigned vo = me ? vx : vg; const int st = me ? 64 : 1024;
#pragma unroll
    for (int i = 0; i < 4; ++i) dv[i] = __builtin_bit_cast(i32x4, __builtin_amdgcn_raw_buffer_load_b128(rs, vo, i * st, 17)); };
  const int ntile = __builtin_popcount(todo);
  int e = -1;
  auto comp = [&](const i32x4* xv, int s, unsigned fv) {
    LAS int* sa = (LAS int*)((LAS char*)lds + ACC_OFF + s * 4096) + lane;
#pragma unroll
    for (int t = 0; t < 4; ++t) {
      i32x4 acc = {0, 0, 0, 0};
#pragma unroll
      for (int i = 0; i < 4; ++i) acc = __builtin_amdgcn_mfma_i32_16x16x64_i8(wf[i][t], xv[i], acc, 0, 0, 0);
#pragma unroll
      for (int j = 0; j < 4; ++j) __hip_atomic_fetch_add(sa + (t * 4 + j) * 64, acc[j], __ATOMIC_RELAXED, __HIP_MEMORY_SCOPE_WORKGROUP);
    }
    int t;
    asm volatile("s_mov_b64 exec, 1\n\tds_add_rtn_u32 %0, %1, %2\n\ts_mov_b64 exec, -1" : "=&v"(t) : "v"(arr + s), "v"(bad ? 0x101 : 1) : "memory");
    seen(fv);
    asm volatile("s_waitcnt lgkmcnt(0)" : "+v"(t) :: "memory");
    if ((__builtin_amdgcn_readfirstlane(t) & 0xff) == CW - 1) {
      const bool self = !(ready & todo);
      const u64 v = (u64)1 << 32 | (self ? 2u : 3u) << 2 * s;
      asm volatile("s_mov_b64 exec, 1\n\tds_add_u64 %0, %1\n\ts_mov_b64 exec, -1" :: "v"(arr + 16), "v"(v) : "memory");
      if (self) e = 2 * s;
    } };
  const u64 t0 = __builtin_amdgcn_s_memrealtime();
  for (unsigned it = 1;; ++it) {
    unsigned m = ready & todo;
    if (m) {
      if (MODE & 4096) __builtin_amdgcn_s_setprio(3);
      i32x4 xa[4], xb[4];
      int sa = __builtin_ctz(m), sb;
      loadu(xa, sa);
      for (;;) {
        todo &= ~(1u << sa); m = ready & todo;
        if (!m) { comp(xa, sa, poll()); break; }
        sb = __builtin_ctz(m); loadu(xb, sb);
        comp(xa, sa, poll());
        todo &= ~(1u << sb); m = ready & todo;
        if (!m) { comp(xb, sb, poll()); break; }
        sa = __builtin_ctz(m); loadu(xa, sa);
        comp(xb, sb, poll());
      }
    } else if (TB && (near & todo)) {
      i32x4 xa[4];
      const int sa = __builtin_ctz(near);
      loadu(xa, sa);
      const unsigned fv = poll();
      int mn = xa[0][0];
#pragma unroll
      for (int j = 1; j < 16; ++j) mn = min(mn, xa[j >> 2][j & 3]);
      if (!__ballot(mn == XMARK)) { todo &= ~near; comp(xa, sa, fv); }
      else seen(fv);
    } else if (todo) seen(poll());
    if (todo && (it & 63) == 0 && expired(tmo, call, t0)) { bad = true; ready = 0xffff; }
    int lane = threadIdx.x & 63;
    asm volatile("" : "+v"(lane));
    if (e < 0) {
      if (ready & todo) continue;
      const i32x2 fq = *(volatile LAS i32x2*)(arr + 16);
      const unsigned fm = __builtin_amdgcn_readfirstlane(fq[0]);
      if (fm) {
        const unsigned rot = fm >> (wid & 1) & 0x55555555u;
        e = rot ? __builtin_ctz(rot) + (wid & 1) : __builtin_ctz(fm);
        int t;
        asm volatile("s_mov_b64 exec, 1\n\tds_and_rtn_b32 %0, %1, %2\n\ts_mov_b64 exec, -1\n\ts_waitcnt lgkmcnt(0)" : "=&v"(t) : "v"(arr + 16), "v"(~(1u << e)) : "memory");
        if (!((__builtin_amdgcn_readfirstlane(t) >> e) & 1)) { e = -1; continue; }
      } else {
        if (!todo && __builtin_amdgcn_readfirstlane(fq[1]) == ntile) break;
        if (MODE & 4096) __builtin_amdgcn_s_setprio(0);
        if (!(near & todo)) __builtin_amdgcn_s_sleep(1);
        continue;
      }
    }
    {
      const int s = e >> 1, h = e & 1;
      e = -1;
      if (MODE & 4096) __builtin_amdgcn_s_setprio(2);
      const unsigned cd = __builtin_amdgcn_readlane(code, s), src = cd >> 5, blk = cd & 31;
      const bool home = blk < HB && src != rank;
      const bool direct = home ? blk >= DH : blk >= HB + DY, tail = home && blk >= HB - RB;
      const int sl = (ns & 2) + h;
      unsigned slot = NSL + ((home ? blk - DH : blk - (HB + DY) + NDH) * 3 + ((src ^ (home ? rank : dest)) & 3) - 1) * 4 + sl;
      __builtin_amdgcn_s_waitcnt(0x0f70);
      if (!ownw && !direct) asm volatile("s_mov_b64 exec, 1\n\tglobal_atomic_add %0, %1, %3, %2 sc0 sc1\n\ts_mov_b64 exec, -1"
                                         : "+v"(slot) : "v"(0), "s"(myws + TK_OFF + (4 + d) * 128), "v"(1) : "memory");
      const LAS int* sa = (LAS int*)((LAS char*)lds + ACC_OFF + s * 4096 + h * 2048) + lane;
      int acc[8]; f32x4 y[2];
#pragma unroll
      for (int j = 0; j < 8; ++j) acc[j] = sa[j * 64];
      const bool bad = (__builtin_amdgcn_readfirstlane(arr[s]) >> 8) != 0;
      if (__builtin_expect(neg, 0)) {
        asm volatile("");
#pragma unroll
        for (int j = 0; j < 8; ++j) acc[j] = -acc[j]; }
      auto getslot = [&]() { asm volatile("s_waitcnt vmcnt(0)" : "+v"(slot) :: "memory");
        return (__builtin_amdgcn_readfirstlane(slot) - (direct ? 0 : (call - 1) * NSL)) & 511; };
      int n64 = 0;
      if (!ownw && !bad && !tail) {
        int n = 0; unsigned kb = 0, pos = 0, nib = 0, t; u64 mk[8]; const int* v = acc;
#pragma unroll
        for (int j = 0; j < 8; ++j) { mk[j] = __ballot(v[j] > 0); n += __builtin_popcountll(mk[j]); pos = mbcnt(mk[j], pos); }
        const int nb = 64 + ((n + 1) & ~1) * 2;
        int eb = nb + ((n + 7) >> 3) * 4;
        *(LAS unsigned*)(scr + nb + lane * 4) = 0;
        LAS char* lp = scr + 64 + 2 * pos;
#define WR(j, k, b) "s_mov_b64 exec, %" #k "\n\tds_write_b16 %0, %" #j "\n\tv_min_u32_sdwa %3, %" #j ", %20 dst_sel:DWORD dst_unused:UNUSED_PAD src0_sel:WORD_1 src1_sel:DWORD\n\t" \
                    "v_add_u32 %0, 2, %0\n\tv_or_b32 %2, " #b ", %2\n\tv_lshl_or_b32 %1, %1, 4, %3\n\t"
        asm volatile(WR(4, 12, 1) WR(5, 13, 2) WR(6, 14, 4) WR(7, 15, 8) WR(8, 16, 16) WR(9, 17, 32) WR(10, 18, 64) WR(11, 19, 0x80) "s_mov_b64 exec, -1"
                     : "+v"(lp), "+v"(nib), "+v"(kb), "=&v"(t)
                     : "v"(v[0]), "v"(v[1]), "v"(v[2]), "v"(v[3]), "v"(v[4]), "v"(v[5]), "v"(v[6]), "v"(v[7]),
                       "s"(mk[0]), "s"(mk[1]), "s"(mk[2]), "s"(mk[3]), "s"(mk[4]), "s"(mk[5]), "s"(mk[6]), "s"(mk[7]), "v"(15) : "memory");
        *(LAS char*)(scr + lane) = (char)kb;
#pragma unroll
        for (int j = 0; j < 8; ++j) mk[j] = __ballot(v[j] >= 15 << 16);
#pragma unroll
        for (int j = 0; j < 8; ++j) if (__builtin_expect(mk[j] != 0, 0)) {
          if (v[j] >= 15 << 16) *(LAS short*)(scr + eb + 2 * mbcnt(mk[j], 0)) = (short)(v[j] >> 16);
          eb += 2 * __builtin_popcountll(mk[j]); }
        LAS unsigned* np = (LAS unsigned*)(scr + nb) + (pos >> 3);
        const unsigned sh0 = (pos & 7) * 4;
        const u64 nv = (u64)nib << sh0;
        __hip_atomic_fetch_or(np, (unsigned)nv, __ATOMIC_RELAXED, __HIP_MEMORY_SCOPE_WAVEFRONT);
        __hip_atomic_fetch_or(np + 1, (unsigned)(nv >> 32), __ATOMIC_RELAXED, __HIP_MEMORY_SCOPE_WAVEFRONT);
        asm volatile("" ::: "memory");
        if (eb <= 1024) {
          n64 = (eb + 63) >> 6;
          const u32x4 d0 = *(LAS u32x4*)(scr + lane * 16);
          char* pk = (direct ? dws + PK_OFF + (rank & 3) * (512 * 1024) : myws + PL_OFF + (d & 3) * (512 * 1024)) + getslot() * 1024 + lane * 16;
          if (lane < n64 * 4) gstore_wt<0>(pk, d0);
          asm volatile("" ::: "memory");
        }
      }
      if (n64 == 0) {
#pragma unroll
        for (int j = 0; j < 8; ++j) y[j >> 2][j & 3] = fmaxf((float)acc[j] * scale, 0.f);
        if (bad) { const float qn = __builtin_nanf(""); y[0] = f32x4{qn, qn, qn, qn}; y[1] = y[0]; }
        float* o = dout + (size_t)(src * MS + blk * 16 + (lane & 15)) * NL + col0 + h * 32 + 4 * (lane >> 4);
        gstore_wt<0>((char*)o, __builtin_bit_cast(u32x4, y[0])); gstore_wt<64>((char*)o, __builtin_bit_cast(u32x4, y[1]));
      }
      if (!ownw && !tail) {
        const unsigned q = getslot(), g = q * 4 + (rank & 3);
        unsigned* fp = direct ? (unsigned*)(dws + PF_OFF) + g % U * 16 + g / U : (unsigned*)(myws + LF_OFF) + (d & 3) * 512 + q;
        if (lane == 0) fstore(fp, call << 16 | (((blk * 4 + (src & 3)) * 4 + (rank & 3)) * 4 + sl) << 5 | n64);
      }
    }
  }
}

extern "C" void kernel_launch(int rank, int nranks, ncclComm_t comm, hipStream_t stream,
                              void* const* d_in, const int* in_sizes, int n_in,
                              void* d_out, int out_size, const int* in_dtypes, int out_dtype,
                              void* const* d_out_all, void* const* d_ws_all, size_t ws_bytes) {
  if (nranks != NR || (unsigned)rank >= NR || ws_bytes < WS_NEED) { fprintf(stderr, "unexpected configuration\n"); return; }
  constexpr int HB = 22, G = 1, WI = 18, WX = 18, PW = 2, NG = 12, NU = -13;
  constexpr int GRID = (3 * WI + 4 * WX + PW - 1) / PW + NG * 16 + (NU > 0 ? NU : 0);
  static_assert(GRID <= 256, "one workgroup per CU, all resident");
  auto kfn = ag_gemm<HB, G, WI, WX, PW, NG, NU, 4, 16, 8, 3, 0, 1, 4096>;
  static bool init = false;
  if (!init) {
    HIP_CHECK(hipFuncSetAttribute((const void*)kfn, hipFuncAttributeMaxDynamicSharedMemorySize, LDS_BYTES));
    init = true;
  }
  Args a;
  a.x = (const char*)d_in[0]; a.w = (const char*)d_in[1];
  a.sx = (const float*)d_in[2]; a.sw = (const float*)d_in[3];
  for (int d = 0; d < NR; ++d) { a.ws[d] = (char*)d_ws_all[rank ^ d]; a.out[d] = (float*)d_out_all[rank ^ d]; }
  a.rank = rank; a.pad = 0;
  hipLaunchKernelGGL(kfn, dim3(GRID), dim3(64 * CW), LDS_BYTES, stream, a);
  HIP_CHECK(hipGetLastError());
}
```

```cpp
#include <hip/hip_runtime.h>
#include <rccl/rccl.h>
#include <cstddef>
#include <cstdio>
#include <utility>

#define HIP_CHECK(expr) do { hipError_t _e = (expr); if (_e != hipSuccess) { \
    fprintf(stderr, "HIP error %s at %s:%d\n", hipGetErrorString(_e), __FILE__, __LINE__); \
    return; } } while (0)

typedef unsigned u32x4 __attribute__((ext_vector_type(4)));
typedef int i32x4 __attribute__((ext_vector_type(4)));
typedef int i32x2 __attribute__((ext_vector_type(2)));
typedef float f32x4 __attribute__((ext_vector_type(4)));
typedef unsigned long long u64;
typedef u64 u64x4 __attribute__((ext_vector_type(4)));
#define LAS __attribute__((address_space(3)))
#define GAS __attribute__((address_space(1)))

constexpr int NR = 8;
constexpr int MS = 512;
constexpr int KD = 4096;
constexpr int NL = 512;
constexpr size_t SHARD = (size_t)MS * KD;
constexpr size_t CNT_OFF = 0;
constexpr size_t TMO_OFF = 16 << 10;
constexpr size_t TK_OFF = 20 << 10;
constexpr size_t DUM_OFF = 512 << 10;
constexpr size_t XF_OFF = 128 << 10;
constexpr size_t WF_OFF = 144 << 10;
constexpr size_t CF_OFF = 152 << 10;
constexpr size_t MF_OFF = 256 << 10;
constexpr size_t PK_OFF = 2u << 20;
constexpr size_t PL_OFF = 4u << 20;
constexpr size_t WQ_OFF = 6u << 20;
constexpr size_t XG_OFF = 8u << 20;
constexpr int ALT = 25;
constexpr size_t WS_NEED = XG_OFF + NR * SHARD + ((size_t)1 << ALT);
constexpr u64 DEADLINE = 20000000ull;
constexpr int CW = 16;
constexpr int SCR = 4096;
constexpr int ACC_OFF = CW * SCR + 256;
constexpr int LDS_BYTES = ACC_OFF + 16 * 4096;
constexpr unsigned MARK = 0xffffffffu;

struct Peer { char* ws; float* out; const char* src; char* dst; };
struct Args { const char* x; const char* w; char* ws; float* out; const float* sx; const float* sw; unsigned rank, pad[3]; Peer p[NR]; };
static_assert(offsetof(Args, rank) == 0x30 && offsetof(Args, p) == 0x40 && sizeof(Peer) == 32, "");

template <typename F, int... I> __device__ __forceinline__ void sfor_impl(F&& f, std::integer_sequence<int, I...>) { (f(std::integral_constant<int, I>{}), ...); }
template <int N, typename F> __device__ __forceinline__ void sfor(F&& f) { sfor_impl(f, std::make_integer_sequence<int, N>{}); }
__device__ __forceinline__ void waitvm0() { asm volatile("s_waitcnt vmcnt(0)" ::: "memory"); }
template <int OFF> __device__ __forceinline__ void gstore_wt(char* q, const u32x4& v) {
  asm volatile("global_store_dwordx4 %0, %1, off offset:%2 sc0 sc1\n\ts_nop 1" ::"v"(q), "v"(v), "n"(OFF) : "memory"); }
__device__ __forceinline__ void fstore(unsigned* fp, unsigned v) {
  asm volatile("global_store_dword %0, %1, off sc0 sc1" ::"v"(fp), "v"(v) : "memory"); }
__device__ __forceinline__ unsigned fload(const unsigned* fp) { return __hip_atomic_load(fp, __ATOMIC_RELAXED, __HIP_MEMORY_SCOPE_SYSTEM); }
__device__ __forceinline__ unsigned mbcnt(u64 m, unsigned base) { return __builtin_amdgcn_mbcnt_hi((unsigned)(m >> 32), __builtin_amdgcn_mbcnt_lo((unsigned)m, base)); }
template <int I> __device__ __forceinline__ void wlane(unsigned& v, unsigned x) { asm("v_writelane_b32 %0, %1, %2" : "+v"(v) : "s"(x), "n"(I)); }
__device__ __forceinline__ float mul0(float x, float y) { float r; asm("v_mul_legacy_f32 %0, %1, %2" : "=v"(r) : "v"(x), "v"(y)); return r; }

__device__ __forceinline__ bool expired(unsigned* tmo, unsigned call, u64 t0) {
  unsigned v;
  asm volatile("global_load_dword %0, %1, %2 sc1\n\ts_waitcnt vmcnt(0)" : "=&v"(v) : "v"(0), "s"(tmo) : "memory");
  if (__builtin_amdgcn_readfirstlane(v) != call && __builtin_amdgcn_s_memrealtime() - t0 <= DEADLINE) return false;
  asm volatile("global_store_dword %0, %1, %2 sc1\n\ts_nop 1" :: "v"(0), "v"(call), "s"(tmo) : "memory");
  return true;
}

constexpr int R0 = 8;
template <int HB> __host__ __device__ __forceinline__ unsigned pos_of(unsigned d, unsigned blk) {
  const int r = d < 4 ? blk : blk - HB + R0, c = r < R0 ? 0 : r - R0 < 32 - HB ? r - R0 : 32 - HB;
  return 3 * r + 4 * c + d - 1; }
template <int HB> __host__ __device__ __forceinline__ unsigned code_of(unsigned p) {
  static_assert(R0 + 32 - HB <= HB, "");
  if (p >= 128 - HB) return p - (128 - HB);
  const unsigned q = p - 3 * R0;
  if (q < 7 * (32 - HB)) return (q % 7 + 1) << 5 | (q / 7 + (q % 7 < 3 ? R0 : HB));
  if (p >= 3 * R0) p -= 4 * (32 - HB);
  return (p % 3 + 1) << 5 | p / 3; }

template <int WN> constexpr int younger(int j, int w) {
  int n = 1 + (j == 0 ? 2 : 0);
  for (int s = j + 8 - w + 1; s < j + 8; ++s) n += 2 + (s % 4 == 0 ? 2 : 0) + (s % 4 == (3 + WN) % 4 ? 1 : 0);
  return n; }
__device__ __forceinline__ void load16(u32x4& d, unsigned vo, const char* sb) { asm volatile("global_load_dwordx4 %0, %1, %2 sc0 sc1" : "=v"(d) : "v"(vo), "s"(sb) : "memory"); }
__device__ __forceinline__ void mark(unsigned vo, char* p) { asm volatile("s_nop 4\n\tglobal_store_dwordx4 %0, %1, %2 sc0 sc1\n\ts_nop 1" :: "v"(vo), "v"(u32x4{MARK, MARK, MARK, MARK}), "s"(p) : "memory"); }
template <int K> __device__ __forceinline__ void step0(u32x4& d, unsigned vq, unsigned vo, char* q, const char* sb) {
  asm volatile("s_waitcnt vmcnt(%5)\n\tglobal_store_dwordx4 %1, %0, %3 offset:%6 sc0 sc1\n\tglobal_load_dwordx4 %0, %2, %4 sc0 sc1"
               : "+v"(d) : "v"(vq), "v"(vo), "s"(q), "s"(sb), "n"(7 + K), "n"(K % 4 * 1024) : "memory"); }
__device__ __forceinline__ void keep(unsigned& a) { asm volatile("" : "+v"(a)); }
__device__ __forceinline__ void landed(unsigned& a, unsigned& b) { asm volatile("" : "+v"(a), "+v"(b)); }
__device__ __forceinline__ void draw_poll(unsigned& tv, unsigned& pv, unsigned vz, char* tp, unsigned vp, const char* pp, u64 pm) {
  asm volatile("s_mov_b64 exec, 1\n\tglobal_atomic_add %0, %2, %3, %4 sc0\n\ts_mov_b64 exec, %7\n\tglobal_load_dword %1, %5, %6 sc0 sc1\n\ts_mov_b64 exec, -1"
               : "+v"(tv), "+v"(pv) : "v"(vz), "v"(1), "s"(tp), "v"(vp), "s"(pp), "s"(pm) : "memory"); }
__device__ __forceinline__ void poll(unsigned& v, unsigned o, const char* p) { asm volatile("global_load_dword %0, %1, %2 sc0 sc1\n\ts_waitcnt vmcnt(0)" : "+v"(v) : "v"(o), "s"(p) : "memory"); }
template <int M, int J> __device__ __forceinline__ void loaded(u32x4& d, char*& q, unsigned& ns, unsigned& rl, unsigned len, char* dum, unsigned tag) {
  unsigned h, t, v;
  asm volatile("s_waitcnt vmcnt(%1)" : "+v"(d) : "n"(M) : "memory");
  asm volatile("s_lshr_b32 %1, %10, %13\n\ts_bitcmp1_b32 %10, 7\n\ts_cbranch_scc0 .LBB7%=_0\n\t"
               "v_readfirstlane_b32 %3, %6\n\tv_max_u32 %5, %8, %9\n\tv_max3_u32 %5, %6, %7, %5\n\tv_cmp_eq_u32 vcc, -1, %5\n\t"
               "s_and_b32 %1, %3, 31\n\ts_max_u32 %1, %1, 1\n\ts_mul_i32 %1, %1, -4\n\ts_xor_b32 %3, %3, %12\n\ts_lshl_b64 vcc, vcc, %1\n\ts_cselect_b32 %3, -1, %3\n\t"
               "s_or_b32 %4, %10, %14\n\ts_cmp_lt_i32 %3, 0\n\ts_cselect_b32 %4, %4, -1\n\ts_cselect_b32 %3, -1, %10\n\ts_and_b32 %2, %2, %4\n\t"
               "s_bitcmp1_b32 %3, %13\n\ts_cselect_b32 %1, 63, %1\n\ts_cselect_b64 %0, %11, %0\n.LBB7%=_0:"
               : "+s"(q), "=&s"(ns), "+s"(rl), "=&s"(h), "=&s"(t), "=&v"(v)
               : "v"(d[0]), "v"(d[1]), "v"(d[2]), "v"(d[3]), "s"(len), "s"(dum), "s"(tag), "n"(8 * J), "n"(~(63 << 8 * J)) : "scc", "vcc"); }
template <int N, bool FLAG>
__device__ __forceinline__ void await(unsigned vd, unsigned fv, char* fa, unsigned fr) {
  asm volatile("s_cmp_lg_u32 %0, 0\n\ts_cbranch_scc1 .LBB8%=_0\n\ts_waitcnt vmcnt(%1)\n.LBB8%=_0:" :: "s"(fr), "n"(N) : "memory", "scc");
  if (FLAG) asm volatile("s_mov_b64 exec, 1\n\tglobal_store_dword %0, %1, %2 sc0 sc1\n\ts_mov_b64 exec, -1" :: "v"(vd), "v"(fv), "s"(fa) : "memory"); }
template <int OFF> __device__ __forceinline__ void step(u32x4& d, unsigned vd, unsigned vo, char* q, const char* sb, unsigned ns, unsigned nl) {
  asm volatile("s_lshr_b64 exec, -1, %5\n\tglobal_store_dwordx4 %1, %0, %3 offset:%7 sc0 sc1\n\t"
               "s_lshr_b64 exec, -1, %6\n\tglobal_load_dwordx4 %0, %2, %4 sc0 sc1\n\ts_mov_b64 exec, -1\n\ts_nop 0"
               : "+v"(d) : "v"(vd), "v"(vo), "s"(q), "s"(sb), "s"(ns), "s"(nl), "n"(OFF) : "memory", "scc"); }
struct Unit { char* q; char* fa; unsigned len; };

template <int HB, int WN, int NG, int NSL, int WI, int WX, int TH, int TY, int MODE>
__device__ __forceinline__ void pusher(unsigned dl, unsigned b, unsigned w0, unsigned nw, unsigned lane) {
  constexpr int YB = 32 - HB, FL = (3 + WN) / 4, FJ = (3 + WN) % 4;
  static_assert(WN >= 5 && WN <= 7 && NSL % 4 == 0 && 6 * WX <= 128, "");
  __builtin_amdgcn_s_waitcnt(0);
  u64x4 ha, pa; unsigned rank;
  asm volatile("s_load_dwordx8 %1, %3, %4 offset:0x40\n\ts_load_dwordx8 %0, %3, 0\n\ts_load_dword %2, %3, 0x30"
               : "=&s"(ha), "=&s"(pa), "=&s"(rank) : "s"(__builtin_amdgcn_kernarg_segment_ptr()), "s"(dl * 32));
  const bool cross = dl >= 4;
  const unsigned vx = (lane & 15) * KD + (lane >> 4) * 16;
  const unsigned vw = (lane >> 2) * NL + (lane & 3) * 16;
  const unsigned vd = lane * 16;
  u32x4 buf[2][4];
  unsigned cv, vl[16], vq[2] = {vd + w0 * 16384, vd + w0 * 16384 + 4096};
  unsigned s0 = cross ? 16 * NL : 64, v0 = cross ? vw + (w0 >> 1) * (256 * NL) + (w0 & 1) * 64 : vx + (w0 >> 2) * (16 * KD) + (w0 & 3) * 1024;
  asm volatile("" : "+s"(s0));
  auto first = [&](int i) { vl[i] = i ? vl[i - 1] + s0 : v0; keep(vl[i]); };
  sfor<8>([&](auto i) { first(i); });
  unsigned vc = b * (CW * 4);
  keep(vc);
  asm volatile("s_waitcnt lgkmcnt(0)" : "+s"(ha), "+s"(pa), "+s"(rank));
  rank &= 7;
  const char* ax = (const char*)ha[0]; char* myws = (char*)ha[2];
  const Peer pr = {(char*)pa[0], nullptr, (const char*)pa[2], (char*)pa[3]};
  char* pws = pr.ws;
  asm volatile("global_load_dword %0, %1, %2 sc1" : "=v"(cv) : "v"(vc), "s"(myws + CNT_OFF) : "memory");
  sfor<8>([&](auto i) { load16(buf[i / 4][i % 4], vl[i], pr.src); });
  sfor<8>([&](auto i) { first(i + 8); });
  asm volatile("s_waitcnt vmcnt(8)" : "+v"(cv), "+v"(vq[0]), "+v"(vq[1]) :: "memory");
  sfor<8>([&](auto ii) { constexpr int i = ii; step0<i>(buf[i / 4][i % 4], vq[i / 4] + ((cv & 1) << ALT), vl[i + 8], pr.dst, pr.src); });
  const unsigned call = __builtin_amdgcn_readfirstlane(cv) + 1, total = cross ? 128 + YB * 16 : HB * 16;
  char* dum = myws + DUM_OFF + (dl * 16 + w0) * 8192;
  sfor<4>([&](auto i) { mark(vd + i * 1024, dum); });
  auto id_of = [&](unsigned u) { return !cross ? u : u < 128 ? 512 + u : HB * 16 + u - 128; };
  auto src_of = [&](int id) { return id >= 512 ? pr.src + (size_t)(((id - 512) >> 3) * 4 + (id & 3)) * (64 * NL) + (id & 4) * 16 : ax + (size_t)(id >> 4) * (16 * KD) + (id & 15) * 256; };
  unsigned* tmo = (unsigned*)(myws + TMO_OFF);
  const unsigned ps = pos_of<HB>(dl, lane & 31), vt = XF_OFF + (ps % NG * 256 + ps / NG) * 4;
  char* dx = pws + XG_OFF + (size_t)rank * SHARD + ((~call & 1) << ALT);
  char* dw = pr.dst - 512 * 4096 + ((~call & 1) << ALT);
  const unsigned need = lane ? call : call * (cross ? TY : TH), vm = lane ? lane * 4 : MF_OFF - CF_OFF + 128 + (dl & 4) * 128;
  const char* mb = myws + CF_OFF + (dl & 3) * 128 - 128;
  u64 pm = cross ? ((1ull << 2 * NG) - 1) << 32 | 1 : 1;
  const unsigned all = total + (cross && !(MODE & 4) ? NSL / 4 : 0);
  const unsigned runs = total - (cross ? TY : TH) * 16, rbase = (call - 1) * (runs / 4 - nw) - 2 * nw;
  unsigned nst = 8, cur = 0, nxt = (w0 + nw) * 4;
  asm volatile("" : "+s"(nst), "+s"(cur));
  unsigned vcall = call, lim = runs, rl = 0x3f3f3fbf, nr = 0;
  char* rq = dum;
  asm volatile("" : "+s"(rl), "+s"(nr));
  const long pld = (myws + PL_OFF + (dl & 3) * (512 * 1024)) - (pws + PK_OFF + (rank & 3) * (512 * 1024));
  keep(vcall);
  auto xw_unit = [&](int i, bool f) { return Unit{(i >= 512 ? dw : dx) + (size_t)i * 4096,
    !f ? dum + 4096 : pws + (i >= 512 ? WF_OFF + ((rank & 3) * 128 - 515 + i) * 16 : __builtin_amdgcn_readlane(vt, i >> 4) + (i & 12) * 64), 0}; };
  const Unit empty = {dum, dum + 4096, 0x3f3f3f3f};
  Unit pu[5];
  sfor<4>([&](auto i) { pu[i] = xw_unit(w0 * 4 + i + cross * 512, i == 3); });
  asm volatile("" : "+s"(pu[2].len), "+s"(pu[3].len));
  const unsigned tbase = (call - 1) * (all - runs + 2 * nw) - runs;
  unsigned tv[2] = {0, 0}, tk[2] = {0, 0}, idle = 0, n = 4;
  unsigned pv[2] = {0, 0};
  bool drew[2] = {false, false};
  const u64 t0 = __builtin_amdgcn_s_memrealtime();
  for (unsigned it = 0;; ++it) {
    bool done = false;
    sfor<2>([&](auto ic) { constexpr int c = ic;
      if (!done) {
        landed(tv[c], pv[c]);
        if (drew[c]) tk[c] = __builtin_amdgcn_readfirstlane(tv[c]) - tbase;
        const bool dr = c == 0 && n < nst && !(n & 2);
        if (dr) cur = nxt;
        else if (c == 0 && n < nst) { nxt = (__builtin_amdgcn_readfirstlane(tv[c]) - rbase) * 4; nst += nxt < runs ? 4 : 0; }
        if (n < nst) tk[c] = cur + (n & 3);
        ++n;
        if (lim < total) {
          u64 no = __ballot((int)(pv[c] - need) < 0) & pm;
          if (tk[c] >= lim) for (unsigned i = 1; no; ++i) {
            poll(pv[c], vm, mb);
            no = __ballot((int)(pv[c] - need) < 0) & pm;
            if ((i & 63) == 0 && expired(tmo, call, t0)) no = 0; }
          if (!no) { lim = total; pm = 2; mb = dum + 6144; } }
        const char* sb; unsigned vo = vd; int st = 1024;
        drew[c] = n + 1 >= nst;
        if (rl != 0x3f3f3fbf) {
          pu[4] = Unit{rq, dum + 4096, rl}; sb = rq + pld; idle = idle < FL ? idle + 1 : FL; drew[c] = false;
          __builtin_amdgcn_s_sleep(MODE >> 12 & 15);
          if ((++nr & 63) == 0 && expired(tmo, call, t0)) { pu[4] = empty; tk[c] = all; }
        } else if (tk[c] < total) { const int i = id_of(tk[c]); pu[4] = xw_unit(i, tk[c] < runs && (i & 3) == 3); sb = src_of(i); vo = i >= 512 ? vw : vx; st = i >= 512 ? 16 * NL : 64; }
        else if (tk[c] < all) {
          idle = idle < FL ? idle + 1 : FL;
          pu[4] = Unit{pws + PK_OFF + ((rank & 3) * 128 + tk[c] - total) * 4096, dum + 4096, rl & 0x80};
          sb = pu[4].q + pld;
        } else { pu[4] = empty; sb = ax; ++idle; drew[c] = false; }
        rl |= 0x3f3f3f3f; rq = pu[2].q;
        const unsigned fr = n > nst + 2 + FL && (MODE & (cross ? 32 : 16)) && tk[c] + (MODE >> 16 & 255) - all < (MODE >> 24 ? MODE >> 24 : 1u  __H0__ ) ? n : 0;
        draw_poll(tv[c], pv[c], vd, drew[c] || dr ? myws + TK_OFF + dl * 128 + (dr ? 24 * 128 : 0) : dum + 5120, vm, mb, pm);
        sfor<4>([&](auto jj) { constexpr int j = jj;
          char* q = pu[2].q; unsigned ns;
          loaded<younger<WN>(j, 8) - (j == 3 ? 3 : 1), j>(buf[c][j], q, ns, rl, pu[2].len, dum, call << 31);
          await<younger<WN>(j, WN), j == FJ>(vd, vcall, pu[2 - FL].fa, fr);
          step<j * 1024>(buf[c][j], vd, vo, q, sb + j * st, ns, pu[4].len >> 8 * j);
        });
        done = idle >= 2 + FL && tk[0] >= all && tk[1] >= all && rl == 0x3f3f3fbf;
        sfor<4>([&](auto ii) { constexpr int i = ii; pu[i] = pu[i + 1]; });
      } });
    if (done) break;
  }
  asm volatile("s_waitcnt vmcnt(0)" :: "v"(buf[0][0]), "v"(buf[0][1]), "v"(buf[0][2]), "v"(buf[0][3]), "v"(buf[1][0]), "v"(buf[1][1]), "v"(buf[1][2]), "v"(buf[1][3]),
               "v"(tv[0]), "v"(tv[1]), "v"(pv[0]), "v"(pv[1]) : "memory");
  if (lane == 0) __hip_atomic_store((GAS unsigned*)(myws + CNT_OFF) + b * CW, call, __ATOMIC_RELAXED, __HIP_MEMORY_SCOPE_AGENT);
  {
    unsigned k;
    asm volatile("s_mov_b64 exec, 1\n\tglobal_atomic_add %0, %1, %2, %3 sc0\n\ts_mov_b64 exec, -1\n\ts_waitcnt vmcnt(0)"
                 : "=&v"(k) : "v"(vd), "v"(1), "s"(myws + TK_OFF + (16 + dl) * 128) : "memory");
    const unsigned p = pos_of<HB>(dl, (cross ? 31 : HB - 1) - (lane >> 2));
    if (__builtin_amdgcn_readfirstlane(k) + 1 == call * nw && lane < (cross ? TY : TH) * 4) fstore((unsigned*)(pws + XF_OFF) + p % NG * 256 + p / NG + (lane & 3) * 64, call); }
}

template <int HB, int WN, int WI, int WX, int NG, int DH, int DY, int RB, int TH, int TY, int HINT, int SL, int FB, int MODE>
__global__ void __launch_bounds__(64 * CW, 4) ag_gemm(Args a) {
  extern __shared__ __attribute__((aligned(16))) char lds[];
  constexpr int NP = 3 * WI + 4 * WX, NPOS = 160 - HB, NUW = (NG - 3) * 16;
  constexpr int NSL = (3 * DH + HB + 3 * DY) * 4, NDH = HB - RB - DH, NFL = NSL + (NDH + 32 - HB - DY) * 12;
  static_assert(NFL <= 512 && 4 * NFL + 48 * RB <= CW * NUW && WI <= 16 && WX <= 16, "");
  static_assert(DH <= HB - RB && NG * 15 >= NPOS && NG <= 16 && NG % 2 == 0 && TY <= 32 - HB - DY && NP <= 85, "");
  const int lane = threadIdx.x & 63, wid = __builtin_amdgcn_readfirstlane(threadIdx.x >> 6);
  const int b = blockIdx.x;
  if (__builtin_expect(b < NP && wid == 0, 1)) {
    constexpr unsigned WM = WI < WX ? WI : WX, WL = WI > WX ? 3 : 4;
    unsigned w0 = b * 37u >> 8, d = 1 + b - w0 * 7;
    if (WI != WX && b >= 7 * WM) { d = (WI > WX ? 1 : 4) + (b - 7 * WM) % WL; w0 = WM + (b - 7 * WM) / WL; }
    pusher<HB, WN, NG, NSL, WI, WX, TH, TY, MODE>(d & 7, b, w0, d >= 4 ? WX : WI, lane);
    return;
  }
  if (b < NP || (MODE & 8)) return;
  const int cb = b - NP, grp = (cb >> 5) * 2 + ((cb >> 2) & 1), ns = (cb & 3) * 4 + ((cb >> 3) & 3);
  const bool ownw = (ns & 1) == 0;
  const int d = ownw ? 0 : 4 + (ns >> 2);
  const char* ax = a.x; const char* aw = a.w; char* myws = a.ws; char* dws = a.p[d].ws; float* dout = a.p[d].out;
  const unsigned rank = a.rank & 7;
  asm volatile("" :: "s"(ax), "s"(aw), "s"(myws), "s"(dws), "s"(dout), "s"(rank));
  unsigned* cnt = (unsigned*)(myws + CNT_OFF) + b * CW + wid;
  typedef const __attribute__((address_space(4))) float* cfp;
  const float sc = *(cfp)a.sx * *(cfp)a.sw, scale = fabsf(sc);
  const bool neg = sc < 0.f;
  unsigned* tmo = (unsigned*)(myws + TMO_OFF);
  LAS char* scr = (LAS char*)lds + wid * SCR;

  asm volatile("" ::: "memory");
  unsigned call = __hip_atomic_load(cnt, __ATOMIC_RELAXED, __HIP_MEMORY_SCOPE_AGENT);
  const unsigned dest = rank ^ d;
  const int col0 = ownw ? (ns >> 1) * 64 : (rank & 3) * 128 + (ns & 2) * 32;
  LAS int* arr = (LAS int*)((LAS char*)lds + CW * SCR);
  if (threadIdx.x < 32) arr[threadIdx.x] = 0;
#pragma unroll
  for (int i = 0; i < 4; ++i) *(LAS u32x4*)((LAS char*)lds + ACC_OFF + i * 16384 + threadIdx.x * 16) = u32x4{0, 0, 0, 0};
  bool bad = false;
  i32x4 wf[4][4];
  if (ownw) {
    const __amdgpu_buffer_rsrc_t rw = __builtin_amdgcn_make_buffer_rsrc((void*)(aw + (size_t)wid * (256 * NL) + col0), 0, 256 * NL, 0x00020000);
#pragma unroll
    for (int i = 0; i < 16; ++i) wf[i >> 2][i & 3] = __builtin_bit_cast(i32x4, __builtin_amdgcn_raw_buffer_load_b128(rw, (lane >> 2) * NL + (lane & 3) * 16, i * 16 * NL, 0));
  }
  call = __builtin_amdgcn_readfirstlane(call) + 1;
  if (lane == 0) __hip_atomic_store(cnt, call, __ATOMIC_RELAXED, __HIP_MEMORY_SCOPE_AGENT);
  const unsigned pz = (lane & 15) * NG + grp, code = code_of<HB>(pz) ^ rank << 5;
  const unsigned late = (unsigned)__ballot(pz < 128 - HB && (code & 31) >= ((code & 31) < HB ? HB - TH : 32 - TY)) & 0xffff;
  const unsigned xo = XG_OFF + ((~call & 1) << ALT) + wid * 4096;
  if (!ownw) {
    const u32x4 mk = {MARK, MARK, MARK, MARK};
    for (unsigned u = (grp * 2 + ((ns >> 1) & 1)) * CW + wid; u < NFL + 24 * RB; u += 2 * NG * CW) {
      const unsigned j = u - NFL;
      gstore_wt<0>(u < NFL ? myws + PK_OFF + ((dest & 3) * 512 + u) * 1024 + lane * 16
                           : (char*)(a.out + ((dest ^ (j / (RB * 8 + !RB) + 1)) * MS + (HB - RB) * 16 + j % (RB * 8 + !RB) / 2 * 4 + (lane >> 4)) * NL + (dest & 3) * 128 + (j & 1) * 64 + (lane & 15) * 4), mk); }
    for (unsigned u = (grp * 2 + ((ns >> 1) & 1)) * CW + wid; u < NSL; u += 2 * NG * CW) gstore_wt<0>(myws + PL_OFF + ((dest & 3) * 512 + u) * 1024 + lane * 16, mk);
    if (ns == 1) for (unsigned m = late; m; m &= m - 1) {
      char* q = myws + xo + __builtin_amdgcn_readlane(code, __builtin_ctz(m)) * (16 * KD) + lane * 16;
      gstore_wt<0>(q, mk); gstore_wt<1024>(q, mk); gstore_wt<2048>(q, mk); gstore_wt<3072>(q, mk); }
    waitvm0();
  }
  asm volatile("s_waitcnt lgkmcnt(0)" ::: "memory");
  __builtin_amdgcn_s_barrier();
  asm volatile("" ::: "memory");
  if (!ownw) {
    if (wid == 0) {
      if (lane < 2) fstore((unsigned*)((lane ? dws : myws) + CF_OFF) + (lane ? 128 + (rank & 3) * 32 : (dest & 3) * 32) + grp * 2 + ((ns >> 1) & 1), call);
      if (ns == 1 && lane < 4) fstore((unsigned*)(myws + XF_OFF) + grp * 256 + lane * 64 + 15, call);
      if (ns == 1) for (unsigned m = late; m; m &= m - 1) {
        const unsigned l = ((__builtin_amdgcn_readlane(code, __builtin_ctz(m)) >> 5) ^ rank) & 7;
        if (lane == 0) __hip_atomic_fetch_add((unsigned*)(a.p[l].ws + MF_OFF + l * 128), 1, __ATOMIC_RELAXED, __HIP_MEMORY_SCOPE_SYSTEM); } }
    const unsigned* fw = lane < 32 ? (const unsigned*)(myws + WF_OFF) + (dest & 3) * 512 + wid * 32 + (ns & 2) * 8
                                   : (const unsigned*)(myws + CF_OFF) + 128 + (dest & 3) * 32 + lane % (2 * NG);
    const u64 t0 = __builtin_amdgcn_s_memrealtime();
    for (unsigned it = 1; __ballot(fload(fw) < call); ++it) {
      if ((it & 63) == 0 && expired(tmo, call, t0)) { bad = true; break; }
      __builtin_amdgcn_s_sleep(4); }
    asm volatile("" ::: "memory");
    const __amdgpu_buffer_rsrc_t rw = __builtin_amdgcn_make_buffer_rsrc((void*)(myws + WQ_OFF + ((~call & 1) << ALT) + (size_t)(dest & 3) * (SHARD / 4) + wid * 32768 + (ns & 2) * 8192), 0, 16384, 0x00020000);
#pragma unroll
    for (int i = 0; i < 16; ++i) wf[i >> 2][i & 3] = __builtin_bit_cast(i32x4, __builtin_amdgcn_raw_buffer_load_b128(rw, lane * 16 + (i & 3) * 1024, (i >> 2) * 4096, 17));
  }
  {
    LAS char* wr = scr + ((lane >> 1) & 1) * 2048 + (lane >> 2) * 32;
    LAS char* rb = scr + (16 * (lane >> 4) + ((lane & 15) >> 1)) * 32 + 8 * (lane & 1);
#pragma unroll
    for (int i = 0; i < 4; ++i) {
#pragma unroll
      for (int j = 0; j < 4; ++j) *(LAS i32x4*)(wr + j * 512 + (((lane ^ j) & 1) * 16)) = wf[i][j];
      asm volatile("" ::: "memory");
#pragma unroll
      for (int t = 0; t < 4; ++t) {
        LAS char* p = rb + (t >> 1) * 2048 + 16 * (((lane >> 4) ^ t) & 1);
        const i32x2 u0 = __builtin_amdgcn_ds_read_tr8_b64_v2i32((LAS i32x2*)p), u1 = __builtin_amdgcn_ds_read_tr8_b64_v2i32((LAS i32x2*)(p + 256));
        wf[i][t] = i32x4{u0[0], u0[1], u1[0], u1[1]}; }
      asm volatile("" ::: "memory");
    }
  }
  unsigned todo = (unsigned)__ballot(pz < (ownw ? NPOS : 128) && (ownw || (code >> 5) != dest)) & 0xffff;
  const unsigned mine = (unsigned)__ballot(pz >= 128 - HB) & 0x7fff;
  const unsigned pofs = ((late & todo) << 16 >> (lane & 31)) & 1 ? xo + code * (16 * KD) : XF_OFF + ((grp * 16 + (wid & 12)) * 16 + (lane & 15)) * 4;
  unsigned sure = bad || (MODE & 1) ? 0xffff : mine, ready = sure;
  const unsigned src = code >> 5, blk = code & 31;
  const bool home = blk < HB && src != rank;
  const unsigned dirm = __ballot(home ? blk >= DH : blk >= HB + DY), pkm = ownw ? 0 : ~(unsigned)__ballot(home && blk >= HB - RB);
  const unsigned info = (NSL + ((home ? blk - DH : blk - (HB + DY) + NDH) * 3 + ((src ^ (home ? rank : dest)) & 3) - 1) * 4 + (ns & 2)) << 16
                        | (((blk * 4 + (src & 3)) * 4 + (rank & 3)) * 4 + (ns & 2)) << 5;
  unsigned fast = !ownw || FB / 1000 ? __ballot(home && blk >= HB - FB % 10) : 0;
  char* oc = (char*)(dout + col0);
  const unsigned thr = wid > 10 ? wid - 9 : 1, c16 = code * (16 * KD);
  typedef int i32x16 __attribute__((ext_vector_type(16)));
  i32x16 wr[4];
#pragma unroll
  for (int i = 0; i < 4; ++i) wr[i] = __builtin_shufflevector(__builtin_shufflevector(wf[i][0], wf[i][1], 0, 1, 2, 3, 4, 5, 6, 7), __builtin_shufflevector(wf[i][2], wf[i][3], 0, 1, 2, 3, 4, 5, 6, 7),
                                                              0, 1, 2, 3, 4, 5, 6, 7, 8, 9, 10, 11, 12, 13, 14, 15);
  const int ntile = __builtin_popcount(todo);
  const u64 t0 = __builtin_amdgcn_s_memrealtime();
  { const unsigned* mf = (const unsigned*)(myws + CF_OFF) + (ns & 12) * 8 + lane % (2 * NG);
    for (unsigned it = 1; __ballot(fload(mf) < call); ++it) {
      if ((it & 63) == 0 && expired(tmo, call, t0)) { bad = true; sure = ready = 0xffff; break; }
      __builtin_amdgcn_s_sleep(4); } }
  __builtin_amdgcn_s_waitcnt(0);
  for (;;) {
#define X4(x, i) "v[" #x "+4*" #i ":" #x "+4*" #i "+3]"
#define L4(x, base, st) "global_load_dwordx4 " X4(x, 0) ", v2, " base " sc0 sc1\n\tglobal_load_dwordx4 " X4(x, 1) ", v2, " base " offset:" st " sc0 sc1\n\t" \
                        "global_load_dwordx4 " X4(x, 2) ", v2, " base " offset:2*" st " sc0 sc1\n\tglobal_load_dwordx4 " X4(x, 3) ", v2, " base " offset:3*" st " sc0 sc1\n\t"
#define LB(n) ".LBB9%=_" #n
#define AT(n) "\n" LB(n) ":\n\t"
#define POLL "global_load_dword v1, %[pofs], %[ws] sc0 sc1\n\t"
#define LOAD(x, S, n) "s_ff1_i32_b32 " S ", %[m]\n\tv_readlane_b32 %[t], %[c16], " S "\n\ts_bitset0_b32 %[todo], " S "\n\ts_bitcmp1_b32 %[ml], " S "\n\ts_cbranch_scc1 " LB(n##1) "\n\t" \
                      "v_add_u32 v2, %[t], %[vg]\n\t" L4(x, "%[ws]", "1024") AT(n##2) POLL
#define OWN(x, n) AT(n##1) "v_add_u32 v2, %[t], %[vx]\n\t" L4(x, "%[xm]", "64") "s_branch " LB(n##2)
#define SEEN "s_waitcnt vmcnt(0)\n\tv_cmp_le_u32 vcc, %[call], v1\n\ts_and_b32 %[t], vcc_lo, 0xffff\n\tv_cmp_ne_u32 vcc, -1, v1\n\ts_or_b32 %[sure], %[sure], %[t]\n\t" \
             "s_and_b32 %[t], vcc_lo, %[ml]\n\ts_lshr_b32 %[t], %[t], 16\n\ts_bitcmp1_b32 %[sure], 15\n\ts_cselect_b32 %[t], %[t], 0\n\ts_or_b32 %[ready], %[sure], %[t]\n\t"
#define MX(x, i) "v_max3_u32 v2, v2, v[" #x "+" #i "], v[" #x "+" #i "+1]\n\t"
#define MM(x, a, t) "v_mfma_i32_16x16x64_i8 " X4(4, a) ", " X4(64, t) ", " X4(x, 0) ", 0\n\tv_mfma_i32_16x16x64_i8 " X4(4, a) ", " X4(80, t) ", " X4(x, 1) ", " X4(4, a) "\n\t" \
                    "v_mfma_i32_16x16x64_i8 " X4(4, a) ", " X4(96, t) ", " X4(x, 2) ", " X4(4, a) "\n\tv_mfma_i32_16x16x64_i8 " X4(4, a) ", " X4(112, t) ", " X4(x, 3) ", " X4(4, a) "\n\t"
#define AD(a, t) "ds_add_u32 v2, v[4+4*" #a "] offset:1024*" #t "\n\tds_add_u32 v2, v[5+4*" #a "] offset:1024*" #t "+256\n\t" \
                 "ds_add_u32 v2, v[6+4*" #a "] offset:1024*" #t "+512\n\tds_add_u32 v2, v[7+4*" #a "] offset:1024*" #t "+768\n\t"
#define COMP(x, S, y, T, n) AT(n##0) "s_and_b32 %[m], %[ready], %[todo]\n\ts_cbranch_scc0 " LB(n##3) "\n\t" LOAD(y, T, n) "s_waitcnt vmcnt(6)\n\ts_branch " LB(n##4) \
      AT(n##3) "s_waitcnt vmcnt(1)" AT(n##4) "s_bitcmp1_b32 %[sure], " S "\n\ts_cbranch_scc1 " LB(n##5) "\n\t" \
      CHK(x) "s_cbranch_vccnz " LB(n##8) \
      AT(n##5) MM(x, 0, 0) MM(x, 1, 1) "v_lshl_add_u32 v2, " S ", 12, %[acc]\n\ts_lshl2_add_u32 %[t], " S ", %[arr]\n\tv_mov_b32 v0, %[t]\n\tv_mov_b32 v3, %[inc]\n\t" \
      AD(0, 0) MM(x, 0, 2) AD(1, 1) MM(x, 1, 3) AD(0, 2) "s_nop 3\n\t" AD(1, 3) \
      "s_mov_b64 exec, 1\n\tds_add_rtn_u32 v3, v0, v3\n\ts_mov_b64 exec, -1\n\ts_cmp_lg_u32 %[m], 0\n\ts_cbranch_scc1 " LB(n##6) "\n\t" SEEN \
      AT(n##6) "s_waitcnt lgkmcnt(0)\n\tv_readfirstlane_b32 %[t], v3\n\ts_and_b32 %[t], %[t], 0xff\n\ts_cmp_lg_u32 %[t], 15\n\ts_cbranch_scc1 " LB(n##7) "\n\t" \
      "s_lshl_b32 %[e], " S ", 1\n\ts_and_b32 %[t], %[ready], %[todo]\n\ts_or_b32 %[t], %[t], %[m]\n\ts_cselect_b32 vcc_lo, 3, 2\n\tv_mov_b32 v2, vcc_lo\n\tv_lshlrev_b32 v2, %[e], v2\n\t" \
      "v_mov_b32 v0, %[arr]\n\tv_mov_b32 v3, 1\n\ts_mov_b64 exec, 1\n\tds_add_u64 v0, v[2:3] offset:64\n\ts_mov_b64 exec, -1\n\ts_cbranch_scc0 " LB(99) \
      AT(n##7) "s_cmp_lg_u32 %[m], 0\n\t"
#define FP(r) "global_load_dword v" #r ", %[pofs], %[ws] sc0 sc1\n\ts_sleep %[fs]\n\t"
#define FC(r) "s_waitcnt vmcnt(3)\n\tv_cmp_ne_u32 vcc, -1, v" #r "\n\ts_cbranch_vccnz " LB(98) "\n\t"
#define CHK(x) "v_max_u32 v2, v[" #x "], v[" #x "+1]\n\t" MX(x, 2) MX(x, 4) MX(x, 6) MX(x, 8) MX(x, 10) MX(x, 12) MX(x, 14) "v_cmp_eq_u32 vcc, -1, v2\n\t"
#define FAIL(S, n, k) AT(n##8) "s_bitset1_b32 %[todo], " S "\n\ts_bitset0_b32 %[ready], " S "\n\ts_cmp_lg_u32 %[m], 0\n\ts_cbranch_scc1 " LB(k##0) "\n\ts_branch " LB(92)
    unsigned m, sa, sb, t, n; int e;
    asm volatile("s_mov_b32 %[n], 64\n\ts_and_b32 %[m], %[ready], %[todo]\n\ts_cbranch_scc1 " LB(10)
                 AT(91) "s_cmp_eq_u32 %[todo], 0\n\ts_cbranch_scc1 " LB(94) "\n\t" POLL
                 AT(92) "s_mov_b32 %[e], -1\n\t" SEEN "s_sub_u32 %[n], %[n], 1\n\ts_cbranch_scc1 " LB(99)
                 AT(93) "s_and_b32 %[m], %[ready], %[todo]\n\ts_cbranch_scc0 " LB(94)
                 AT(10) "s_setprio 3\n\t" LOAD(12, "%[sa]", 1)
                 COMP(12, "%[sa]", 28, "%[sb]", 2) "s_cbranch_scc0 " LB(93) COMP(28, "%[sb]", 12, "%[sa]", 3) "s_cbranch_scc1 " LB(20) "\n\ts_branch " LB(93)
                 OWN(12, 1) OWN(28, 2) OWN(12, 3) FAIL("%[sa]", 2, 3) FAIL("%[sb]", 3, 2)
                 AT(95) "v_readfirstlane_b32 %[t], v3\n\ts_mov_b32 %[e], -2\n\ts_lshr_b32 %[m], %[np], 1\n\ts_xor_b32 %[t], %[t], %[m]\n\ts_or_b32 %[t], %[t], %[todo]\n\ts_cbranch_scc0 " LB(99)
                        "\n\tv_readfirstlane_b32 %[t], v3\n\ts_setprio 0\n\ts_sub_u32 %[m], %[m], %[t]\n\ts_lshr_b32 %[t], %[sure], 15\n\ts_cmp_eq_u32 %[m], %[t]\n\ts_cselect_b32 %[t], %[todo], 0\n\t"
                        "s_and_b32 %[t], %[t], %[fast]\n\ts_cbranch_scc1 " LB(96) "\n\ts_sleep %[slp]\n\ts_branch " LB(91)
                 AT(96) "s_ff1_i32_b32 %[sa], %[todo]\n\ts_mov_b32 %[fast], 0\n\ts_mov_b32 %[sb], %[sa]\n\ts_lshl_b64 exec, 0x10000, %[sa]\n\ts_mov_b32 %[m], 31\n\t"
                        "v_readlane_b32 %[t], %[c16], %[sa]\n\t" FP(28) FP(29) FP(30)
                 AT(97) FP(31) FC(28) FP(28) FC(29) FP(29) FC(30) FP(30) FC(31) "s_sub_u32 %[m], %[m], 1\n\ts_cbranch_scc0 " LB(97) "\n\ts_mov_b64 exec, -1\n\ts_branch " LB(91)
                 AT(98) "s_mov_b64 exec, -1\n\ts_setprio 3\n\ts_mov_b32 %[todo], 0\n\ts_mov_b32 %[m], 0\n\ts_mov_b32 %[e], 3"
                 AT(89) "v_add_u32 v2, %[t], %[vg]\n\t" L4(12, "%[ws]", "1024") "s_sleep %[fp]\n\t" L4(28, "%[ws]", "1024")
                        "s_waitcnt vmcnt(4)\n\t" CHK(12) "s_cbranch_vccz " LB(25) "\n\ts_waitcnt vmcnt(0)\n\t" CHK(28) "s_cbranch_vccz " LB(35)
                        "\n\ts_sub_u32 %[e], %[e], 1\n\ts_cbranch_scc0 " LB(89) "\n\ts_bitset1_b32 %[todo], %[sa]\n\ts_branch " LB(91)
                 AT(94) "v_mov_b32 v0, %[arr]\n\tds_read_b64 v[2:3], v0 offset:64\n\ts_and_b32 %[sa], %[np], 1\n\ts_lshl_b32 %[sa], 0x55555555, %[sa]\n\ts_waitcnt lgkmcnt(0)\n\t"
                        "v_readfirstlane_b32 %[t], v2\n\ts_cmp_eq_u32 %[t], 0\n\ts_cbranch_scc1 " LB(95) "\n\ts_and_b32 %[m], %[t], %[sa]\n\ts_cselect_b32 %[t], %[m], %[t]\n\t"
                        "s_ff1_i32_b32 %[e], %[t]\n\ts_mov_b32 %[t], -1\n\ts_bitset0_b32 %[t], %[e]\n\tv_mov_b32 v2, %[t]\n\t"
                        "s_mov_b64 exec, 1\n\tds_and_rtn_b32 v2, v0, v2 offset:64\n\ts_mov_b64 exec, -1\n\ts_waitcnt lgkmcnt(0)\n\t"
                        "v_readfirstlane_b32 %[t], v2\n\ts_bitcmp1_b32 %[t], %[e]\n\ts_cbranch_scc0 " LB(91)
                 AT(99)
                 : [todo] "+s"(todo), [sure] "+s"(sure), [ready] "+s"(ready), [fast] "+s"(fast), [e] "=&s"(e), [m] "=&s"(m), [sa] "=&s"(sa), [sb] "=&s"(sb), [t] "=&s"(t), [n] "=&s"(n)
                 : [ml] "s"(mine | late << 16), [call] "s"(call), [ws] "s"(myws), [xm] "s"(ax + wid * 256 - rank * SHARD), [inc] "s"(bad ? 0x101 : 1), [arr] "s"(arr), [np] "s"(ntile * 2 + (wid & 1)), [slp] "n"(MODE & 64 ? 16 : 1), [fs] "n"(FB / 10 % 10), [fp] "n"(FB / 100 % 10),
                   [c16] "v"(c16), [pofs] "v"(pofs), [vg] "v"(lane * 16 + xo), [vx] "v"((lane & 15) * KD + (lane >> 4) * 16), [acc] "v"((LAS char*)lds + ACC_OFF + lane * 4),
                   "{v[64:79]}"(wr[0]), "{v[80:95]}"(wr[1]), "{v[96:111]}"(wr[2]), "{v[112:127]}"(wr[3])
                 : "memory", "scc", "vcc", "v0", "v1", "v2", "v3", "v4", "v5", "v6", "v7", "v8", "v9", "v10", "v11", "v12", "v13", "v14", "v15", "v16", "v17", "v18", "v19", "v20", "v21",
                   "v22", "v23", "v24", "v25", "v26", "v27", "v28", "v29", "v30", "v31", "v32", "v33", "v34", "v35", "v36", "v37", "v38", "v39", "v40", "v41", "v42", "v43");
    if (__builtin_expect(e < 0, 0)) {
      if (e == -2) break;
      if (expired(tmo, call, t0)) { bad = true; sure = ready = 0xffff; }
      continue; }
    {
      const int s = e >> 1, h = e & 1;
      const LAS int* sa = (LAS int*)((LAS char*)lds + ACC_OFF + e * 2048) + lane;
      int acc[8]; f32x4 y[2];
#pragma unroll
      for (int j = 0; j < 8; ++j) acc[j] = sa[j * 64];
      const int na = arr[s];
      __builtin_amdgcn_sched_barrier(0);
      __builtin_amdgcn_s_setprio(2);
      const bool direct = (dirm >> s) & 1, packed = (pkm >> s) & 1;
      const unsigned inf = __builtin_amdgcn_readlane(info, s), hd = (inf & 0xffff) | h << 5 | call << 31;
      unsigned slot = (inf >> 16) + h;
      if (packed && !direct) asm volatile("s_mov_b64 exec, 1\n\tglobal_atomic_add %0, %1, %3, %2 sc0 sc1\n\ts_mov_b64 exec, -1"
                                          : "+v"(slot) : "v"(0), "s"(myws + TK_OFF + (4 + d) * 128), "v"(1) : "memory");
      char* pk = direct ? dws + PK_OFF + (rank & 3) * (512 * 1024) : myws + PL_OFF + (d & 3) * (512 * 1024);
      char* o = oc + ((unsigned)__builtin_amdgcn_readlane(c16, s) / 2 + h * 128);
      const unsigned vo = (lane & 15) * (NL * 4) + (lane & 48);
      __builtin_amdgcn_sched_barrier(0);
      const bool bad = (__builtin_amdgcn_readfirstlane(na) >> 8) != 0;
      if (__builtin_expect(neg, 0)) {
        asm volatile("");
#pragma unroll
        for (int j = 0; j < 8; ++j) acc[j] = -acc[j]; }
      auto getslot = [&]() { asm volatile("s_waitcnt vmcnt(0)" : "+v"(slot) :: "memory");
        return (__builtin_amdgcn_readfirstlane(slot) - (direct ? 0 : (call - 1) * NSL)) & 511; };
      int n64 = 0;
      if (packed && !bad) {
        int n = 0; unsigned kb = 0, pos = 0, nib = 0, t; u64 mk[8]; const int* v = acc;
#pragma unroll
        for (int j = 0; j < 8; ++j) { mk[j] = __ballot(v[j] > 0); n += __builtin_popcountll(mk[j]); pos = mbcnt(mk[j], pos); }
        const int nb = 68 + ((n + 1) & ~1) * 2, e0 = nb + ((n + 7) >> 3) * 4;
        int eb = e0;
        *(LAS unsigned*)(scr + nb + lane * 4) = 0;
        LAS char* lp = scr + 68 + 2 * pos;
#define WR(j, k) "s_mov_b64 exec, %" #k "\n\tds_write_b16 %0, %" #j "\n\tv_bfe_u32 %2, %" #j ", 16, 4\n\tv_add_u32 %0, 2, %0\n\tv_lshl_or_b32 %1, %1, 4, %2\n\t"
        asm volatile(WR(3, 11) WR(4, 12) WR(5, 13) WR(6, 14) WR(7, 15) WR(8, 16) WR(9, 17) WR(10, 18) "s_mov_b64 exec, -1"
                     : "+v"(lp), "+v"(nib), "=&v"(t)
                     : "v"(v[0]), "v"(v[1]), "v"(v[2]), "v"(v[3]), "v"(v[4]), "v"(v[5]), "v"(v[6]), "v"(v[7]),
                       "s"(mk[0]), "s"(mk[1]), "s"(mk[2]), "s"(mk[3]), "s"(mk[4]), "s"(mk[5]), "s"(mk[6]), "s"(mk[7]) : "memory");
        LAS unsigned* np = (LAS unsigned*)(scr + nb) + (pos >> 3);
        const unsigned sh0 = (pos & 7) * 4;
        const u64 nv = (u64)nib << sh0;
        __hip_atomic_fetch_or(np, (unsigned)nv, __ATOMIC_RELAXED, __HIP_MEMORY_SCOPE_WAVEFRONT);
        __hip_atomic_fetch_or(np + 1, (unsigned)(nv >> 32), __ATOMIC_RELAXED, __HIP_MEMORY_SCOPE_WAVEFRONT);
        sfor<16>([&](auto ii) { constexpr int i = ii; wlane<i>(kb, mk[i / 2] >> i % 2 * 32); });
        if (lane < 16) *(LAS unsigned*)(scr + 4 + lane * 4) = kb;
        const int mx = max(max(max(v[0], v[1]), max(v[2], v[3])), max(max(v[4], v[5]), max(v[6], v[7])));
        if (__ballot(mx >= 1 << 20)) {
#pragma unroll
          for (int j = 0; j < 8; ++j) mk[j] = __ballot(v[j] >= 1 << 20);
#pragma unroll
          for (int j = 0; j < 8; ++j) if (__builtin_expect(mk[j] != 0, 0)) {
            if (v[j] >= 1 << 20) *(LAS unsigned*)(scr + eb + 4 * mbcnt(mk[j], 0)) = v[j] << 9 | (lane & 15) << 5 | (j & 4) << 2 | (lane & 48) >> 2 | (j & 3);
            eb += 4 * __builtin_popcountll(mk[j]); }
          if (__ballot(mx >= 1 << 23) || eb - e0 >= 256) eb = 2048; }
        asm volatile("" ::: "memory");
        if (eb <= 1024) {
          n64 = (eb + 63) >> 6;
          u32x4 d0 = *(LAS u32x4*)(scr + lane * 16);
          char* q = pk + getslot() * 1024;
          const u64 lm = ~0ull >> (64 - n64 * 4);
          asm("v_writelane_b32 %0, %1, 0" : "+v"(d0[0]) : "s"(hd | n << 22 | (eb - e0) << 14 | n64));
          if (__ballot(max(max(d0[0], d0[1]), max(d0[2], d0[3])) == MARK) & lm) n64 = 0;
          else asm volatile("s_mov_b64 exec, %3\n\tglobal_store_dwordx4 %0, %1, %2 sc0 sc1\n\ts_mov_b64 exec, -1\n\ts_nop 0" :: "v"(lane * 16), "v"(d0), "s"(q), "s"(lm) : "memory");
        }
      }
      if (n64 == 0) {
#pragma unroll
        for (int j = 0; j < 8; ++j) y[j >> 2][j & 3] = fmaxf((float)acc[j] * scale, 0.f);
        if (__builtin_expect(bad, 0)) { asm volatile(""); const float qn = __builtin_nanf(""); y[0] = f32x4{qn, qn, qn, qn}; y[1] = y[0]; }
        asm volatile("global_store_dwordx4 %0, %1, %3 sc0 sc1\n\tglobal_store_dwordx4 %0, %2, %3 offset:64 sc0 sc1\n\ts_nop 1"
                     :: "v"(vo), "v"(y[0]), "v"(y[1]), "s"(o) : "memory");
      }
      if (packed) {
        if (n64 == 0) { const unsigned q = getslot(); if (lane < 16) fstore((unsigned*)(pk + q * 1024) + lane, hd); }
      }
      if (todo) {
        const int s = __builtin_ctz(todo);
        if (HINT && sure >> 15 && (late >> s) & 1)
#pragma nounroll
          for (int i = 0; i < 64 && __builtin_amdgcn_s_getreg(7 | 3 << 11); ++i)
            if ((__builtin_amdgcn_readfirstlane(*(volatile LAS int*)(arr + s)) & 0xff) >= thr) { ready |= 1u << s; break; }
      }
    }
  }
  const int ug = (128 - HB - 1 + NG - grp % NG) % NG - 3, gi = wid * NUW + ug * 16 + ns;
  if (ug < 0 || gi >= 4 * NFL + 48 * RB || (MODE & 4)) return;
  __builtin_amdgcn_s_setprio(0);
  char* out = (char*)a.out;
  float sv = scale;
  asm volatile("" : "+s"(out)); asm volatile("" : "+v"(sv));
  const unsigned vo = (lane & 15) * (NL * 4) + (lane & 48);
  const u64 tend = bad ? 0 : t0 + DEADLINE;
  u32x4 p[4];
#define MX4(x) "v_max3_u32 v[" #x "], v[" #x "], v[" #x "+1], v[" #x "+2]\n\tv_max_u32 v[" #x "], v[" #x "], v[" #x "+3]\n\t"
#define LATE "s_add_u32 %[n], %[n], 0x1000000\n\ts_cbranch_scc0 " LB(1) "\n\ts_memrealtime vcc\n\ts_waitcnt lgkmcnt(0)\n\ts_sub_u32 vcc_lo, %[tl], vcc_lo\n\ts_subb_u32 vcc_hi, %[th], vcc_hi\n\ts_cbranch_scc0 " LB(1)
  if (gi >= 4 * NFL) {
    const int t = gi - 4 * NFL, sn = (t >> 2) & 3;
    const int ti = ((HB - RB + (t >> 4) / 3) * 4 + (sn ^ ((t >> 4) % 3 + 1))) * 16 + (t & 15);
    char* o = out + (size_t)((((rank ^ 4) & 4) | ((ti >> 4) & 3)) * MS + (ti >> 6) * 16) * (NL * 4) + (ti & 15) * 128;
#define LD2(x) "global_load_dwordx4 v[" #x ":" #x "+3], %[vo], %[o] sc0 sc1\n\tglobal_load_dwordx4 v[" #x "+4:" #x "+7], %[vo], %[o] offset:64 sc0 sc1\n\t"
#define CK2(x) "s_sleep %[sl]\n\ts_waitcnt vmcnt(2)\n\t" MX4(x) "v_max3_u32 v[" #x "], v[" #x "], v[" #x "+4], v[" #x "+5]\n\tv_max3_u32 v[" #x "], v[" #x "], v[" #x "+6], v[" #x "+7]\n\t" \
               "v_cmp_eq_u32 vcc, -1, v[" #x "]\n\ts_cbranch_vccz " LB(2) "\n\t"
    unsigned n;
    asm volatile("s_mov_b32 %[n], 0\n\ts_nop 4\n\t" LD2(0) "s_sleep %[sl]" AT(1) LD2(8) CK2(0) LD2(0) CK2(8) LATE AT(2)
                 : "={v[0:3]}"(p[0]), "={v[4:7]}"(p[1]), "={v[8:11]}"(p[2]), "={v[12:15]}"(p[3]), [n] "=&s"(n)
                 : [vo] "{v16}"(vo), [o] "s"(o), [tl] "s"((unsigned)tend), [th] "s"((unsigned)(tend >> 32)), [sl] "n"(SL) : "memory", "scc", "vcc");
    __builtin_amdgcn_endpgm();
  }
  const unsigned so = PK_OFF + ((gi & 12) * 128 + (gi >> 4) * 4 + (gi & 3)) * 1024;
  u64 lm; unsigned w[8], hd, ok, n;
#pragma unroll
  for (int j = 0; j < 8; ++j) { w[j] = 0; keep(w[j]); }
#define LD1(x) "global_load_dwordx4 v[" #x ":" #x "+3], %[vo], %[o] sc0 sc1\n\t"
#define CK1(x, n) "s_sleep %[sl]\n\ts_waitcnt vmcnt(1)\n\tv_readfirstlane_b32 %[hd], v" #x "\n\tv_max3_u32 v8, v[" #x "], v[" #x "+1], v[" #x "+2]\n\tv_max_u32 v8, v8, v[" #x "+3]\n\t" \
                  "s_lshl_b32 %[ok], %[hd], 6\n\ts_and_b32 %[ok], %[ok], 0x7c0\n\tv_cmp_gt_u32 %[lm], %[ok], %[vo]\n\tv_cmp_eq_u32 vcc, -1, v8\n\ts_and_b64 vcc, vcc, %[lm]\n\ts_cbranch_scc0 " LB(n) "\n\t"
  asm volatile("s_mov_b32 %[n], 0\n\ts_nop 4\n\t" LD1(0) "s_sleep %[sl]" AT(1) LD1(4) CK1(0, 3) LD1(0) CK1(4, 2) LATE "\n\ts_mov_b32 %[ok], -1"
               AT(2) "v_mov_b32 v8, v4\n\tv_mov_b32 v9, v5\n\tv_mov_b32 v10, v6\n\tv_mov_b32 v11, v7\n\ts_branch " LB(4)
               AT(3) "v_mov_b32 v8, v0\n\tv_mov_b32 v9, v1\n\tv_mov_b32 v10, v2\n\tv_mov_b32 v11, v3" AT(4)
               : "={v[0:3]}"(p[0]), "={v[4:7]}"(p[1]), "={v[8:11]}"(p[2]), [hd] "=&s"(hd), [ok] "=&s"(ok), [n] "=&s"(n), [lm] "=&s"(lm)
               : [vo] "{v16}"(lane * 16), [o] "s"(myws + so), [tl] "s"((unsigned)tend), [th] "s"((unsigned)(tend >> 32)), [sl] "n"(SL) : "memory", "scc", "vcc");
  hd = __builtin_amdgcn_readfirstlane(hd);
  if (ok == 0) __builtin_amdgcn_endpgm();
  __builtin_amdgcn_s_setprio(2);
  *(LAS u32x4*)(scr + lane * 16) = p[2];
  asm volatile("" ::: "memory");
  unsigned pos = 0; u64 mk[8];
  sfor<8>([&](auto jj) { constexpr int j = jj;
    mk[j] = (unsigned)__builtin_amdgcn_readlane(p[2][(2 * j + 1) % 4], (2 * j + 1) / 4) | (u64)(unsigned)__builtin_amdgcn_readlane(p[2][(2 * j + 2) % 4], (2 * j + 2) / 4) << 32;
    pos = mbcnt(mk[j], pos); });
  const int nv = (hd >> 22) & 511, nb = 68 + ((nv + 1) & ~1) * 2;
  const LAS unsigned* np = (const LAS unsigned*)(scr + nb) + (pos >> 3);
  LAS char* lp = scr + 68 + 2 * pos;
#define RD(j, k) "s_mov_b64 exec, %" #k "\n\tds_read_u16 %" #j ", %8\n\tv_add_u32 %8, 2, %8\n\t"
  u64 nn; unsigned ev;
  asm volatile("ds_read2_b32 %0, %2 offset1:1\n\tds_read_b32 %1, %3" : "=&v"(nn), "=&v"(ev) : "v"(np), "v"(scr + nb + ((nv + 7) >> 3) * 4 + lane * 4) : "memory");
  asm volatile(RD(0, 9) RD(1, 10) RD(2, 11) RD(3, 12) RD(4, 13) RD(5, 14) RD(6, 15) RD(7, 16) "s_mov_b64 exec, -1"
               : "+v"(w[0]), "+v"(w[1]), "+v"(w[2]), "+v"(w[3]), "+v"(w[4]), "+v"(w[5]), "+v"(w[6]), "+v"(w[7]), "+v"(lp)
               : "s"(mk[0]), "s"(mk[1]), "s"(mk[2]), "s"(mk[3]), "s"(mk[4]), "s"(mk[5]), "s"(mk[6]), "s"(mk[7]) : "memory");
  const int ti = (hd >> 5) & 2047, tb = ti >> 6, ts = (tb < HB ? (rank ^ 4) & 4 : rank & 4) | ((ti >> 4) & 3);
  char* o = out + (size_t)(ts * MS + tb * 16) * (NL * 4) + (ti & 15) * 128;
  asm volatile("" : "+s"(o));
  asm volatile("s_waitcnt lgkmcnt(8)" : "+v"(nn), "+v"(ev) :: "memory");
  unsigned nib = __builtin_amdgcn_alignbit((unsigned)(nn >> 32), (unsigned)nn, (pos & 7) * 4);
  float ey = (float)(ev >> 9) * sv;
  unsigned eo = (ev & 0x1e0) * (NL * 4 / 32) + (ev & 31) * 4;
  asm volatile("" : "+v"(ey), "+v"(eo));
#define HI(j, k) "s_mov_b64 exec, %" #k "\n\tv_and_b32_sdwa %" #j ", %8, %17 dst_sel:WORD_1 dst_unused:UNUSED_PRESERVE src0_sel:DWORD src1_sel:DWORD\n\tv_lshrrev_b32 %8, 4, %8\n\t"
  asm volatile("s_waitcnt lgkmcnt(0)\n\t" HI(7, 16) HI(6, 15) HI(5, 14) HI(4, 13) HI(3, 12) HI(2, 11) HI(1, 10) HI(0, 9) "s_mov_b64 exec, -1"
               : "+v"(w[0]), "+v"(w[1]), "+v"(w[2]), "+v"(w[3]), "+v"(w[4]), "+v"(w[5]), "+v"(w[6]), "+v"(w[7]), "+v"(nib)
               : "s"(mk[0]), "s"(mk[1]), "s"(mk[2]), "s"(mk[3]), "s"(mk[4]), "s"(mk[5]), "s"(mk[6]), "s"(mk[7]), "v"(15) : "memory");
  f32x4 y[2];
#pragma unroll
  for (int j = 0; j < 8; ++j) y[j >> 2][j & 3] = mul0((float)w[j], sv);
  if (__builtin_expect((int)ok < 0, 0)) { asm volatile(""); const float qn = __builtin_nanf(""); y[0] = f32x4{qn, qn, qn, qn}; y[1] = y[0]; hd = 0; }
#define UST(X) asm volatile("global_store_dwordx4 %0, %1, %3" X "\n\tglobal_store_dwordx4 %0, %2, %3 offset:64" X "\n\t" \
               "s_bfm_b64 exec, %6, 0\n\tglobal_store_dword %4, %5, %3" X "\n\ts_mov_b64 exec, -1\n\ts_nop 1" \
               :: "v"(vo), "v"(y[0]), "v"(y[1]), "s"(o), "v"(eo), "v"(ey), "s"(hd >> 16), "v"(p[0]), "v"(p[1]) : "memory")
  if ((MODE & 384) == 0) UST(" sc0 sc1"); else if ((MODE & 384) == 128) UST(""); else if ((MODE & 384) == 256) UST(" nt"); else UST(" sc1");
}

extern "C" void kernel_launch(int rank, int nranks, ncclComm_t comm, hipStream_t stream,
                              void* const* d_in, const int* in_sizes, int n_in,
                              void* d_out, int out_size, const int* in_dtypes, int out_dtype,
                              void* const* d_out_all, void* const* d_ws_all, size_t ws_bytes) {
  if (nranks != NR || (unsigned)rank >= NR || ws_bytes < WS_NEED) { fprintf(stderr, "unexpected configuration\n"); return; }
  constexpr int HB = 22, WN = 6, WI = 9, WX = 9, NG = 12;
  constexpr int GRID = 3 * WI + 4 * WX + NG * 16;
  static_assert(GRID <= 256, "one workgroup per CU, all resident");
  auto kfn = ag_gemm<HB, WN, WI, WX, NG, 18, 7, 1, 10, 3, 1, 6, 0, 0x4040>;
  static bool init = false;
  if (!init) {
    HIP_CHECK(hipFuncSetAttribute((const void*)kfn, hipFuncAttributeMaxDynamicSharedMemorySize, LDS_BYTES));
    init = true;
  }
  Args a = {};
  a.x = (const char*)d_in[0]; a.w = (const char*)d_in[1];
  a.sx = (const float*)d_in[2]; a.sw = (const float*)d_in[3];
  a.ws = (char*)d_ws_all[rank]; a.out = (float*)d_out_all[rank]; a.rank = rank;
  for (int d = 0; d < NR; ++d) {
    char* w = (char*)d_ws_all[rank ^ d];
    a.p[d] = Peer{w, (float*)d_out_all[rank ^ d], d < 4 ? a.x : a.w + ((rank ^ d) & 3) * 128,
                  w + (d < 4 ? XG_OFF + rank * SHARD : WQ_OFF + (rank & 3) * (SHARD / 4))};
  }
  hipLaunchKernelGGL(kfn, dim3(GRID), dim3(64 * CW), LDS_BYTES, stream, a);
  HIP_CHECK(hipGetLastError());
}
```

```cpp
#include <hip/hip_runtime.h>
#include <rccl/rccl.h>
#include <cstddef>
#include <cstdio>
#include <utility>

#define HIP_CHECK(expr) do { hipError_t _e = (expr); if (_e != hipSuccess) { \
    fprintf(stderr, "HIP error %s at %s:%d\n", hipGetErrorString(_e), __FILE__, __LINE__); \
    return; } } while (0)

typedef unsigned u32x4 __attribute__((ext_vector_type(4)));
typedef int i32x4 __attribute__((ext_vector_type(4)));
typedef int i32x2 __attribute__((ext_vector_type(2)));
typedef float f32x4 __attribute__((ext_vector_type(4)));
typedef unsigned long long u64;
typedef u64 u64x4 __attribute__((ext_vector_type(4)));
#define LAS __attribute__((address_space(3)))
#define GAS __attribute__((address_space(1)))

constexpr int NR = 8;
constexpr int MS = 512;
constexpr int KD = 4096;
constexpr int NL = 512;
constexpr size_t SHARD = (size_t)MS * KD;
constexpr size_t CNT_OFF = 0;
constexpr size_t TMO_OFF = 16 << 10;
constexpr size_t TK_OFF = 20 << 10;
constexpr size_t DUM_OFF = 512 << 10;
constexpr size_t XF_OFF = 128 << 10;
constexpr size_t WF_OFF = 144 << 10;
constexpr size_t CF_OFF = 152 << 10;
constexpr size_t MF_OFF = 256 << 10;
constexpr size_t PK_OFF = 2u << 20;
constexpr size_t PL_OFF = 4u << 20;
constexpr size_t XG_OFF = 8u << 20;
constexpr int ALT = 25;
constexpr size_t WS_NEED = XG_OFF + NR * SHARD + ((size_t)1 << ALT);
constexpr u64 DEADLINE = 20000000ull;
constexpr int CW = 16;
constexpr int SCR = 4096;
constexpr int ACC_OFF = CW * SCR + 256;
constexpr int LDS_BYTES = ACC_OFF + 16 * 4096;
constexpr unsigned MARK = 0xffffffffu;

struct Peer { char* ws; float* out; const char* src; char* dst; };
struct Args { const char* x; const char* w; char* ws; float* out; const float* sx; const float* sw; unsigned rank, pad[3]; Peer p[NR]; };
static_assert(offsetof(Args, rank) == 0x30 && offsetof(Args, p) == 0x40 && sizeof(Peer) == 32, "");

template <typename F, int... I> __device__ __forceinline__ void sfor_impl(F&& f, std::integer_sequence<int, I...>) { (f(std::integral_constant<int, I>{}), ...); }
template <int N, typename F> __device__ __forceinline__ void sfor(F&& f) { sfor_impl(f, std::make_integer_sequence<int, N>{}); }
__device__ __forceinline__ void waitvm0() { asm volatile("s_waitcnt vmcnt(0)" ::: "memory"); }
template <int OFF> __device__ __forceinline__ void gstore_wt(char* q, const u32x4& v) {
  asm volatile("global_store_dwordx4 %0, %1, off offset:%2 sc0 sc1\n\ts_nop 1" ::"v"(q), "v"(v), "n"(OFF) : "memory"); }
__device__ __forceinline__ void fstore(unsigned* fp, unsigned v) {
  asm volatile("global_store_dword %0, %1, off sc0 sc1" ::"v"(fp), "v"(v) : "memory"); }
__device__ __forceinline__ unsigned fload(const unsigned* fp) { return __hip_atomic_load(fp, __ATOMIC_RELAXED, __HIP_MEMORY_SCOPE_SYSTEM); }
__device__ __forceinline__ unsigned mbcnt(u64 m, unsigned base) { return __builtin_amdgcn_mbcnt_hi((unsigned)(m >> 32), __builtin_amdgcn_mbcnt_lo((unsigned)m, base)); }
template <int I> __device__ __forceinline__ void wlane(unsigned& v, unsigned x) { asm("v_writelane_b32 %0, %1, %2" : "+v"(v) : "s"(x), "n"(I)); }
__device__ __forceinline__ float mul0(float x, float y) { float r; asm("v_mul_legacy_f32 %0, %1, %2" : "=v"(r) : "v"(x), "v"(y)); return r; }

__device__ __forceinline__ bool expired(unsigned* tmo, unsigned call, u64 t0) {
  unsigned v;
  asm volatile("global_load_dword %0, %1, %2 sc1\n\ts_waitcnt vmcnt(0)" : "=&v"(v) : "v"(0), "s"(tmo) : "memory");
  if (__builtin_amdgcn_readfirstlane(v) != call && __builtin_amdgcn_s_memrealtime() - t0 <= DEADLINE) return false;
  asm volatile("global_store_dword %0, %1, %2 sc1\n\ts_nop 1" :: "v"(0), "v"(call), "s"(tmo) : "memory");
  return true;
}

constexpr int R0 = 8;
template <int HB> __host__ __device__ __forceinline__ unsigned pos_of(unsigned d, unsigned blk) {
  const int r = d < 4 ? blk : blk - HB + R0, c = r < R0 ? 0 : r - R0 < 32 - HB ? r - R0 : 32 - HB;
  return 3 * r + 4 * c + d - 1; }
template <int HB> __host__ __device__ __forceinline__ unsigned code_of(unsigned p) {
  static_assert(R0 + 32 - HB <= HB, "");
  if (p >= 128 - HB) return p - (128 - HB);
  const unsigned q = p - 3 * R0;
  if (q < 7 * (32 - HB)) return (q % 7 + 1) << 5 | (q / 7 + (q % 7 < 3 ? R0 : HB));
  if (p >= 3 * R0) p -= 4 * (32 - HB);
  return (p % 3 + 1) << 5 | p / 3; }

template <int WN> constexpr int younger(int j, int w) {
  int n = 1 + (j == 0 ? 2 : 0);
  for (int s = j + 8 - w + 1; s < j + 8; ++s) n += 2 + (s % 4 == 0 ? 2 : 0) + (s % 4 == (3 + WN) % 4 ? 1 : 0);
  return n; }
__device__ __forceinline__ void load16(u32x4& d, unsigned vo, const char* sb) { asm volatile("global_load_dwordx4 %0, %1, %2 sc0 sc1" : "=v"(d) : "v"(vo), "s"(sb) : "memory"); }
__device__ __forceinline__ void mark(unsigned vo, char* p) { asm volatile("s_nop 4\n\tglobal_store_dwordx4 %0, %1, %2 sc0 sc1\n\ts_nop 1" :: "v"(vo), "v"(u32x4{MARK, MARK, MARK, MARK}), "s"(p) : "memory"); }
template <int K> __device__ __forceinline__ void step0(u32x4& d, unsigned vq, unsigned vo, char* q, const char* sb) {
  asm volatile("s_waitcnt vmcnt(%5)\n\tglobal_store_dwordx4 %1, %0, %3 offset:%6 sc0 sc1\n\tglobal_load_dwordx4 %0, %2, %4 sc0 sc1"
               : "+v"(d) : "v"(vq), "v"(vo), "s"(q), "s"(sb), "n"(7 + K), "n"(K % 4 * 1024) : "memory"); }
__device__ __forceinline__ void keep(unsigned& a) { asm volatile("" : "+v"(a)); }
__device__ __forceinline__ void landed(unsigned& a, unsigned& b) { asm volatile("" : "+v"(a), "+v"(b)); }
__device__ __forceinline__ void draw_poll(unsigned& tv, unsigned& pv, unsigned vz, char* tp, unsigned vp, const char* pp, u64 pm) {
  asm volatile("s_mov_b64 exec, 1\n\tglobal_atomic_add %0, %2, %3, %4 sc0\n\ts_mov_b64 exec, %7\n\tglobal_load_dword %1, %5, %6 sc0 sc1\n\ts_mov_b64 exec, -1"
               : "+v"(tv), "+v"(pv) : "v"(vz), "v"(1), "s"(tp), "v"(vp), "s"(pp), "s"(pm) : "memory"); }
__device__ __forceinline__ void poll(unsigned& v, unsigned o, const char* p) { asm volatile("global_load_dword %0, %1, %2 sc0 sc1\n\ts_waitcnt vmcnt(0)" : "+v"(v) : "v"(o), "s"(p) : "memory"); }
template <int M, int J> __device__ __forceinline__ void loaded(u32x4& d, char*& q, unsigned& ns, unsigned& rl, unsigned len, char* dum, unsigned tag) {
  unsigned h, t, v;
  asm volatile("s_waitcnt vmcnt(%1)" : "+v"(d) : "n"(M) : "memory");
  asm volatile("s_lshr_b32 %1, %10, %13\n\ts_bitcmp1_b32 %10, 7\n\ts_cbranch_scc0 .LBB7%=_0\n\t"
               "v_readfirstlane_b32 %3, %6\n\tv_max_u32 %5, %8, %9\n\tv_max3_u32 %5, %6, %7, %5\n\tv_cmp_eq_u32 vcc, -1, %5\n\t"
               "s_and_b32 %1, %3, 31\n\ts_max_u32 %1, %1, 1\n\ts_mul_i32 %1, %1, -4\n\ts_xor_b32 %3, %3, %12\n\ts_lshl_b64 vcc, vcc, %1\n\ts_cselect_b32 %3, -1, %3\n\t"
               "s_or_b32 %4, %10, %14\n\ts_cmp_lt_i32 %3, 0\n\ts_cselect_b32 %4, %4, -1\n\ts_cselect_b32 %3, -1, %10\n\ts_and_b32 %2, %2, %4\n\t"
               "s_bitcmp1_b32 %3, %13\n\ts_cselect_b32 %1, 63, %1\n\ts_cselect_b64 %0, %11, %0\n.LBB7%=_0:"
               : "+s"(q), "=&s"(ns), "+s"(rl), "=&s"(h), "=&s"(t), "=&v"(v)
               : "v"(d[0]), "v"(d[1]), "v"(d[2]), "v"(d[3]), "s"(len), "s"(dum), "s"(tag), "n"(8 * J), "n"(~(63 << 8 * J)) : "scc", "vcc"); }
template <int N, bool FLAG> __device__ __forceinline__ void await(unsigned vd, unsigned fv, char* fa) {
  asm volatile("s_waitcnt vmcnt(%0)" :: "n"(N) : "memory");
  if (FLAG) asm volatile("s_mov_b64 exec, 1\n\tglobal_store_dword %0, %1, %2 sc0 sc1\n\ts_mov_b64 exec, -1" :: "v"(vd), "v"(fv), "s"(fa) : "memory"); }
template <int OFF> __device__ __forceinline__ void step(u32x4& d, unsigned vd, unsigned vo, char* q, const char* sb, unsigned ns, unsigned nl) {
  asm volatile("s_lshr_b64 exec, -1, %5\n\tglobal_store_dwordx4 %1, %0, %3 offset:%7 sc0 sc1\n\t"
               "s_lshr_b64 exec, -1, %6\n\tglobal_load_dwordx4 %0, %2, %4 sc0 sc1\n\ts_mov_b64 exec, -1\n\ts_nop 0"
               : "+v"(d) : "v"(vd), "v"(vo), "s"(q), "s"(sb), "s"(ns), "s"(nl), "n"(OFF) : "memory", "scc"); }
struct Unit { char* q; char* fa; unsigned len; };

template <int HB, int WN, int NG, int NSL, int WI, int WX, int TH, int TY, int MODE>
__device__ __forceinline__ void pusher(unsigned dl, unsigned b, unsigned w0, unsigned nw, unsigned lane) {
  constexpr int YB = 32 - HB, FL = (3 + WN) / 4, FJ = (3 + WN) % 4;
  static_assert(WN >= 5 && WN <= 7 && NSL % 4 == 0 && 6 * WX <= 128, "");
  __builtin_amdgcn_s_waitcnt(0);
  u64x4 ha, pa; unsigned rank;
  asm volatile("s_load_dwordx8 %1, %3, %4 offset:0x40\n\ts_load_dwordx8 %0, %3, 0\n\ts_load_dword %2, %3, 0x30"
               : "=&s"(ha), "=&s"(pa), "=&s"(rank) : "s"(__builtin_amdgcn_kernarg_segment_ptr()), "s"(dl * 32));
  const bool cross = dl >= 4;
  const unsigned vx = (lane & 15) * KD + (lane >> 4) * 16;
  const unsigned vw = (lane >> 2) * NL + (lane & 3) * 16;
  const unsigned vd = lane * 16;
  u32x4 buf[2][4];
  unsigned cv, vl[16], vq[2] = {vd + w0 * 16384, vd + w0 * 16384 + 4096};
  unsigned s0 = cross ? 16 * NL : 64, v0 = cross ? vw + (w0 >> 1) * (256 * NL) + (w0 & 1) * 64 : vx + (w0 >> 2) * (16 * KD) + (w0 & 3) * 1024;
  asm volatile("" : "+s"(s0));
  auto first = [&](int i) { vl[i] = i ? vl[i - 1] + s0 : v0; keep(vl[i]); };
  sfor<8>([&](auto i) { first(i); });
  unsigned vc = b * (CW * 4);
  keep(vc);
  asm volatile("s_waitcnt lgkmcnt(0)" : "+s"(ha), "+s"(pa), "+s"(rank));
  rank &= 7;
  const char* ax = (const char*)ha[0]; char* myws = (char*)ha[2];
  const Peer pr = {(char*)pa[0], nullptr, (const char*)pa[2], (char*)pa[3]};
  char* pws = pr.ws;
  asm volatile("global_load_dword %0, %1, %2 sc1" : "=v"(cv) : "v"(vc), "s"(myws + CNT_OFF) : "memory");
  sfor<8>([&](auto i) { load16(buf[i / 4][i % 4], vl[i], pr.src); });
  sfor<8>([&](auto i) { first(i + 8); });
  asm volatile("s_waitcnt vmcnt(8)" : "+v"(cv), "+v"(vq[0]), "+v"(vq[1]) :: "memory");
  sfor<8>([&](auto ii) { constexpr int i = ii; step0<i>(buf[i / 4][i % 4], vq[i / 4] + ((cv & 1) << ALT), vl[i + 8], pr.dst, pr.src); });
  const unsigned call = __builtin_amdgcn_readfirstlane(cv) + 1, total = cross ? 128 + YB * 16 : HB * 16;
  char* dum = myws + DUM_OFF + (dl * 16 + w0) * 8192;
  sfor<4>([&](auto i) { mark(vd + i * 1024, dum); });
  auto id_of = [&](unsigned u) { return !cross ? u : u < 128 ? 512 + u : HB * 16 + u - 128; };
  auto src_of = [&](int id) { return id >= 512 ? pr.src + (size_t)(((id - 512) >> 3) * 4 + (id & 3)) * (64 * NL) + (id & 4) * 16 : ax + (size_t)(id >> 4) * (16 * KD) + (id & 15) * 256; };
  unsigned* tmo = (unsigned*)(myws + TMO_OFF);
  const unsigned ps = pos_of<HB>(dl, lane & 31), vt = XF_OFF + (ps % NG * 256 + ps / NG) * 4;
  char* dx = pws + XG_OFF + (size_t)rank * SHARD + ((~call & 1) << ALT);
  char* dw = pr.dst - 512 * 4096 + ((~call & 1) << ALT);
  const unsigned need = lane ? call : call * (cross ? TY : TH), vm = lane ? CF_OFF - 128 + (dl & 3) * 128 + lane * 4 : MF_OFF + dl * 128;
  const u64 lr = cross ? ((1ull << 2 * NG) - 1) << 32 | 1 : 1;
  unsigned vz = DUM_OFF + (dl * 16 + w0) * 8192 + 6144, vp;
  keep(vz); vp = vz;
  const unsigned all = total + (cross && !(MODE & 4) ? NSL / 4 : 0);
  const unsigned runs = total - (cross ? TY : TH) * 16, rbase = (call - 1) * (runs / 4 - nw) - 2 * nw;
  unsigned nst = 8, cur = 0, nxt = (w0 + nw) * 4;
  asm volatile("" : "+s"(nst), "+s"(cur));
  unsigned vcall = call, lim = runs, rl = 0x3f3f3fbf, nr = 0;
  char* rq = dum;
  asm volatile("" : "+s"(rl), "+s"(nr));
  const long pld = (myws + PL_OFF + (dl & 3) * (512 * 1024)) - (pws + PK_OFF + (rank & 3) * (512 * 1024));
  keep(vcall);
  auto xw_unit = [&](int i, bool f) { return Unit{(i >= 512 ? dw : dx) + (size_t)i * 4096,
    !f ? dum + 4096 : pws + (i >= 512 ? WF_OFF + ((rank & 3) * 128 - 515 + i) * 16 : __builtin_amdgcn_readlane(vt, i >> 4) + (i & 12) * 64), 0}; };
  const Unit empty = {dum, dum + 4096, 0x3f3f3f3f};
  Unit pu[5];
  sfor<4>([&](auto i) { pu[i] = xw_unit(w0 * 4 + i + cross * 512, i == 3); });
  asm volatile("" : "+s"(pu[2].len), "+s"(pu[3].len));
  const unsigned tbase = (call - 1) * (all - runs + 2 * nw) - runs;
  unsigned tv[2] = {0, 0}, tk[2] = {0, 0}, idle = 0, n = 4;
  unsigned pv[2] = {0, 0};
  bool drew[2] = {false, false};
  const u64 t0 = __builtin_amdgcn_s_memrealtime();
  for (unsigned it = 0;; ++it) {
    bool done = false;
    sfor<2>([&](auto ic) { constexpr int c = ic;
      if (!done) {
        landed(tv[c], pv[c]);
        if (drew[c]) tk[c] = __builtin_amdgcn_readfirstlane(tv[c]) - tbase;
        const bool dr = c == 0 && n < nst && !(n & 2);
        if (dr) cur = nxt;
        else if (c == 0 && n < nst) { nxt = (__builtin_amdgcn_readfirstlane(tv[c]) - rbase) * 4; nst += nxt < runs ? 4 : 0; }
        if (n < nst) tk[c] = cur + (n & 3);
        ++n;
        if (lim < total) {
          if (c == 1 && n == 18) vp = vm;
          u64 no = __ballot((int)(pv[c] - need) < 0) & lr;
          if (tk[c] >= lim) for (unsigned i = 1; no; ++i) {
            poll(pv[c], vm, myws);
            no = __ballot((int)(pv[c] - need) < 0) & lr;
            if ((i & 63) == 0 && expired(tmo, call, t0)) no = 0; }
          if (!no) { lim = total; vp = vz; } }
        const char* sb; unsigned vo = vd; int st = 1024;
        drew[c] = n + 1 >= nst;
        if (rl != 0x3f3f3fbf) {
          pu[4] = Unit{rq, dum + 4096, rl}; sb = rq + pld; idle = idle < FL ? idle + 1 : FL; drew[c] = false;
          __builtin_amdgcn_s_sleep(MODE >> 12 & 15);
          if ((++nr & 63) == 0 && expired(tmo, call, t0)) { pu[4] = empty; tk[c] = all; }
        } else if (tk[c] < total) { const int i = id_of(tk[c]); pu[4] = xw_unit(i, tk[c] < runs && (i & 3) == 3); sb = src_of(i); vo = i >= 512 ? vw : vx; st = i >= 512 ? 16 * NL : 64; }
        else if (tk[c] < all) {
          idle = idle < FL ? idle + 1 : FL;
          pu[4] = Unit{pws + PK_OFF + ((rank & 3) * 128 + tk[c] - total) * 4096, dum + 4096, rl & 0x80};
          sb = pu[4].q + pld;
        } else { pu[4] = empty; sb = ax; ++idle; drew[c] = false; }
        rl |= 0x3f3f3f3f; rq = pu[2].q;
        draw_poll(tv[c], pv[c], vd, drew[c] || dr ? myws + TK_OFF + dl * 128 + (dr ? 24 * 128 : 0) : dum + 5120, vp, myws, lr);
        sfor<4>([&](auto jj) { constexpr int j = jj;
          char* q = pu[2].q; unsigned ns;
          loaded<younger<WN>(j, 8) - (j == 3 ? 3 : 1), j>(buf[c][j], q, ns, rl, pu[2].len, dum, call << 31);
          await<younger<WN>(j, WN), j == FJ>(vd, vcall, pu[2 - FL].fa);
          step<j * 1024>(buf[c][j], vd, vo, q, sb + j * st, ns, pu[4].len >> 8 * j);
        });
        done = idle >= 2 + FL && tk[0] >= all && tk[1] >= all && rl == 0x3f3f3fbf;
        sfor<4>([&](auto ii) { constexpr int i = ii; pu[i] = pu[i + 1]; });
      } });
    if (done) break;
  }
  asm volatile("s_waitcnt vmcnt(0)" :: "v"(buf[0][0]), "v"(buf[0][1]), "v"(buf[0][2]), "v"(buf[0][3]), "v"(buf[1][0]), "v"(buf[1][1]), "v"(buf[1][2]), "v"(buf[1][3]),
               "v"(tv[0]), "v"(tv[1]), "v"(pv[0]), "v"(pv[1]) : "memory");
  if (lane == 0) __hip_atomic_store((GAS unsigned*)(myws + CNT_OFF) + b * CW, call, __ATOMIC_RELAXED, __HIP_MEMORY_SCOPE_AGENT);
  {
    unsigned k;
    asm volatile("s_mov_b64 exec, 1\n\tglobal_atomic_add %0, %1, %2, %3 sc0\n\ts_mov_b64 exec, -1\n\ts_waitcnt vmcnt(0)"
                 : "=&v"(k) : "v"(vd), "v"(1), "s"(myws + TK_OFF + (16 + dl) * 128) : "memory");
    const unsigned p = pos_of<HB>(dl, (cross ? 31 : HB - 1) - (lane >> 2));
    if (__builtin_amdgcn_readfirstlane(k) + 1 == call * nw && lane < (cross ? TY : TH) * 4) fstore((unsigned*)(pws + XF_OFF) + p % NG * 256 + p / NG + (lane & 3) * 64, call); }
}

template <int HB, int WN, int WI, int WX, int NG, int DH, int DY, int RB, int TH, int TY, int HINT, int SL, int FB, int MODE>
__global__ void __launch_bounds__(64 * CW, 4) ag_gemm(Args a) {
  extern __shared__ __attribute__((aligned(16))) char lds[];
  constexpr int NP = 3 * WI + 4 * WX, NPOS = 160 - HB, NUW = (NG - 3) * 16;
  constexpr int NSL = (3 * DH + HB + 3 * DY) * 4, NDH = HB - RB - DH, NFL = NSL + (NDH + 32 - HB - DY) * 12;
  static_assert(NFL <= 510 && 4 * NFL + 48 * RB <= CW * NUW && WI <= 16 && WX <= 16, "");
  static_assert(DH <= HB - RB && NG * 15 >= NPOS && NG <= 16 && NG % 2 == 0 && TY <= 32 - HB - DY && NP <= 85, "");
  const int lane = threadIdx.x & 63, wid = __builtin_amdgcn_readfirstlane(threadIdx.x >> 6);
  const int b = blockIdx.x;
  if (__builtin_expect(b < NP && wid == 0, 1)) {
    constexpr unsigned WM = WI < WX ? WI : WX, WL = WI > WX ? 3 : 4;
    unsigned w0 = b * 37u >> 8, d = 1 + b - w0 * 7;
    if (WI != WX && b >= 7 * WM) { d = (WI > WX ? 1 : 4) + (b - 7 * WM) % WL; w0 = WM + (b - 7 * WM) / WL; }
    pusher<HB, WN, NG, NSL, WI, WX, TH, TY, MODE>(d & 7, b, w0, d >= 4 ? WX : WI, lane);
    return;
  }
  if (b < NP || (MODE & 8)) return;
  const int cb = b - NP, grp = (cb >> 5) * 2 + ((cb >> 2) & 1), ns = (cb & 3) * 4 + ((cb >> 3) & 3);
  const bool ownw = (ns & 1) == 0;
  const int d = ownw ? 0 : 4 + (ns >> 2);
  const char* ax = a.x; const char* aw = a.w; char* myws = a.ws; char* dws = a.p[d].ws; float* dout = a.p[d].out;
  const unsigned rank = a.rank & 7;
  asm volatile("" :: "s"(ax), "s"(aw), "s"(myws), "s"(dws), "s"(dout), "s"(rank));
  unsigned* cnt = (unsigned*)(myws + CNT_OFF) + b * CW + wid;
  typedef const __attribute__((address_space(4))) float* cfp;
  const float sc = *(cfp)a.sx * *(cfp)a.sw, scale = fabsf(sc);
  const bool neg = sc < 0.f;
  unsigned* tmo = (unsigned*)(myws + TMO_OFF);
  LAS char* scr = (LAS char*)lds + wid * SCR;

  asm volatile("" ::: "memory");
  unsigned call = __hip_atomic_load(cnt, __ATOMIC_RELAXED, __HIP_MEMORY_SCOPE_AGENT);
  const unsigned dest = rank ^ d;
  const int col0 = ownw ? (ns >> 1) * 64 : (rank & 3) * 128 + (ns & 2) * 32;
  LAS int* arr = (LAS int*)((LAS char*)lds + CW * SCR);
  if (threadIdx.x < 32) arr[threadIdx.x] = 0;
#pragma unroll
  for (int i = 0; i < 4; ++i) *(LAS u32x4*)((LAS char*)lds + ACC_OFF + i * 16384 + threadIdx.x * 16) = u32x4{0, 0, 0, 0};
  bool bad = false;
  i32x4 wf[4][4];
  if (ownw) {
    const __amdgpu_buffer_rsrc_t rw = __builtin_amdgcn_make_buffer_rsrc((void*)(aw + (size_t)wid * (256 * NL) + col0), 0, 256 * NL, 0x00020000);
#pragma unroll
    for (int i = 0; i < 16; ++i) wf[i >> 2][i & 3] = __builtin_bit_cast(i32x4, __builtin_amdgcn_raw_buffer_load_b128(rw, (lane >> 2) * NL + (lane & 3) * 16, i * 16 * NL, 0));
  }
  call = __builtin_amdgcn_readfirstlane(call) + 1;
  if (lane == 0) __hip_atomic_store(cnt, call, __ATOMIC_RELAXED, __HIP_MEMORY_SCOPE_AGENT);
  const unsigned pz = (lane & 15) * NG + grp, code = code_of<HB>(pz) ^ rank << 5;
  const unsigned late = (unsigned)__ballot(pz < 128 - HB && (code & 31) >= ((code & 31) < HB ? HB - TH : 32 - TY)) & 0xffff;
  const unsigned xo = XG_OFF + ((~call & 1) << ALT) + wid * 4096, mkm = late & 0x101 << (ns >> 1);
  asm volatile("s_waitcnt lgkmcnt(0)" ::: "memory");
  __builtin_amdgcn_s_barrier();
  asm volatile("" ::: "memory");
  if (!ownw) {
    for (int i = wid * (MODE >> 24 & 15); i > 0; --i) __builtin_amdgcn_s_sleep(2);
    const u32x4 mk = {MARK, MARK, MARK, MARK};
    for (unsigned u = (grp * 2 + ((ns >> 1) & 1)) * CW + wid; u < NFL + 24 * RB; u += 2 * NG * CW) {
      const unsigned j = u - NFL;
      gstore_wt<0>(u < NFL ? myws + PK_OFF + ((dest & 3) * 512 + u) * 1024 + lane * 16
                           : (char*)(a.out + ((dest ^ (j / (RB * 8 + !RB) + 1)) * MS + (HB - RB) * 16 + j % (RB * 8 + !RB) / 2 * 4 + (lane >> 4)) * NL + (dest & 3) * 128 + (j & 1) * 64 + (lane & 15) * 4), mk); }
    for (unsigned u = (grp * 2 + ((ns >> 1) & 1)) * CW + wid; u < NSL; u += 2 * NG * CW) gstore_wt<0>(myws + PL_OFF + ((dest & 3) * 512 + u) * 1024 + lane * 16, mk);
    for (unsigned m = mkm; m; m &= m - 1) {
      char* q = myws + xo + __builtin_amdgcn_readlane(code, __builtin_ctz(m)) * (16 * KD) + lane * 16;
      gstore_wt<0>(q, mk); gstore_wt<1024>(q, mk); gstore_wt<2048>(q, mk); gstore_wt<3072>(q, mk); }
    waitvm0();
    int nw = 0;
    if (lane == 0) nw = __hip_atomic_fetch_add(arr + 18, 1, __ATOMIC_RELAXED, __HIP_MEMORY_SCOPE_WORKGROUP);
    if (__builtin_amdgcn_readfirstlane(nw) == CW - 1) {
      if (lane < 2) fstore((unsigned*)((lane ? dws : myws) + CF_OFF) + (lane ? 128 + (rank & 3) * 32 : (dest & 3) * 32) + grp * 2 + ((ns >> 1) & 1), call);
      for (unsigned m = mkm; m; m &= m - 1) {
        const unsigned l = ((__builtin_amdgcn_readlane(code, __builtin_ctz(m)) >> 5) ^ rank) & 7;
        if (lane == 0) __hip_atomic_fetch_add((unsigned*)(a.p[l].ws + MF_OFF + l * 128), 1, __ATOMIC_RELAXED, __HIP_MEMORY_SCOPE_SYSTEM); }
      unsigned k = 0;
      if (lane == 0) k = __hip_atomic_fetch_add((unsigned*)(myws + XF_OFF) + grp * 256 + 32, 1, __ATOMIC_RELAXED, __HIP_MEMORY_SCOPE_SYSTEM);
      if (__builtin_amdgcn_readfirstlane(k) + 1 == call * 8 && lane < 4) fstore((unsigned*)(myws + XF_OFF) + grp * 256 + lane * 64 + 15, call); }
    const unsigned* fw = (const unsigned*)(myws + WF_OFF) + (dest & 3) * 512 + wid * 32 + (ns & 2) * 8;
    const u64 t0 = __builtin_amdgcn_s_memrealtime();
    for (unsigned it = 1; __ballot(fload(fw) < call); ++it) {
      if ((it & 63) == 0 && expired(tmo, call, t0)) { bad = true; break; }
      __builtin_amdgcn_s_sleep(4); }
    asm volatile("" ::: "memory");
    const __amdgpu_buffer_rsrc_t rw = __builtin_amdgcn_make_buffer_rsrc((void*)(myws + XG_OFF + ((~call & 1) << ALT) + (size_t)dest * SHARD + wid * 32768 + (ns & 2) * 8192), 0, 16384, 0x00020000);
#pragma unroll
    for (int i = 0; i < 16; ++i) wf[i >> 2][i & 3] = __builtin_bit_cast(i32x4, __builtin_amdgcn_raw_buffer_load_b128(rw, lane * 16 + (i & 3) * 1024, (i >> 2) * 4096, 17));
  }
  {
    LAS char* wr = scr + ((lane >> 1) & 1) * 2048 + (lane >> 2) * 32;
    LAS char* rb = scr + (16 * (lane >> 4) + ((lane & 15) >> 1)) * 32 + 8 * (lane & 1);
#pragma unroll
    for (int i = 0; i < 4; ++i) {
#pragma unroll
      for (int j = 0; j < 4; ++j) *(LAS i32x4*)(wr + j * 512 + (((lane ^ j) & 1) * 16)) = wf[i][j];
      asm volatile("" ::: "memory");
#pragma unroll
      for (int t = 0; t < 4; ++t) {
        LAS char* p = rb + (t >> 1) * 2048 + 16 * (((lane >> 4) ^ t) & 1);
        const i32x2 u0 = __builtin_amdgcn_ds_read_tr8_b64_v2i32((LAS i32x2*)p), u1 = __builtin_amdgcn_ds_read_tr8_b64_v2i32((LAS i32x2*)(p + 256));
        wf[i][t] = i32x4{u0[0], u0[1], u1[0], u1[1]}; }
      asm volatile("" ::: "memory");
    }
  }
  unsigned todo = (unsigned)__ballot(pz < (ownw ? NPOS : 128) && (ownw || (code >> 5) != dest)) & 0xffff;
  const unsigned mine = (unsigned)__ballot(pz >= 128 - HB) & 0x7fff;
  const unsigned pofs = ((late & todo) << 16 >> (lane & 31)) & 1 ? xo + code * (16 * KD) : XF_OFF + ((grp * 16 + (wid & 12)) * 16 + (lane & 15)) * 4;
  unsigned sure = bad || (MODE & 1) ? 0xffff : mine, ready = sure;
  const unsigned src = code >> 5, blk = code & 31;
  const bool home = blk < HB && src != rank;
  const unsigned dirm = __ballot(home ? blk >= DH : blk >= HB + DY), pkm = ownw ? 0 : ~(unsigned)__ballot(home && blk >= HB - RB);
  const unsigned info = (NSL + ((home ? blk - DH : blk - (HB + DY) + NDH) * 3 + ((src ^ (home ? rank : dest)) & 3) - 1) * 4 + (ns & 2)) << 16
                        | (((blk * 4 + (src & 3)) * 4 + (rank & 3)) * 4 + (ns & 2)) << 5;
  unsigned fast = !ownw || FB / 1000 ? __ballot(home && blk >= HB - FB % 10) : 0;
  char* oc = (char*)(dout + col0);
  const unsigned thr = wid > 10 ? wid - 9 : 1, c16 = code * (16 * KD);
  typedef int i32x16 __attribute__((ext_vector_type(16)));
  i32x16 wr[4];
#pragma unroll
  for (int i = 0; i < 4; ++i) wr[i] = __builtin_shufflevector(__builtin_shufflevector(wf[i][0], wf[i][1], 0, 1, 2, 3, 4, 5, 6, 7), __builtin_shufflevector(wf[i][2], wf[i][3], 0, 1, 2, 3, 4, 5, 6, 7),
                                                              0, 1, 2, 3, 4, 5, 6, 7, 8, 9, 10, 11, 12, 13, 14, 15);
  const int ntile = __builtin_popcount(todo);
  const u64 t0 = __builtin_amdgcn_s_memrealtime();
  if (!ownw && wid == CW - 1) { const unsigned* mf = (const unsigned*)(myws + CF_OFF) + (lane < 32 ? (ns & 12) * 8 : 128 + (dest & 3) * 32) + lane % (2 * NG);
    for (unsigned it = 1; __ballot(fload(mf) < call); ++it) {
      if ((it & 63) == 0 && expired(tmo, call, t0)) { bad = true; sure = ready = 0xffff; break; }
      __builtin_amdgcn_s_sleep(4); } }
  __builtin_amdgcn_s_waitcnt(0);
  for (;;) {
#define X4(x, i) "v[" #x "+4*" #i ":" #x "+4*" #i "+3]"
#define L4(x, base, st) "global_load_dwordx4 " X4(x, 0) ", v2, " base " sc0 sc1\n\tglobal_load_dwordx4 " X4(x, 1) ", v2, " base " offset:" st " sc0 sc1\n\t" \
                        "global_load_dwordx4 " X4(x, 2) ", v2, " base " offset:2*" st " sc0 sc1\n\tglobal_load_dwordx4 " X4(x, 3) ", v2, " base " offset:3*" st " sc0 sc1\n\t"
#define LB(n) ".LBB9%=_" #n
#define AT(n) "\n" LB(n) ":\n\t"
#define POLL "global_load_dword v1, %[pofs], %[ws] sc0 sc1\n\t"
#define LOAD(x, S, n) "s_ff1_i32_b32 " S ", %[m]\n\tv_readlane_b32 %[t], %[c16], " S "\n\ts_bitset0_b32 %[todo], " S "\n\ts_bitcmp1_b32 %[ml], " S "\n\ts_cbranch_scc1 " LB(n##1) "\n\t" \
                      "v_add_u32 v2, %[t], %[vg]\n\t" L4(x, "%[ws]", "1024") AT(n##2) POLL
#define OWN(x, n) AT(n##1) "v_add_u32 v2, %[t], %[vx]\n\t" L4(x, "%[xm]", "64") "s_branch " LB(n##2)
#define SEEN "s_waitcnt vmcnt(0)\n\tv_cmp_le_u32 vcc, %[call], v1\n\ts_and_b32 %[t], vcc_lo, 0xffff\n\tv_cmp_ne_u32 vcc, -1, v1\n\ts_or_b32 %[sure], %[sure], %[t]\n\t" \
             "s_and_b32 %[t], vcc_lo, %[ml]\n\ts_lshr_b32 %[t], %[t], 16\n\ts_bitcmp1_b32 %[sure], 15\n\ts_cselect_b32 %[t], %[t], 0\n\ts_or_b32 %[ready], %[sure], %[t]\n\t"
#define MX(x, i) "v_max3_u32 v2, v2, v[" #x "+" #i "], v[" #x "+" #i "+1]\n\t"
#define MM(x, a, t) "v_mfma_i32_16x16x64_i8 " X4(4, a) ", " X4(64, t) ", " X4(x, 0) ", 0\n\tv_mfma_i32_16x16x64_i8 " X4(4, a) ", " X4(80, t) ", " X4(x, 1) ", " X4(4, a) "\n\t" \
                    "v_mfma_i32_16x16x64_i8 " X4(4, a) ", " X4(96, t) ", " X4(x, 2) ", " X4(4, a) "\n\tv_mfma_i32_16x16x64_i8 " X4(4, a) ", " X4(112, t) ", " X4(x, 3) ", " X4(4, a) "\n\t"
#define AD(a, t) "ds_add_u32 v2, v[4+4*" #a "] offset:1024*" #t "\n\tds_add_u32 v2, v[5+4*" #a "] offset:1024*" #t "+256\n\t" \
                 "ds_add_u32 v2, v[6+4*" #a "] offset:1024*" #t "+512\n\tds_add_u32 v2, v[7+4*" #a "] offset:1024*" #t "+768\n\t"
#define COMP(x, S, y, T, n) AT(n##0) "s_and_b32 %[m], %[ready], %[todo]\n\ts_cbranch_scc0 " LB(n##3) "\n\t" LOAD(y, T, n) "s_waitcnt vmcnt(6)\n\ts_branch " LB(n##4) \
      AT(n##3) "s_waitcnt vmcnt(1)" AT(n##4) "s_bitcmp1_b32 %[sure], " S "\n\ts_cbranch_scc1 " LB(n##5) "\n\t" \
      CHK(x) "s_cbranch_vccnz " LB(n##8) \
      AT(n##5) MM(x, 0, 0) MM(x, 1, 1) "v_lshl_add_u32 v2, " S ", 12, %[acc]\n\ts_lshl2_add_u32 %[t], " S ", %[arr]\n\tv_mov_b32 v0, %[t]\n\tv_mov_b32 v3, %[inc]\n\t" \
      AD(0, 0) MM(x, 0, 2) AD(1, 1) MM(x, 1, 3) AD(0, 2) "s_nop 3\n\t" AD(1, 3) \
      "s_mov_b64 exec, 1\n\tds_add_rtn_u32 v3, v0, v3\n\ts_mov_b64 exec, -1\n\ts_cmp_lg_u32 %[m], 0\n\ts_cbranch_scc1 " LB(n##6) "\n\t" SEEN \
      AT(n##6) "s_waitcnt lgkmcnt(0)\n\tv_readfirstlane_b32 %[t], v3\n\ts_and_b32 %[t], %[t], 0xff\n\ts_cmp_lg_u32 %[t], 15\n\ts_cbranch_scc1 " LB(n##7) "\n\t" \
      "s_lshl_b32 %[e], " S ", 1\n\ts_and_b32 %[t], %[ready], %[todo]\n\ts_or_b32 %[t], %[t], %[m]\n\ts_cselect_b32 vcc_lo, 3, 2\n\tv_mov_b32 v2, vcc_lo\n\tv_lshlrev_b32 v2, %[e], v2\n\t" \
      "v_mov_b32 v0, %[arr]\n\tv_mov_b32 v3, 1\n\ts_mov_b64 exec, 1\n\tds_add_u64 v0, v[2:3] offset:64\n\ts_mov_b64 exec, -1\n\ts_cbranch_scc0 " LB(99) \
      AT(n##7) "s_cmp_lg_u32 %[m], 0\n\t"
#define FP(r) "global_load_dword v" #r ", %[pofs], %[ws] sc0 sc1\n\ts_sleep %[fs]\n\t"
#define FC(r) "s_waitcnt vmcnt(3)\n\tv_cmp_ne_u32 vcc, -1, v" #r "\n\ts_cbranch_vccnz " LB(98) "\n\t"
#define CHK(x) "v_max_u32 v2, v[" #x "], v[" #x "+1]\n\t" MX(x, 2) MX(x, 4) MX(x, 6) MX(x, 8) MX(x, 10) MX(x, 12) MX(x, 14) "v_cmp_eq_u32 vcc, -1, v2\n\t"
#define FAIL(S, n, k) AT(n##8) "s_bitset1_b32 %[todo], " S "\n\ts_bitset0_b32 %[ready], " S "\n\ts_cmp_lg_u32 %[m], 0\n\ts_cbranch_scc1 " LB(k##0) "\n\ts_branch " LB(92)
    unsigned m, sa, sb, t, n; int e;
    asm volatile("s_mov_b32 %[n], 64\n\ts_and_b32 %[m], %[ready], %[todo]\n\ts_cbranch_scc1 " LB(10)
                 AT(91) "s_cmp_eq_u32 %[todo], 0\n\ts_cbranch_scc1 " LB(94) "\n\t" POLL
                 AT(92) "s_mov_b32 %[e], -1\n\t" SEEN "s_sub_u32 %[n], %[n], 1\n\ts_cbranch_scc1 " LB(99)
                 AT(93) "s_and_b32 %[m], %[ready], %[todo]\n\ts_cbranch_scc0 " LB(94)
                 AT(10) "s_setprio 3\n\t" LOAD(12, "%[sa]", 1)
                 COMP(12, "%[sa]", 28, "%[sb]", 2) "s_cbranch_scc0 " LB(93) COMP(28, "%[sb]", 12, "%[sa]", 3) "s_cbranch_scc1 " LB(20) "\n\ts_branch " LB(93)
                 OWN(12, 1) OWN(28, 2) OWN(12, 3) FAIL("%[sa]", 2, 3) FAIL("%[sb]", 3, 2)
                 AT(95) "v_readfirstlane_b32 %[t], v3\n\ts_mov_b32 %[e], -2\n\ts_lshr_b32 %[m], %[np], 1\n\ts_xor_b32 %[t], %[t], %[m]\n\ts_or_b32 %[t], %[t], %[todo]\n\ts_cbranch_scc0 " LB(99)
                        "\n\tv_readfirstlane_b32 %[t], v3\n\ts_setprio 0\n\ts_sub_u32 %[m], %[m], %[t]\n\ts_lshr_b32 %[t], %[sure], 15\n\ts_cmp_eq_u32 %[m], %[t]\n\ts_cselect_b32 %[t], %[todo], 0\n\t"
                        "s_and_b32 %[t], %[t], %[fast]\n\ts_cbranch_scc1 " LB(96) "\n\ts_sleep %[slp]\n\ts_branch " LB(91)
                 AT(96) "s_ff1_i32_b32 %[sa], %[todo]\n\ts_mov_b32 %[fast], 0\n\ts_mov_b32 %[sb], %[sa]\n\ts_lshl_b64 exec, 0x10000, %[sa]\n\ts_mov_b32 %[m], 31\n\t"
                        "v_readlane_b32 %[t], %[c16], %[sa]\n\t" FP(28) FP(29) FP(30)
                 AT(97) FP(31) FC(28) FP(28) FC(29) FP(29) FC(30) FP(30) FC(31) "s_sub_u32 %[m], %[m], 1\n\ts_cbranch_scc0 " LB(97) "\n\ts_mov_b64 exec, -1\n\ts_branch " LB(91)
                 AT(98) "s_mov_b64 exec, -1\n\ts_setprio 3\n\ts_mov_b32 %[todo], 0\n\ts_mov_b32 %[m], 0\n\ts_mov_b32 %[e], 3"
                 AT(89) "v_add_u32 v2, %[t], %[vg]\n\t" L4(12, "%[ws]", "1024") "s_sleep %[fp]\n\t" L4(28, "%[ws]", "1024")
                        "s_waitcnt vmcnt(4)\n\t" CHK(12) "s_cbranch_vccz " LB(25) "\n\ts_waitcnt vmcnt(0)\n\t" CHK(28) "s_cbranch_vccz " LB(35)
                        "\n\ts_sub_u32 %[e], %[e], 1\n\ts_cbranch_scc0 " LB(89) "\n\ts_bitset1_b32 %[todo], %[sa]\n\ts_branch " LB(91)
                 AT(94) "v_mov_b32 v0, %[arr]\n\tds_read_b64 v[2:3], v0 offset:64\n\ts_and_b32 %[sa], %[np], 1\n\ts_lshl_b32 %[sa], 0x55555555, %[sa]\n\ts_waitcnt lgkmcnt(0)\n\t"
                        "v_readfirstlane_b32 %[t], v2\n\ts_cmp_eq_u32 %[t], 0\n\ts_cbranch_scc1 " LB(95) "\n\ts_and_b32 %[m], %[t], %[sa]\n\ts_cselect_b32 %[t], %[m], %[t]\n\t"
                        "s_ff1_i32_b32 %[e], %[t]\n\ts_mov_b32 %[t], -1\n\ts_bitset0_b32 %[t], %[e]\n\tv_mov_b32 v2, %[t]\n\t"
                        "s_mov_b64 exec, 1\n\tds_and_rtn_b32 v2, v0, v2 offset:64\n\ts_mov_b64 exec, -1\n\ts_waitcnt lgkmcnt(0)\n\t"
                        "v_readfirstlane_b32 %[t], v2\n\ts_bitcmp1_b32 %[t], %[e]\n\ts_cbranch_scc0 " LB(91)
                 AT(99)
                 : [todo] "+s"(todo), [sure] "+s"(sure), [ready] "+s"(ready), [fast] "+s"(fast), [e] "=&s"(e), [m] "=&s"(m), [sa] "=&s"(sa), [sb] "=&s"(sb), [t] "=&s"(t), [n] "=&s"(n)
                 : [ml] "s"(mine | late << 16), [call] "s"(call), [ws] "s"(myws), [xm] "s"(ax + wid * 256 - rank * SHARD), [inc] "s"(bad ? 0x101 : 1), [arr] "s"(arr), [np] "s"(ntile * 2 + (wid & 1)), [slp] "n"(MODE & 64 ? 16 : 1), [fs] "n"(FB / 10 % 10), [fp] "n"(FB / 100 % 10),
                   [c16] "v"(c16), [pofs] "v"(pofs), [vg] "v"(lane * 16 + xo), [vx] "v"((lane & 15) * KD + (lane >> 4) * 16), [acc] "v"((LAS char*)lds + ACC_OFF + lane * 4),
                   "{v[64:79]}"(wr[0]), "{v[80:95]}"(wr[1]), "{v[96:111]}"(wr[2]), "{v[112:127]}"(wr[3])
                 : "memory", "scc", "vcc", "v0", "v1", "v2", "v3", "v4", "v5", "v6", "v7", "v8", "v9", "v10", "v11", "v12", "v13", "v14", "v15", "v16", "v17", "v18", "v19", "v20", "v21",
                   "v22", "v23", "v24", "v25", "v26", "v27", "v28", "v29", "v30", "v31", "v32", "v33", "v34", "v35", "v36", "v37", "v38", "v39", "v40", "v41", "v42", "v43");
    if (__builtin_expect(e < 0, 0)) {
      if (e == -2) break;
      if (expired(tmo, call, t0)) { bad = true; sure = ready = 0xffff; }
      continue; }
    {
      const int s = e >> 1, h = e & 1;
      const LAS int* sa = (LAS int*)((LAS char*)lds + ACC_OFF + e * 2048) + lane;
      int acc[8]; f32x4 y[2];
#pragma unroll
      for (int j = 0; j < 8; ++j) acc[j] = sa[j * 64];
      const int na = arr[s];
      __builtin_amdgcn_sched_barrier(0);
      __builtin_amdgcn_s_setprio(2);
      const bool direct = (dirm >> s) & 1, packed = (pkm >> s) & 1;
      const unsigned inf = __builtin_amdgcn_readlane(info, s), hd = (inf & 0xffff) | h << 5 | call << 31;
      unsigned slot = (inf >> 16) + h;
      if (packed && !direct) asm volatile("s_mov_b64 exec, 1\n\tglobal_atomic_add %0, %1, %3, %2 sc0 sc1\n\ts_mov_b64 exec, -1"
                                          : "+v"(slot) : "v"(0), "s"(myws + TK_OFF + (4 + d) * 128), "v"(1) : "memory");
      char* pk = direct ? dws + PK_OFF + (rank & 3) * (512 * 1024) : myws + PL_OFF + (d & 3) * (512 * 1024);
      char* o = oc + ((unsigned)__builtin_amdgcn_readlane(c16, s) / 2 + h * 128);
      const unsigned vo = (lane & 15) * (NL * 4) + (lane & 48);
      __builtin_amdgcn_sched_barrier(0);
      const bool bad = (__builtin_amdgcn_readfirstlane(na) >> 8) != 0;
      if (__builtin_expect(neg, 0)) {
        asm volatile("");
#pragma unroll
        for (int j = 0; j < 8; ++j) acc[j] = -acc[j]; }
      auto getslot = [&]() { asm volatile("s_waitcnt vmcnt(0)" : "+v"(slot) :: "memory");
        return (__builtin_amdgcn_readfirstlane(slot) - (direct ? 0 : (call - 1) * NSL)) & 511; };
      int n64 = 0;
      if (packed && !bad) {
        int n = 0; unsigned kb = 0, pos = 0, nib = 0, t; u64 mk[8]; const int* v = acc;
#pragma unroll
        for (int j = 0; j < 8; ++j) { mk[j] = __ballot(v[j] > 0); n += __builtin_popcountll(mk[j]); pos = mbcnt(mk[j], pos); }
        const int nb = 68 + ((n + 1) & ~1) * 2, e0 = nb + ((n + 7) >> 3) * 4;
        int eb = e0;
        *(LAS unsigned*)(scr + nb + lane * 4) = 0;
        LAS char* lp = scr + 68 + 2 * pos;
#define WR(j, k) "s_mov_b64 exec, %" #k "\n\tds_write_b16 %0, %" #j "\n\tv_bfe_u32 %2, %" #j ", 16, 4\n\tv_add_u32 %0, 2, %0\n\tv_lshl_or_b32 %1, %1, 4, %2\n\t"
        asm volatile(WR(3, 11) WR(4, 12) WR(5, 13) WR(6, 14) WR(7, 15) WR(8, 16) WR(9, 17) WR(10, 18) "s_mov_b64 exec, -1"
                     : "+v"(lp), "+v"(nib), "=&v"(t)
                     : "v"(v[0]), "v"(v[1]), "v"(v[2]), "v"(v[3]), "v"(v[4]), "v"(v[5]), "v"(v[6]), "v"(v[7]),
                       "s"(mk[0]), "s"(mk[1]), "s"(mk[2]), "s"(mk[3]), "s"(mk[4]), "s"(mk[5]), "s"(mk[6]), "s"(mk[7]) : "memory");
        LAS unsigned* np = (LAS unsigned*)(scr + nb) + (pos >> 3);
        const unsigned sh0 = (pos & 7) * 4;
        const u64 nv = (u64)nib << sh0;
        __hip_atomic_fetch_or(np, (unsigned)nv, __ATOMIC_RELAXED, __HIP_MEMORY_SCOPE_WAVEFRONT);
        __hip_atomic_fetch_or(np + 1, (unsigned)(nv >> 32), __ATOMIC_RELAXED, __HIP_MEMORY_SCOPE_WAVEFRONT);
        sfor<16>([&](auto ii) { constexpr int i = ii; wlane<i>(kb, mk[i / 2] >> i % 2 * 32); });
        if (lane < 16) *(LAS unsigned*)(scr + 4 + lane * 4) = kb;
        const int mx = max(max(max(v[0], v[1]), max(v[2], v[3])), max(max(v[4], v[5]), max(v[6], v[7])));
        if (__ballot(mx >= 1 << 20)) {
#pragma unroll
          for (int j = 0; j < 8; ++j) mk[j] = __ballot(v[j] >= 1 << 20);
#pragma unroll
          for (int j = 0; j < 8; ++j) if (__builtin_expect(mk[j] != 0, 0)) {
            if (v[j] >= 1 << 20) *(LAS unsigned*)(scr + eb + 4 * mbcnt(mk[j], 0)) = v[j] << 9 | (lane & 15) << 5 | (j & 4) << 2 | (lane & 48) >> 2 | (j & 3);
            eb += 4 * __builtin_popcountll(mk[j]); }
          if (__ballot(mx >= 1 << 23) || eb - e0 >= 256) eb = 2048; }
        asm volatile("" ::: "memory");
        if (eb <= 1024) {
          n64 = (eb + 63) >> 6;
          u32x4 d0 = *(LAS u32x4*)(scr + lane * 16);
          char* q = pk + getslot() * 1024;
          const u64 lm = ~0ull >> (64 - n64 * 4);
          asm("v_writelane_b32 %0, %1, 0" : "+v"(d0[0]) : "s"(hd | n << 22 | (eb - e0) << 14 | n64));
          if (__ballot(max(max(d0[0], d0[1]), max(d0[2], d0[3])) == MARK) & lm) n64 = 0;
          else asm volatile("s_mov_b64 exec, %3\n\tglobal_store_dwordx4 %0, %1, %2 sc0 sc1\n\ts_mov_b64 exec, -1\n\ts_nop 0" :: "v"(lane * 16), "v"(d0), "s"(q), "s"(lm) : "memory");
        }
      }
      if (n64 == 0) {
#pragma unroll
        for (int j = 0; j < 8; ++j) y[j >> 2][j & 3] = fmaxf((float)acc[j] * scale, 0.f);
        if (__builtin_expect(bad, 0)) { asm volatile(""); const float qn = __builtin_nanf(""); y[0] = f32x4{qn, qn, qn, qn}; y[1] = y[0]; }
        asm volatile("global_store_dwordx4 %0, %1, %3 sc0 sc1\n\tglobal_store_dwordx4 %0, %2, %3 offset:64 sc0 sc1\n\ts_nop 1"
                     :: "v"(vo), "v"(y[0]), "v"(y[1]), "s"(o) : "memory");
      }
      if (packed) {
        if (n64 == 0) { const unsigned q = getslot(); if (lane < 16) fstore((unsigned*)(pk + q * 1024) + lane, hd); }
      }
      if (todo) {
        const int s = __builtin_ctz(todo);
        if (HINT && sure >> 15 && (late >> s) & 1)
#pragma nounroll
          for (int i = 0; i < 64 && __builtin_amdgcn_s_getreg(7 | 3 << 11); ++i)
            if ((__builtin_amdgcn_readfirstlane(*(volatile LAS int*)(arr + s)) & 0xff) >= thr) { ready |= 1u << s; break; }
      }
    }
  }
  const int ug = (128 - HB - 1 + NG - grp % NG) % NG - 3, gi = wid * NUW + ug * 16 + ns;
  if (ug < 0 || gi >= 4 * NFL + 48 * RB || (MODE & 4)) return;
  __builtin_amdgcn_s_setprio(0);
  if (!pkm) { const unsigned* mf = (const unsigned*)(myws + CF_OFF) + (ns & 12) * 8 + lane % (2 * NG);
    for (unsigned it = 1; __ballot(fload(mf) < call); ++it) {
      if ((it & 63) == 0 && expired(tmo, call, t0)) { bad = true; break; }
      __builtin_amdgcn_s_sleep(4); } }
  char* out = (char*)a.out;
  float sv = scale;
  asm volatile("" : "+s"(out)); asm volatile("" : "+v"(sv));
  const unsigned vo = (lane & 15) * (NL * 4) + (lane & 48);
  const u64 tend = bad ? 0 : t0 + DEADLINE;
  u32x4 p[4];
#define MX4(x) "v_max3_u32 v[" #x "], v[" #x "], v[" #x "+1], v[" #x "+2]\n\tv_max_u32 v[" #x "], v[" #x "], v[" #x "+3]\n\t"
#define LATE "s_add_u32 %[n], %[n], 0x1000000\n\ts_cbranch_scc0 " LB(1) "\n\ts_memrealtime vcc\n\ts_waitcnt lgkmcnt(0)\n\ts_sub_u32 vcc_lo, %[tl], vcc_lo\n\ts_subb_u32 vcc_hi, %[th], vcc_hi\n\ts_cbranch_scc0 " LB(1)
  if (gi >= 4 * NFL) {
    const int t = gi - 4 * NFL, sn = (t >> 2) & 3;
    const int ti = ((HB - RB + (t >> 4) / 3) * 4 + (sn ^ ((t >> 4) % 3 + 1))) * 16 + (t & 15);
    char* o = out + (size_t)((((rank ^ 4) & 4) | ((ti >> 4) & 3)) * MS + (ti >> 6) * 16) * (NL * 4) + (ti & 15) * 128;
#define LD2(x) "global_load_dwordx4 v[" #x ":" #x "+3], %[vo], %[o] sc0 sc1\n\tglobal_load_dwordx4 v[" #x "+4:" #x "+7], %[vo], %[o] offset:64 sc0 sc1\n\t"
#define CK2(x) "s_sleep %[sl]\n\ts_waitcnt vmcnt(2)\n\t" MX4(x) "v_max3_u32 v[" #x "], v[" #x "], v[" #x "+4], v[" #x "+5]\n\tv_max3_u32 v[" #x "], v[" #x "], v[" #x "+6], v[" #x "+7]\n\t" \
               "v_cmp_eq_u32 vcc, -1, v[" #x "]\n\ts_cbranch_vccz " LB(2) "\n\t"
    unsigned n;
    asm volatile("s_mov_b32 %[n], 0\n\ts_nop 4\n\t" LD2(0) "s_sleep %[sl]" AT(1) LD2(8) CK2(0) LD2(0) CK2(8) LATE AT(2)
                 : "={v[0:3]}"(p[0]), "={v[4:7]}"(p[1]), "={v[8:11]}"(p[2]), "={v[12:15]}"(p[3]), [n] "=&s"(n)
                 : [vo] "{v16}"(vo), [o] "s"(o), [tl] "s"((unsigned)tend), [th] "s"((unsigned)(tend >> 32)), [sl] "n"(SL) : "memory", "scc", "vcc");
    __builtin_amdgcn_endpgm();
  }
  const unsigned so = PK_OFF + ((gi & 12) * 128 + (gi >> 4) * 4 + (gi & 3)) * 1024;
  u64 lm; unsigned w[8], hd, ok, n;
#pragma unroll
  for (int j = 0; j < 8; ++j) { w[j] = 0; keep(w[j]); }
#define LD1(x) "global_load_dwordx4 v[" #x ":" #x "+3], %[vo], %[o] sc0 sc1\n\t"
#define CK1(x, n) "s_sleep %[sl]\n\ts_waitcnt vmcnt(1)\n\tv_readfirstlane_b32 %[hd], v" #x "\n\tv_max3_u32 v8, v[" #x "], v[" #x "+1], v[" #x "+2]\n\tv_max_u32 v8, v8, v[" #x "+3]\n\t" \
                  "s_lshl_b32 %[ok], %[hd], 6\n\ts_and_b32 %[ok], %[ok], 0x7c0\n\tv_cmp_gt_u32 %[lm], %[ok], %[vo]\n\tv_cmp_eq_u32 vcc, -1, v8\n\ts_and_b64 vcc, vcc, %[lm]\n\ts_cbranch_scc0 " LB(n) "\n\t"
  asm volatile("s_mov_b32 %[n], 0\n\ts_nop 4\n\t" LD1(0) "s_sleep %[sl]" AT(1) LD1(4) CK1(0, 3) LD1(0) CK1(4, 2) LATE "\n\ts_mov_b32 %[ok], -1"
               AT(2) "v_mov_b32 v8, v4\n\tv_mov_b32 v9, v5\n\tv_mov_b32 v10, v6\n\tv_mov_b32 v11, v7\n\ts_branch " LB(4)
               AT(3) "v_mov_b32 v8, v0\n\tv_mov_b32 v9, v1\n\tv_mov_b32 v10, v2\n\tv_mov_b32 v11, v3" AT(4)
               : "={v[0:3]}"(p[0]), "={v[4:7]}"(p[1]), "={v[8:11]}"(p[2]), [hd] "=&s"(hd), [ok] "=&s"(ok), [n] "=&s"(n), [lm] "=&s"(lm)
               : [vo] "{v16}"(lane * 16), [o] "s"(myws + so), [tl] "s"((unsigned)tend), [th] "s"((unsigned)(tend >> 32)), [sl] "n"(SL) : "memory", "scc", "vcc");
  hd = __builtin_amdgcn_readfirstlane(hd);
  if (ok == 0) __builtin_amdgcn_endpgm();
  __builtin_amdgcn_s_setprio(2);
  *(LAS u32x4*)(scr + lane * 16) = p[2];
  asm volatile("" ::: "memory");
  unsigned pos = 0; u64 mk[8];
  sfor<8>([&](auto jj) { constexpr int j = jj;
    mk[j] = (unsigned)__builtin_amdgcn_readlane(p[2][(2 * j + 1) % 4], (2 * j + 1) / 4) | (u64)(unsigned)__builtin_amdgcn_readlane(p[2][(2 * j + 2) % 4], (2 * j + 2) / 4) << 32;
    pos = mbcnt(mk[j], pos); });
  const int nv = (hd >> 22) & 511, nb = 68 + ((nv + 1) & ~1) * 2;
  const LAS unsigned* np = (const LAS unsigned*)(scr + nb) + (pos >> 3);
  LAS char* lp = scr + 68 + 2 * pos;
#define RD(j, k) "s_mov_b64 exec, %" #k "\n\tds_read_u16 %" #j ", %8\n\tv_add_u32 %8, 2, %8\n\t"
  u64 nn; unsigned ev;
  asm volatile("ds_read2_b32 %0, %2 offset1:1\n\tds_read_b32 %1, %3" : "=&v"(nn), "=&v"(ev) : "v"(np), "v"(scr + nb + ((nv + 7) >> 3) * 4 + lane * 4) : "memory");
  asm volatile(RD(0, 9) RD(1, 10) RD(2, 11) RD(3, 12) RD(4, 13) RD(5, 14) RD(6, 15) RD(7, 16) "s_mov_b64 exec, -1"
               : "+v"(w[0]), "+v"(w[1]), "+v"(w[2]), "+v"(w[3]), "+v"(w[4]), "+v"(w[5]), "+v"(w[6]), "+v"(w[7]), "+v"(lp)
               : "s"(mk[0]), "s"(mk[1]), "s"(mk[2]), "s"(mk[3]), "s"(mk[4]), "s"(mk[5]), "s"(mk[6]), "s"(mk[7]) : "memory");
  const int ti = (hd >> 5) & 2047, tb = ti >> 6, ts = (tb < HB ? (rank ^ 4) & 4 : rank & 4) | ((ti >> 4) & 3);
  char* o = out + (size_t)(ts * MS + tb * 16) * (NL * 4) + (ti & 15) * 128;
  asm volatile("" : "+s"(o));
  asm volatile("s_waitcnt lgkmcnt(8)" : "+v"(nn), "+v"(ev) :: "memory");
  unsigned nib = __builtin_amdgcn_alignbit((unsigned)(nn >> 32), (unsigned)nn, (pos & 7) * 4);
  float ey = (float)(ev >> 9) * sv;
  unsigned eo = (ev & 0x1e0) * (NL * 4 / 32) + (ev & 31) * 4;
  asm volatile("" : "+v"(ey), "+v"(eo));
#define HI(j, k) "s_mov_b64 exec, %" #k "\n\tv_and_b32_sdwa %" #j ", %8, %17 dst_sel:WORD_1 dst_unused:UNUSED_PRESERVE src0_sel:DWORD src1_sel:DWORD\n\tv_lshrrev_b32 %8, 4, %8\n\t"
  asm volatile("s_waitcnt lgkmcnt(0)\n\t" HI(7, 16) HI(6, 15) HI(5, 14) HI(4, 13) HI(3, 12) HI(2, 11) HI(1, 10) HI(0, 9) "s_mov_b64 exec, -1"
               : "+v"(w[0]), "+v"(w[1]), "+v"(w[2]), "+v"(w[3]), "+v"(w[4]), "+v"(w[5]), "+v"(w[6]), "+v"(w[7]), "+v"(nib)
               : "s"(mk[0]), "s"(mk[1]), "s"(mk[2]), "s"(mk[3]), "s"(mk[4]), "s"(mk[5]), "s"(mk[6]), "s"(mk[7]), "v"(15) : "memory");
  f32x4 y[2];
#pragma unroll
  for (int j = 0; j < 8; ++j) y[j >> 2][j & 3] = mul0((float)w[j], sv);
  if (__builtin_expect((int)ok < 0, 0)) { asm volatile(""); const float qn = __builtin_nanf(""); y[0] = f32x4{qn, qn, qn, qn}; y[1] = y[0]; hd = 0; }
#define UST(X) asm volatile("global_store_dwordx4 %0, %1, %3" X "\n\tglobal_store_dwordx4 %0, %2, %3 offset:64" X "\n\t" \
               "s_bfm_b64 exec, %6, 0\n\tglobal_store_dword %4, %5, %3" X "\n\ts_mov_b64 exec, -1\n\ts_nop 1" \
               :: "v"(vo), "v"(y[0]), "v"(y[1]), "s"(o), "v"(eo), "v"(ey), "s"(hd >> 16), "v"(p[0]), "v"(p[1]) : "memory")
  if ((MODE & 384) == 0) UST(" sc0 sc1"); else if ((MODE & 384) == 128) UST(""); else if ((MODE & 384) == 256) UST(" nt"); else UST(" sc1");
}

extern "C" void kernel_launch(int rank, int nranks, ncclComm_t comm, hipStream_t stream,
                              void* const* d_in, const int* in_sizes, int n_in,
                              void* d_out, int out_size, const int* in_dtypes, int out_dtype,
                              void* const* d_out_all, void* const* d_ws_all, size_t ws_bytes) {
  if (nranks != NR || (unsigned)rank >= NR || ws_bytes < WS_NEED) { fprintf(stderr, "unexpected configuration\n"); return; }
  constexpr int HB = 22, WN = 6, WI = 9, WX = 9, NG = 12;
  constexpr int GRID = 3 * WI + 4 * WX + NG * 16;
  static_assert(GRID <= 256, "one workgroup per CU, all resident");
  auto kfn = ag_gemm<HB, WN, WI, WX, NG, 18, 7, 1, 8, 3, 1, 6, 0, 0x05004040>;
  static bool init = false;
  if (!init) {
    HIP_CHECK(hipFuncSetAttribute((const void*)kfn, hipFuncAttributeMaxDynamicSharedMemorySize, LDS_BYTES));
    init = true;
  }
  Args a = {};
  a.x = (const char*)d_in[0]; a.w = (const char*)d_in[1];
  a.sx = (const float*)d_in[2]; a.sw = (const float*)d_in[3];
  a.ws = (char*)d_ws_all[rank]; a.out = (float*)d_out_all[rank]; a.rank = rank;
  for (int d = 0; d < NR; ++d) {
    char* w = (char*)d_ws_all[rank ^ d];
    a.p[d] = Peer{w, (float*)d_out_all[rank ^ d], d < 4 ? a.x : a.w + ((rank ^ d) & 3) * 128,
                  w + XG_OFF + rank * SHARD};
  }
  hipLaunchKernelGGL(kfn, dim3(GRID), dim3(64 * CW), LDS_BYTES, stream, a);
  HIP_CHECK(hipGetLastError());
}
```
